```python
import numpy as np

kernel_name = "gpu_gemm_ar_m4096_k4096_n16384_e5m2e5m2_dqsilu"


ABSMAX_RTOL = 0.05

COMPUTE_DTYPE = "float32"
EPILOGUE = "dequant_silu"
IN_QUANT = None
OUT_QUANT = None
IN_FP8 = "e5m2e5m2"
IN_DTYPES = {"x": "e5m2", "w_mat": "e5m2",
             "scale_x": "float32", "scale_w": "float32"}
WORLD_SIZE = 8

def _snap_fp8(x, mant_bits, max_abs, min_normal):
    a = np.abs(x.astype(np.float64))
    _m, _e = np.frexp(a)
    q = np.where(a >= min_normal, np.ldexp(1.0, _e - 1 - mant_bits),
                 min_normal * 2.0 ** -mant_bits)
    snapped = np.minimum(np.round(a / q) * q, max_abs)
    return (np.sign(x) * snapped).astype(np.float32)

def setup_inputs():

    x = _snap_fp8(np.random.randn(4096, 4096).astype(np.float32), 2, 57344.0, 2.0 ** -14)
    w_mat = _snap_fp8(np.random.randn(4096, 16384).astype(np.float32), 2, 57344.0, 2.0 ** -14)
    scale_x = (0.02 * (1.0 + np.random.rand(1))).astype(np.float32)
    scale_w = (0.02 * (1.0 + np.random.rand(1))).astype(np.float32)
    return {"x": x, "w_mat": w_mat, "scale_x": scale_x, "scale_w": scale_w}

def reference(x, w_mat, scale_x, scale_w):
    acc = (x @ w_mat).astype(np.float32)
    y = (acc * (scale_x * scale_w)).astype(np.float32)
    return (y / (1.0 + np.exp(-np.clip(y, -60.0, 60.0)))).astype(np.float32)
```

```cpp
#include <hip/hip_runtime.h>
#include <rccl/rccl.h>
#include <cstdio>
#include <algorithm>

#define HIP_CHECK(expr) do { hipError_t _e = (expr); if (_e != hipSuccess) { \
    fprintf(stderr, "HIP error %s at %s:%d\n", hipGetErrorString(_e), __FILE__, __LINE__); \
    return; } } while (0)

namespace {
#define LAS __attribute__((address_space(3)))
#define GAS __attribute__((address_space(1)))
typedef int v8i __attribute__((ext_vector_type(8)));
typedef int v4i __attribute__((ext_vector_type(4)));
typedef int v2i __attribute__((ext_vector_type(2)));
typedef unsigned u32x4 __attribute__((ext_vector_type(4)));
typedef float f32x4 __attribute__((ext_vector_type(4)));
typedef unsigned long long u64;

constexpr int M = 4096, N = 16384, KL = 512, NR = 8;
constexpr int NPUSH = 16, NWG = 256, NTHR = 512;
constexpr int NPAN = 15;
constexpr int NCHUNK = 4 * NPAN + 4 * 64;
constexpr unsigned XS_SLAB = 256u * 2048u;
constexpr unsigned WS_SLAB = 1024u * 2048u;
constexpr unsigned X_BYTES = 32u * XS_SLAB, W_BYTES = 32u * WS_SLAB, PAR_BYTES = X_BYTES + W_BYTES;
constexpr unsigned CTR_OFF = 0;
constexpr unsigned TMO_OFF = 32768;
constexpr unsigned FX_OFF = 65536;
constexpr unsigned FW_OFF = 73728;
constexpr unsigned DBG_OFF = 131072;
constexpr unsigned DATA_OFF = 1u << 20;
constexpr size_t WS_NEED = (size_t)DATA_OFF + 2ull * PAR_BYTES;
constexpr u64 DEADLINE_TICKS = 100000000ull;
constexpr int A0B = 16384, A1B = 20480, ABUF = A0B + A1B, BH = 16384;
constexpr int LDS_B = 2 * ABUF, LDS_SNAP = LDS_B + 4 * BH, LDS_MBOX = LDS_SNAP + 2048, LDS_BYTES = LDS_MBOX + 64;
#define PUSH_WIN 4

struct Params {
    const unsigned char* x; const unsigned char* w; const float* sx; const float* sw; float* out;
    unsigned char* ws[NR];
    int rank; int pad;
    unsigned order[NCHUNK + 4];
};

__device__ __forceinline__ unsigned rfl(unsigned v) { return (unsigned)__builtin_amdgcn_readfirstlane((int)v); }
__device__ __forceinline__ unsigned char* rfl_ptr(unsigned char* p) {
    return (unsigned char*)(((u64)rfl((unsigned)((u64)p >> 32)) << 32) | (u64)rfl((unsigned)(u64)p));
}
__device__ __forceinline__ f32x4 silu4(f32x4 a, float s, float sl) {
    f32x4 o;
#pragma unroll
    for (int j = 0; j < 4; ++j) {
        float t = a[j] * sl;
        t = __builtin_amdgcn_fmed3f(t, -86.56170245f, 86.56170245f);
        const float e = __builtin_amdgcn_exp2f(t);
        o[j] = (a[j] * s) * __builtin_amdgcn_rcpf(1.0f + e);
    }
    return o;
}

#define WAIT_V(n) asm volatile("s_waitcnt vmcnt(" #n ")" ::: "memory")
#define WAIT_L(n) asm volatile("s_waitcnt lgkmcnt(" #n ")" ::: "memory")
#define BAR __builtin_amdgcn_s_barrier()
#define SCHED __builtin_amdgcn_sched_barrier(0)
#define STAMP(i)

__device__ __forceinline__ void pusher(const Params& P, LAS unsigned char* lds, unsigned call, int wid, int lane) {
    LAS unsigned* cnt = (LAS unsigned*)(lds + 65536);
    if (threadIdx.x < 64) cnt[threadIdx.x] = 0u;
    __syncthreads();
    const unsigned hh = blockIdx.x & 1u;
    const int rank = P.rank;
    unsigned char* dst[NR];
#pragma unroll
    for (int q = 0; q < NR; ++q) {
        const unsigned p = (unsigned)(rank + q + wid + (int)blockIdx.x) & 7u;
        dst[q] = rfl_ptr(P.ws[p] + DATA_OFF + (size_t)(call & 1u) * PAR_BYTES);
    }
    unsigned char* const flag_base = P.ws[lane & 7];
    const unsigned wbuf = (unsigned)wid * 8192u;
    const unsigned w_src = (unsigned)((lane >> 1) * N + (lane & 1) * 16 + wid * 32);
    const unsigned x_src = (unsigned)((lane & 15) * KL + (lane >> 4) * 16);
    const unsigned tr_off = (unsigned)(((lane >> 4) * 16 + ((lane & 15) >> 1)) * 32 + (lane & 1) * 8);
    const int c0 = blockIdx.x >> 1;
    const int nun = (NCHUNK - c0 + 7) >> 3;

    auto stage = [&](int i) {
        const unsigned e = P.order[c0 + 8 * i], j = (e >> 8) & 3u, pan = e & 127u;
        LAS unsigned char* b = lds + wbuf + (unsigned)(i & 1) * 4096u;
        if (e & 0x8000u) {
            const unsigned char* s = P.w + (size_t)(j * 128u + hh * 64u) * N + pan * 256u + w_src;
            __builtin_amdgcn_global_load_lds((const GAS void*)s, (LAS void*)b, 16, 0, 0);
            __builtin_amdgcn_global_load_lds((const GAS void*)(s + 32 * N), (LAS void*)(b + 1024), 16, 0, 0);
        } else {
            const unsigned R = pan == NPAN - 1 ? 18u : 17u;
            const unsigned char* s = P.x + (size_t)((pan * 17u + (unsigned)wid) * 16u) * KL + j * 128u + hh * 64u + x_src;
            __builtin_amdgcn_global_load_lds((const GAS void*)s, (LAS void*)b, 16, 0, 0);
            __builtin_amdgcn_global_load_lds((const GAS void*)(s + 128 * KL), (LAS void*)(b + 1024), 16, 0, 0);
            if ((unsigned)wid + 16u < R) __builtin_amdgcn_global_load_lds((const GAS void*)(s + 256 * KL), (LAS void*)(b + 2048), 16, 0, 0);
        }
    };
    auto signal = [&](int i) {
        unsigned old = 0;
        if (lane == 0) {
            const unsigned a = (unsigned)(size_t)(cnt + i), one = 1u;
            asm volatile("ds_add_rtn_u32 %0, %1, %2\n\ts_waitcnt lgkmcnt(0)" : "=&v"(old) : "v"(a), "v"(one) : "memory");
        }
        if (rfl(old) == 7u) {
            const unsigned e = P.order[c0 + 8 * i], j = (e >> 8) & 3u, pan = e & 127u, slot = j * 8u + (unsigned)rank;
            const unsigned fo = (e & 0x8000u) ? FW_OFF + (pan & 15u) * 512u + (pan >> 4) * 128u + slot * 4u : FX_OFF + pan * 128u + slot * 4u;
            if (lane < NR) __hip_atomic_fetch_add((unsigned*)(flag_base + fo), 1u, __ATOMIC_RELAXED, __HIP_MEMORY_SCOPE_SYSTEM);
        }
    };
    auto send = [&](u32x4 v, unsigned off) {
#pragma unroll
        for (int q = 0; q < NR; ++q) {
            asm volatile("global_store_dwordx4 %0, %1, %2 sc0 sc1\n\ts_nop 1" :: "v"(off), "v"(v), "s"(dst[q]) : "memory");
            asm volatile("s_waitcnt vmcnt(%0)" :: "n"(PUSH_WIN) : "memory");
        }
    };

    stage(0);
    WAIT_V(0);
    for (int i = 0; i < nun; ++i) {
        if (i + 1 < nun) stage(i + 1);
        const unsigned e = P.order[c0 + 8 * i], j = (e >> 8) & 3u, pan = e & 127u, slot = j * 8u + (unsigned)rank;
        const unsigned b = wbuf + (unsigned)(i & 1) * 4096u;
        const bool isw = (e & 0x8000u) != 0u;
        const unsigned R = pan == NPAN - 1 ? 18u : 17u;
        const int np = (!isw && (unsigned)wid + 16u < R) ? 3 : 2;
        const unsigned d0 = isw ? X_BYTES + slot * WS_SLAB + (pan * 16u + (unsigned)wid * 2u) * 2048u : slot * XS_SLAB + (pan * 17u + (unsigned)wid) * 2048u;
        const unsigned dstep = isw ? 2048u : 16384u;
        for (int k = 0; k < np; ++k) {
            u32x4 v;
            if (isw) {
                v2i lo, hi;
                const unsigned a = (unsigned)(size_t)lds + b + tr_off + (unsigned)k * 16u;
                asm volatile("ds_read_b64_tr_b8 %0, %2\n\tds_read_b64_tr_b8 %1, %2 offset:256\n\ts_waitcnt lgkmcnt(0)" : "=&v"(lo), "=&v"(hi) : "v"(a) : "memory");
                v = (u32x4){(unsigned)lo.x, (unsigned)lo.y, (unsigned)hi.x, (unsigned)hi.y};
            } else {
                const unsigned a = (unsigned)(size_t)lds + b + (unsigned)k * 1024u + (unsigned)lane * 16u;
                asm volatile("ds_read_b128 %0, %1\n\ts_waitcnt lgkmcnt(0)" : "=&v"(v) : "v"(a) : "memory");
            }
            send(v, d0 + (unsigned)k * dstep + hh * 1024u + (unsigned)lane * 16u);
            if (k == 0 && i > 0) signal(i - 1);
        }
    }
    WAIT_V(0);
    signal(nun - 1);
}

__global__ __launch_bounds__(NTHR, 2) void fused_kernel(const Params P) {
    extern __shared__ __attribute__((aligned(16))) unsigned char shm[];
    LAS unsigned char* lds = (LAS unsigned char*)shm;
    const int tid = threadIdx.x, wid = __builtin_amdgcn_readfirstlane(tid >> 6), lane = tid & 63;
    unsigned char* const wsl = P.ws[P.rank];
    STAMP(0);
    unsigned call;
    {
        unsigned t = 0;
        if (lane == 0) t = __hip_atomic_fetch_add((unsigned*)(wsl + CTR_OFF) + blockIdx.x * 16 + wid, 1u, __ATOMIC_RELAXED, __HIP_MEMORY_SCOPE_AGENT);
        call = rfl(t) + 1u;
    }
    if (blockIdx.x < NPUSH) { pusher(P, lds, call, wid, lane); STAMP(1); return; }

    const int wr = wid >> 2, wc = wid & 3;
    const int idx = (int)(blockIdx.x - NPUSH) >> 3, pan = idx % NPAN, g = (int)(blockIdx.x & 7) * 2 + idx / NPAN;
    const bool big = pan == NPAN - 1;
    const bool extra = (wr == 0) || big;
    const unsigned thr = 2u * call;
    unsigned char* const dbase = wsl + DATA_OFF + (size_t)(call & 1u) * PAR_BYTES;
    const __amdgpu_buffer_rsrc_t rsX = __builtin_amdgcn_make_buffer_rsrc((void*)dbase, 0, X_BYTES, 0x00020000);
    const __amdgpu_buffer_rsrc_t rsW = __builtin_amdgcn_make_buffer_rsrc((void*)(dbase + X_BYTES), 0, W_BYTES, 0x00020000);
    const __amdgpu_buffer_rsrc_t rsF = __builtin_amdgcn_make_buffer_rsrc((void*)wsl, 0, DATA_OFF, 0x00020000);
    const __amdgpu_buffer_rsrc_t rsO = __builtin_amdgcn_make_buffer_rsrc((void*)P.out, 0, (unsigned)M * N * 4u, 0x00020000);
    const unsigned voff = (unsigned)(wid * 2048 + lane * 16);
    const unsigned ldsw = (unsigned)wid * 2048u;
    int aoff0 = wr * 8192 + lane * 16, aoff1 = A0B + wr * 10240 + lane * 16, boff = LDS_B + wc * 4096 + lane * 16;
    asm volatile("" : "+v"(aoff0), "+v"(aoff1), "+v"(boff));
    const unsigned snap_src = lane < 32 ? FW_OFF + (unsigned)g * 512u + (unsigned)lane * 16u : FX_OFF + (unsigned)pan * 128u + (unsigned)(lane & 7) * 16u;
    const unsigned a_pan = (unsigned)pan * 17u * 2048u;

#define DMA(rs, ldsoff, vo, so, imm) __builtin_amdgcn_raw_ptr_buffer_load_lds(rs, (LAS void*)(lds + (ldsoff)), 16, vo, so, imm, 17)
#define STAGE_A0(b, so) do { DMA(rsX, (b) * ABUF + ldsw, voff, so, 0); DMA(rsX, (b) * ABUF + ldsw, voff, so, 1024); } while (0)
#define STAGE_A1(b, so) do { DMA(rsX, (b) * ABUF + A0B + ldsw, voff, (so) + A0B, 0); DMA(rsX, (b) * ABUF + A0B + ldsw, voff, (so) + A0B, 1024); \
        if (wid == 4 || (wid == 5 && big)) { DMA(rsX, (b) * ABUF + A0B + 8192 + ldsw, voff, (so) + A0B + 8192, 0); DMA(rsX, (b) * ABUF + A0B + 8192 + ldsw, voff, (so) + A0B + 8192, 1024); } } while (0)
#define STAGE_B(b, h, so) do { DMA(rsW, LDS_B + ((b) * 2 + (h)) * BH + ldsw, voff, (so) + (h) * BH, 0); DMA(rsW, LDS_B + ((b) * 2 + (h)) * BH + ldsw, voff, (so) + (h) * BH, 1024); } while (0)
#define LDF(off) __builtin_shufflevector(*(const LAS v4i*)(lds + (off)), *(const LAS v4i*)(lds + (off) + 1024), 0, 1, 2, 3, 4, 5, 6, 7)
#define LDA0(b) do { _Pragma("unroll") for (int m = 0; m < 4; ++m) At[m] = LDF((b) * ABUF + aoff0 + m * 2048); } while (0)
#define LDA1(b) do { _Pragma("unroll") for (int m = 0; m < 4; ++m) At[m] = LDF((b) * ABUF + aoff1 + m * 2048); if (extra) Ax = LDF((b) * ABUF + aoff1 + 4 * 2048); } while (0)
#define LDB(dst, b, h) do { _Pragma("unroll") for (int n = 0; n < 2; ++n) dst[n] = LDF(((b) * 2 + (h)) * BH + boff + n * 2048); } while (0)
#define MF(c, bb, aa) asm volatile("v_mfma_f32_16x16x128_f8f6f4 %0, %1, %2, %0 cbsz:1 blgp:1" : "+v"(c) : "v"(bb), "v"(aa))
#define MMA(ai, bj, Bt) do { __builtin_amdgcn_s_setprio(1); _Pragma("unroll") for (int m = 0; m < 4; ++m) _Pragma("unroll") for (int n = 0; n < 2; ++n) MF(acc[ai][bj][m][n], Bt[n], At[m]); \
        if ((ai) == 1 && extra) { MF(accx[bj][0], Bt[0], Ax); MF(accx[bj][1], Bt[1], Ax); } __builtin_amdgcn_s_setprio(0); } while (0)

    f32x4 acc[2][2][4][2], accx[2][2];
    v8i At[4], Ax, B0[2], B1[2];
#pragma unroll
    for (int a = 0; a < 2; ++a)
#pragma unroll
        for (int b = 0; b < 2; ++b) {
            accx[a][b] = (f32x4){0.f, 0.f, 0.f, 0.f};
#pragma unroll
            for (int m = 0; m < 4; ++m)
#pragma unroll
                for (int n = 0; n < 2; ++n) acc[a][b][m][n] = (f32x4){0.f, 0.f, 0.f, 0.f};
        }
    Ax = (v8i){0, 0, 0, 0, 0, 0, 0, 0};

    unsigned xm = 0u, wm = 0u, used = 0u, bad = 0u;
    int ts = 0;
    auto slow_wait = [&]() {
        if (wid == 0) {
            const unsigned* fp = (const unsigned*)(wsl + (lane < 32 ? FW_OFF + (unsigned)g * 512u + (unsigned)ts * 128u + (unsigned)lane * 4u
                                                                    : FX_OFF + (unsigned)pan * 128u + (unsigned)(lane - 32) * 4u));
            unsigned* tmo = (unsigned*)(wsl + TMO_OFF);
            const u64 t0 = __builtin_amdgcn_s_memrealtime();
            u64 mk; unsigned dead = 0u;
            for (;;) {
                const unsigned v = __hip_atomic_load(fp, __ATOMIC_RELAXED, __HIP_MEMORY_SCOPE_SYSTEM);
                mk = __builtin_amdgcn_ballot_w64((int)(v - thr) >= 0);
                if (__builtin_popcount((unsigned)mk & (unsigned)(mk >> 32) & ~used) >= 2) break;
                if (rfl(__hip_atomic_load(tmo, __ATOMIC_RELAXED, __HIP_MEMORY_SCOPE_AGENT)) == call || __builtin_amdgcn_s_memrealtime() - t0 > DEADLINE_TICKS) {
                    if (lane == 0) __hip_atomic_store(tmo, call, __ATOMIC_RELAXED, __HIP_MEMORY_SCOPE_AGENT);
                    mk = ~0ull; dead = 1u; break;
                }
                __builtin_amdgcn_s_sleep(4);
            }
            if (lane == 0) *(LAS u32x4*)(lds + LDS_MBOX) = (u32x4){(unsigned)mk, (unsigned)(mk >> 32), dead, 0u};
            asm volatile("s_waitcnt vmcnt(0) lgkmcnt(0)" ::: "memory");
        }
        asm volatile("" ::: "memory"); BAR; asm volatile("" ::: "memory");
        const u32x4 mb = *(const LAS u32x4*)(lds + LDS_MBOX);
        wm |= rfl(mb.x); xm |= rfl(mb.y); bad |= rfl(mb.z);
    };
    unsigned sA, sB;
    auto pick = [&]() {
        unsigned av = xm & wm & ~used;
        if (__builtin_popcount(av) < 2) { slow_wait(); av = xm & wm & ~used; }
        sA = (unsigned)__builtin_ctz(av); av &= av - 1u; sB = (unsigned)__builtin_ctz(av);
        used |= (1u << sA) | (1u << sB);
    };

    const float s = P.sx[0] * P.sw[0], sl = -s * 1.44269504089f;
    pick();
    STAMP(1);
    unsigned b_pan = (unsigned)g * 32768u;
    unsigned a1 = sB * XS_SLAB + a_pan;
    {
        const unsigned a0 = sA * XS_SLAB + a_pan, b0 = sA * WS_SLAB + b_pan, b1 = sB * WS_SLAB + b_pan;
        STAGE_A1(0, a0); STAGE_B(0, 0, b0); STAGE_A0(0, a0); STAGE_B(0, 1, b0);
        if (wr == 1) BAR;
        WAIT_V(2); BAR;
        STAGE_B(1, 0, b1); STAGE_A0(1, a1); STAGE_B(1, 1, b1);
        WAIT_V(6); BAR;
    }
    unsigned snapv = 0u;
    int snap_ts = -1;
    for (int r = 0; r < 4; ++r) {
        for (int it = 0; it < 16; ++it) {
            unsigned a2, b2, a3, b3;
            if (it == 15) { ++ts; wm = 0u; used = 0u; b_pan += 16u * 32768u; }
            if (ts < 4) {
                if (snap_ts == ts) { const u64 mk = __builtin_amdgcn_ballot_w64((int)(snapv - thr) >= 0); wm |= (unsigned)mk; xm |= (unsigned)(mk >> 32); }
                pick();
                a2 = sA * XS_SLAB + a_pan; b2 = sA * WS_SLAB + b_pan; a3 = sB * XS_SLAB + a_pan; b3 = sB * WS_SLAB + b_pan;
            } else {
                a2 = a3 = a_pan; b2 = b3 = (unsigned)g * 32768u;
            }
            const unsigned snap = LDS_SNAP + (unsigned)(it & 1) * 1024u;
            LDB(B0, 0, 0); SCHED; LDA0(0); STAGE_A1(1, a1);
            if (wid == 0) DMA(rsF, snap, snap_src, 0, 0);
            WAIT_L(8); BAR; WAIT_L(0); SCHED; MMA(0, 0, B0); BAR; SCHED;
            LDB(B1, 0, 1); STAGE_B(0, 0, b2);
            BAR; WAIT_L(0); SCHED; MMA(0, 1, B1); BAR; SCHED;
            LDA1(0); STAGE_A0(0, a2);
            BAR; WAIT_L(0); SCHED; MMA(1, 0, B0); BAR; SCHED;
            STAGE_B(0, 1, b2);
            WAIT_V(6); BAR; SCHED; MMA(1, 1, B1); BAR; SCHED;
            LDB(B0, 1, 0); SCHED; LDA0(1); STAGE_A1(0, a2);
            WAIT_L(8); BAR; WAIT_L(0); SCHED; MMA(0, 0, B0); BAR; SCHED;
            LDB(B1, 1, 1); STAGE_B(1, 0, b3);
            BAR; WAIT_L(0); SCHED; MMA(0, 1, B1); BAR; SCHED;
            LDA1(1); STAGE_A0(1, a3);
            BAR; WAIT_L(0); SCHED; MMA(1, 0, B0); BAR; SCHED;
            STAGE_B(1, 1, b3);
            snap_ts = (it == 14) ? ts + 1 : ts;
            snapv = *(const LAS unsigned*)(lds + snap + (lane < 32 ? (unsigned)(snap_ts & 3) * 128u + (unsigned)lane * 4u : 384u + (unsigned)lane * 4u));
            WAIT_V(6); BAR; SCHED; MMA(1, 1, B1); BAR; SCHED;
            a1 = a3;
        }
        STAMP(2 + 2 * r);
        asm volatile("s_nop 15\n\ts_nop 15" ::: "memory");
        {
            int tz = threadIdx.x; asm volatile("" : "+v"(tz));
            const unsigned ov = (unsigned)(tz & 15) * (unsigned)(N * 4) + (unsigned)((tz >> 4) & 3) * 16u;
            const unsigned ob = (unsigned)(pan * 17 * 16) * (unsigned)(N * 4) + (unsigned)((r * 16 + g) * 256 + wc * 32) * 4u;
            const float se = bad ? __builtin_nanf("") : s;
#pragma unroll
            for (int ai = 0; ai < 2; ++ai)
#pragma unroll
                for (int m = 0; m < 4; ++m) {
                    const unsigned rowo = ob + (unsigned)((ai ? 8 + wr * 5 : wr * 4) + m) * (unsigned)(16 * N * 4);
#pragma unroll
                    for (int bj = 0; bj < 2; ++bj)
#pragma unroll
                        for (int n = 0; n < 2; ++n) {
                            const f32x4 o = silu4(acc[ai][bj][m][n], se, sl);
                            __builtin_amdgcn_raw_buffer_store_b128(__builtin_bit_cast(u32x4, o), rsO, ov + bj * 512 + n * 64, rowo, 0);
                            acc[ai][bj][m][n] = (f32x4){0.f, 0.f, 0.f, 0.f};
                        }
                }
            if (extra) {
                const unsigned rowo = ob + (unsigned)(8 + wr * 5 + 4) * (unsigned)(16 * N * 4);
#pragma unroll
                for (int bj = 0; bj < 2; ++bj)
#pragma unroll
                    for (int n = 0; n < 2; ++n) {
                        const f32x4 o = silu4(accx[bj][n], se, sl);
                        __builtin_amdgcn_raw_buffer_store_b128(__builtin_bit_cast(u32x4, o), rsO, ov + bj * 512 + n * 64, rowo, 0);
                        accx[bj][n] = (f32x4){0.f, 0.f, 0.f, 0.f};
                    }
            }
        }
        STAMP(3 + 2 * r);
    }
    WAIT_V(0);
    if (wr == 0) BAR;
    STAMP(10);
}

void build_order(int s, unsigned* out) {
#define O_DELTA 1.0
#define O_TAU0 2.0
#define O_LEAD 0.0
#define O_XS 8.0
#define O_TR 45.0
    struct It { double key; unsigned short code; };
    It it[NCHUNK]; int n = 0;
    const double delta = O_DELTA, tau0 = O_TAU0, lead = O_LEAD, xs = O_XS, Tr = O_TR, T1 = 32 * tau0;
    for (int j = 0; j < 4; ++j)
        for (int p = 0; p < NPAN; ++p)
            it[n++] = It{(8 * j - lead) * tau0 + xs * tau0 * (((p + 2 * s) % NPAN) / (double)NPAN) - 0.01, (unsigned short)((j << 8) | p)};
    for (int r = 0; r < 4; ++r)
        for (int g = 0; g < 16; ++g)
            for (int j = 0; j < 4; ++j) {
                const int pos = 8 * j + ((s + g) & 7);
                const double key = (r == 0 ? g * delta + pos * tau0 : T1 + (r - 1) * Tr + g * delta + pos * (Tr / 34.0)) + 1e-4 * g;
                it[n++] = It{key, (unsigned short)(0x8000 | (j << 8) | (r * 16 + g))};
            }
    std::stable_sort(it, it + NCHUNK, [](const It& a, const It& b) { return a.key < b.key; });
    for (int i = 0; i < NCHUNK; ++i) out[i] = it[i].code;
}
}

extern "C" void kernel_launch(int rank, int nranks, ncclComm_t comm,
                              hipStream_t stream,
                              void* const* d_in, const int* in_sizes, int n_in,
                              void* d_out, int out_size,
                              const int* in_dtypes, int out_dtype,
                              void* const* d_out_all,
                              void* const* d_ws_all, size_t ws_bytes) {
    (void)comm; (void)in_dtypes; (void)out_dtype; (void)d_out_all;
    if (nranks != NR || (unsigned)rank >= (unsigned)NR || n_in != 4 || ws_bytes < WS_NEED || out_size != M * N ||
        in_sizes[0] != M * KL || in_sizes[1] != KL * N) {
        fprintf(stderr, "kernel_launch: unexpected problem shape\n");
        return;
    }
    Params P;
    P.x = (const unsigned char*)d_in[0]; P.w = (const unsigned char*)d_in[1];
    P.sx = (const float*)d_in[2]; P.sw = (const float*)d_in[3];
    P.out = (float*)d_out;
    for (int p = 0; p < NR; ++p) P.ws[p] = (unsigned char*)d_ws_all[p];
    P.rank = rank; P.pad = 0;
    build_order(rank, P.order);
    for (int i = NCHUNK; i < NCHUNK + 4; ++i) P.order[i] = 0;
    HIP_CHECK(hipFuncSetAttribute((const void*)fused_kernel, hipFuncAttributeMaxDynamicSharedMemorySize, LDS_BYTES));
    hipLaunchKernelGGL(fused_kernel, dim3(NWG), dim3(NTHR), LDS_BYTES, stream, P);
    HIP_CHECK(hipGetLastError());
}
```

```cpp
#include <hip/hip_runtime.h>
#include <rccl/rccl.h>
#include <cstdio>
#include <algorithm>

#define HIP_CHECK(expr) do { hipError_t _e = (expr); if (_e != hipSuccess) { \
    fprintf(stderr, "HIP error %s at %s:%d\n", hipGetErrorString(_e), __FILE__, __LINE__); \
    return; } } while (0)

namespace {
#define LAS __attribute__((address_space(3)))
#define GAS __attribute__((address_space(1)))
typedef int v8i __attribute__((ext_vector_type(8)));
typedef int v4i __attribute__((ext_vector_type(4)));
typedef int v2i __attribute__((ext_vector_type(2)));
typedef unsigned u32x4 __attribute__((ext_vector_type(4)));
typedef float f32x4 __attribute__((ext_vector_type(4)));
typedef unsigned long long u64;

constexpr int M = 4096, N = 16384, KL = 512, NR = 8;
constexpr int NPUSH = 16, NWG = 256, NTHR = 512;
constexpr int NPAN = 15;
constexpr int NCHUNK = 4 * NPAN + 4 * 64;
constexpr unsigned XS_SLAB = 256u * 2048u;
constexpr unsigned WS_SLAB = 1024u * 2048u;
constexpr unsigned X_BYTES = 32u * XS_SLAB, W_BYTES = 32u * WS_SLAB, PAR_BYTES = X_BYTES + W_BYTES;
constexpr unsigned CTR_OFF = 0;
constexpr unsigned TMO_OFF = 32768;
constexpr unsigned FX_OFF = 65536;
constexpr unsigned FW_OFF = 73728;
constexpr unsigned DBG_OFF = 131072;
constexpr unsigned DATA_OFF = 1u << 20;
constexpr size_t PART_OFF = (size_t)DATA_OFF + 2ull * PAR_BYTES;
constexpr size_t WS_NEED = PART_OFF + (size_t)(NWG - NPUSH) * 3 * 8 * 36864;
constexpr u64 DEADLINE_TICKS = 100000000ull;
constexpr int A0B = 16384, A1B = 20480, ABUF = A0B + A1B, BH = 16384;
constexpr int LDS_B = 2 * ABUF, LDS_SNAP = LDS_B + 4 * BH, LDS_MBOX = LDS_SNAP + 2048, LDS_BYTES = LDS_MBOX + 64;
#define PARK 0
#define OUT_POL " nt"
#define PUSH_WIN 7

struct Params {
    const unsigned char* x; const unsigned char* w; const float* sx; const float* sw; float* out;
    unsigned char* ws[NR];
    int rank; int pad;
    unsigned order[NCHUNK + 4];
};

__device__ __forceinline__ unsigned rfl(unsigned v) { return (unsigned)__builtin_amdgcn_readfirstlane((int)v); }
__device__ __forceinline__ unsigned char* rfl_ptr(unsigned char* p) {
    return (unsigned char*)(((u64)rfl((unsigned)((u64)p >> 32)) << 32) | (u64)rfl((unsigned)(u64)p));
}
typedef float f32x2 __attribute__((ext_vector_type(2)));
__device__ __forceinline__ float mul1(float a, float b) { float r; asm("v_mul_f32 %0, %1, %2" : "=v"(r) : "s"(b), "v"(a)); return r; }
__device__ __forceinline__ f32x4 silu4(float a0, float a1, float a2, float a3, float sl, float c) {
    const f32x2 t0 = {mul1(a0, sl), mul1(a1, sl)}, t1 = {mul1(a2, sl), mul1(a3, sl)};
    const float L = 86.56170245f;
    f32x2 e0, e1;
    e0.x = __builtin_amdgcn_exp2f(__builtin_amdgcn_fmed3f(t0.x, -L, L)); e0.y = __builtin_amdgcn_exp2f(__builtin_amdgcn_fmed3f(t0.y, -L, L));
    e1.x = __builtin_amdgcn_exp2f(__builtin_amdgcn_fmed3f(t1.x, -L, L)); e1.y = __builtin_amdgcn_exp2f(__builtin_amdgcn_fmed3f(t1.y, -L, L));
    const f32x2 cv = {c, c};
    const f32x2 d0 = __builtin_elementwise_fma(e0, cv, cv), d1 = __builtin_elementwise_fma(e1, cv, cv);
    const f32x2 r0 = {__builtin_amdgcn_rcpf(d0.x), __builtin_amdgcn_rcpf(d0.y)}, r1 = {__builtin_amdgcn_rcpf(d1.x), __builtin_amdgcn_rcpf(d1.y)};
    const f32x2 o0 = t0 * r0, o1 = t1 * r1;
    return (f32x4){o0.x, o0.y, o1.x, o1.y};
}

#define WAIT_V(n) asm volatile("s_waitcnt vmcnt(" #n ")" ::: "memory")
#define WAIT_L(n) asm volatile("s_waitcnt lgkmcnt(" #n ")" ::: "memory")
#define BAR __builtin_amdgcn_s_barrier()
#define SCHED __builtin_amdgcn_sched_barrier(0)
#define STAMP(i)

struct Mover {
    const Params& P; LAS unsigned char* lds; int wid, lane;
    unsigned w_src, x_src, tr_off, sc_off, w_dst;
    __device__ __forceinline__ Mover(const Params& P_, LAS unsigned char* lds_, int wid_, int lane_) : P(P_), lds(lds_), wid(wid_), lane(lane_) {
        w_src = (unsigned)(((wid >> 2) * 32 + (lane >> 2)) * N + (wid & 3) * 64 + (lane & 3) * 16);
        x_src = (unsigned)((lane & 15) * KL + (lane >> 4) * 16);
        tr_off = (unsigned)(((lane & 15) >> 1) * 64 + (lane >> 4) * 16 + (lane & 1) * 8);
        sc_off = (unsigned)(2048 + (lane & 3) * 512 + (lane >> 2) * 16);
        w_dst = (unsigned)(((wid & 3) * 2 + (lane >> 5)) * 2048 + (wid >> 2) * 512 + (lane & 31) * 16);
    }
    __device__ __forceinline__ void stage(int u, unsigned b) const {
        const unsigned e = P.order[u >> 1], hh = (unsigned)u & 1u, j = (e >> 8) & 3u, pan = e & 127u;
        if (e & 0x8000u) {
            const unsigned char* s = P.w + (size_t)(j * 128u + hh * 64u) * N + pan * 256u + w_src;
            __builtin_amdgcn_global_load_lds((const GAS void*)s, (LAS void*)(lds + b), 16, 0, 0);
            __builtin_amdgcn_global_load_lds((const GAS void*)(s + 16 * N), (LAS void*)(lds + b + 1024), 16, 0, 0);
        } else {
            const unsigned R = pan == NPAN - 1 ? 18u : 17u;
            const unsigned char* s = P.x + (size_t)((pan * 17u + (unsigned)wid) * 16u) * KL + j * 128u + hh * 64u + x_src;
            __builtin_amdgcn_global_load_lds((const GAS void*)s, (LAS void*)(lds + b), 16, 0, 0);
            __builtin_amdgcn_global_load_lds((const GAS void*)(s + 128 * KL), (LAS void*)(lds + b + 1024), 16, 0, 0);
            if ((unsigned)wid + 16u < R) __builtin_amdgcn_global_load_lds((const GAS void*)(s + 256 * KL), (LAS void*)(lds + b + 2048), 16, 0, 0);
        }
    }
    __device__ __forceinline__ int pieces(int u) const {
        const unsigned e = P.order[u >> 1], pan = e & 127u;
        return (!(e & 0x8000u) && (unsigned)wid + 16u < (pan == NPAN - 1 ? 18u : 17u)) ? 3 : 2;
    }
    __device__ __forceinline__ u32x4 fetch(int u, unsigned b, int k, unsigned& off) const {
        const unsigned e = P.order[u >> 1], hh = (unsigned)u & 1u, j = (e >> 8) & 3u, pan = e & 127u, slot = j * 8u + (unsigned)P.rank;
        const unsigned lb = (unsigned)(size_t)lds + b;
        unsigned a = lb + (unsigned)k * 1024u + (unsigned)lane * 16u;
        if (e & 0x8000u) {
            if (k == 0) {
                v2i g0, g1, g2, g3;
                const unsigned ra = lb + tr_off, wa = lb + sc_off;
                asm volatile("ds_read_b64_tr_b8 %0, %4\n\tds_read_b64_tr_b8 %1, %4 offset:512\n\t"
                             "ds_read_b64_tr_b8 %2, %4 offset:1024\n\tds_read_b64_tr_b8 %3, %4 offset:1536\n\ts_waitcnt lgkmcnt(0)\n\t"
                             "ds_write_b64 %5, %0\n\tds_write_b64 %5, %1 offset:8\n\tds_write_b64 %5, %2 offset:256\n\tds_write_b64 %5, %3 offset:264\n\t"
                             "s_waitcnt lgkmcnt(0)"
                             : "=&v"(g0), "=&v"(g1), "=&v"(g2), "=&v"(g3) : "v"(ra), "v"(wa) : "memory");
            }
            a += 2048u;
            off = X_BYTES + slot * WS_SLAB + (pan * 16u + (unsigned)k * 8u) * 2048u + w_dst;
        } else {
            off = slot * XS_SLAB + (pan * 17u + (unsigned)wid + (unsigned)k * 8u) * 2048u + (unsigned)lane * 16u;
        }
        u32x4 v;
        asm volatile("ds_read_b128 %0, %1\n\ts_waitcnt lgkmcnt(0)" : "=&v"(v) : "v"(a) : "memory");
        off += hh * 1024u;
        return v;
    }
    __device__ __forceinline__ unsigned flag_off(int u) const {
        const unsigned e = P.order[u >> 1], j = (e >> 8) & 3u, pan = e & 127u, slot = j * 8u + (unsigned)P.rank;
        return (e & 0x8000u) ? FW_OFF + (pan & 15u) * 512u + (pan >> 4) * 128u + slot * 4u : FX_OFF + pan * 128u + slot * 4u;
    }
};
#define STORE_WT(off, v, base) asm volatile("global_store_dwordx4 %0, %1, %2 sc0 sc1\n\ts_nop 1" :: "v"(off), "v"(v), "s"(base) : "memory")

__device__ __forceinline__ void pusher(const Params& P, LAS unsigned char* lds, unsigned call, int wid, int lane) {
    const Mover mv(P, lds, wid, lane);
    const unsigned wbuf = (unsigned)wid * 8192u;
    const int u0 = blockIdx.x, nun = (2 * NCHUNK - u0 + NPUSH - 1) / NPUSH;
    mv.stage(u0, wbuf);
    __builtin_amdgcn_s_setprio(3);
    unsigned char* dst[NR - 1];
#pragma unroll
    for (int q = 0; q < NR - 1; ++q) {
        const unsigned p = (unsigned)(P.rank + 1 + (q + wid + (int)blockIdx.x) % (NR - 1)) & 7u;
        dst[q] = rfl_ptr(P.ws[p] + DATA_OFF + (size_t)(call & 1u) * PAR_BYTES);
    }
    unsigned char* const flag_base = P.ws[(P.rank + 1 + lane) & 7];
    auto signal = [&](int i) {
        if (wid == (i & 7)) {
            if (lane < NR - 1) __hip_atomic_fetch_add((unsigned*)(flag_base + mv.flag_off(u0 + NPUSH * i)), 1u, __ATOMIC_RELAXED, __HIP_MEMORY_SCOPE_SYSTEM);
        }
    };
    WAIT_V(0);
    for (int i = 0; i < nun; ++i) {
        const int u = u0 + NPUSH * i;
        if (i + 1 < nun) mv.stage(u + NPUSH, wbuf + (unsigned)((i + 1) & 1) * 4096u);
        const int np = mv.pieces(u);
        for (int k = 0; k < np; ++k) {
            unsigned off;
            const u32x4 v = mv.fetch(u, wbuf + (unsigned)(i & 1) * 4096u, k, off);
#pragma unroll
            for (int q = 0; q < NR - 1; ++q) {
                STORE_WT(off, v, dst[q]);
                asm volatile("s_waitcnt vmcnt(%0)" :: "n"(PUSH_WIN) : "memory");
                if (q == PUSH_WIN - 1 && k == 0 && i > 0) { BAR; signal(i - 1); }
            }
        }
    }
    WAIT_V(0);
    BAR;
    signal(nun - 1);
}
static_assert(PUSH_WIN >= 1 && PUSH_WIN <= NR - 1, "window");

__device__ __forceinline__ void local_copy(const Params& P, LAS unsigned char* lds, unsigned call, int wid, int lane) {
    const Mover mv(P, lds, wid, lane);
    constexpr int NG = NWG - NPUSH;
    const int li = (int)blockIdx.x - NPUSH, nun = li + 2 * NG < 2 * NCHUNK ? 3 : 2;
    const unsigned wbuf = (unsigned)wid * 12288u;
    for (int i = 0; i < nun; ++i) mv.stage(li + NG * i, wbuf + (unsigned)i * 4096u);
    unsigned char* const wsl = P.ws[P.rank];
    unsigned char* const dst = rfl_ptr(wsl + DATA_OFF + (size_t)(call & 1u) * PAR_BYTES);
    WAIT_V(0);
    for (int i = 0; i < nun; ++i) {
        const int np = mv.pieces(li + NG * i);
        for (int k = 0; k < np; ++k) {
            unsigned off;
            const u32x4 v = mv.fetch(li + NG * i, wbuf + (unsigned)i * 4096u, k, off);
            STORE_WT(off, v, dst);
        }
    }
    WAIT_V(0);
    __syncthreads();
    if (wid < nun && lane == 0) __hip_atomic_fetch_add((unsigned*)(wsl + mv.flag_off(li + NG * wid)), 1u, __ATOMIC_RELAXED, __HIP_MEMORY_SCOPE_SYSTEM);
}

__global__ __launch_bounds__(NTHR, 2) void fused_kernel(const Params P) {
    extern __shared__ __attribute__((aligned(16))) unsigned char shm[];
    LAS unsigned char* lds = (LAS unsigned char*)shm;
    const int tid = threadIdx.x, wid = __builtin_amdgcn_readfirstlane(tid >> 6), lane = tid & 63;
    unsigned char* const wsl = P.ws[P.rank];
    STAMP(0);
    unsigned call;
    {
        unsigned t = 0;
        if (lane == 0) t = __hip_atomic_fetch_add((unsigned*)(wsl + CTR_OFF) + blockIdx.x * 16 + wid, 1u, __ATOMIC_RELAXED, __HIP_MEMORY_SCOPE_AGENT);
        call = rfl(t) + 1u;
    }
    if (blockIdx.x < NPUSH) { pusher(P, lds, call, wid, lane); STAMP(1); return; }
    local_copy(P, lds, call, wid, lane);
    STAMP(11);

    const int wr = wid >> 2, wc = wid & 3;
    const int idx = (int)(blockIdx.x - NPUSH) >> 3, cell = (int)(blockIdx.x & 7) * 6 + idx / 5;
    const int pan5 = (cell >> 4) * 5, i5 = idx % 5, g = cell & 15;
    auto pan_of = [&](int r) { const int t = i5 + r; return pan5 + (t >= 5 ? t - 5 : t); };
    const unsigned thr = 2u * call;
    unsigned char* const dbase = wsl + DATA_OFF + (size_t)(call & 1u) * PAR_BYTES;
    const __amdgpu_buffer_rsrc_t rsX = __builtin_amdgcn_make_buffer_rsrc((void*)dbase, 0, X_BYTES, 0x00020000);
    const __amdgpu_buffer_rsrc_t rsW = __builtin_amdgcn_make_buffer_rsrc((void*)(dbase + X_BYTES), 0, W_BYTES, 0x00020000);
    const __amdgpu_buffer_rsrc_t rsF = __builtin_amdgcn_make_buffer_rsrc((void*)wsl, 0, DATA_OFF, 0x00020000);
    const unsigned voff = (unsigned)(wid * 2048 + lane * 16);
    const unsigned ldsw = (unsigned)wid * 2048u;
    int aoff0 = wr * 8192 + lane * 16, aoff1 = A0B + wr * 10240 + lane * 16, boff = LDS_B + wc * 4096 + lane * 16;
    asm volatile("" : "+v"(aoff0), "+v"(aoff1), "+v"(boff));
    const unsigned snap_src = lane < 32 ? FW_OFF + (unsigned)g * 512u + (unsigned)lane * 16u
                                        : FX_OFF + (unsigned)pan_of((lane >> 3) & 3) * 128u + (unsigned)(lane & 7) * 16u;
    const unsigned snap_rd = lane < 32 ? (unsigned)lane * 4u : 384u + (unsigned)lane * 4u;
    const unsigned lmask = 0x01010101u << P.rank;

#define DMA(rs, ldsoff, vo, so, imm) __builtin_amdgcn_raw_ptr_buffer_load_lds(rs, (LAS void*)(lds + (ldsoff)), 16, vo, so, imm, 17)
#define STAGE_A0(b, so) do { DMA(rsX, (b) * ABUF + ldsw, voff, so, 0); DMA(rsX, (b) * ABUF + ldsw, voff, so, 1024); } while (0)
#define STAGE_A1(b, so, more) do { DMA(rsX, (b) * ABUF + A0B + ldsw, voff, (so) + A0B, 0); DMA(rsX, (b) * ABUF + A0B + ldsw, voff, (so) + A0B, 1024); \
        if (more) { DMA(rsX, (b) * ABUF + A0B + 8192 + ldsw, voff, (so) + A0B + 8192, 0); DMA(rsX, (b) * ABUF + A0B + 8192 + ldsw, voff, (so) + A0B + 8192, 1024); } } while (0)
#define STAGE_B(b, h, so) do { DMA(rsW, LDS_B + ((b) * 2 + (h)) * BH + ldsw, voff, (so) + (h) * BH, 0); DMA(rsW, LDS_B + ((b) * 2 + (h)) * BH + ldsw, voff, (so) + (h) * BH, 1024); } while (0)
#define LDF(off) __builtin_shufflevector(*(const LAS v4i*)(lds + (off)), *(const LAS v4i*)(lds + (off) + 1024), 0, 1, 2, 3, 4, 5, 6, 7)
#define LDA0(b) do { _Pragma("unroll") for (int m = 0; m < 4; ++m) At[m] = LDF((b) * ABUF + aoff0 + m * 2048); } while (0)
#define LDA1(b) do { _Pragma("unroll") for (int m = 0; m < 4; ++m) At[m] = LDF((b) * ABUF + aoff1 + m * 2048); if (extra) Ax = LDF((b) * ABUF + aoff1 + 4 * 2048); } while (0)
#define LDB(dst, b, h) do { _Pragma("unroll") for (int n = 0; n < 2; ++n) dst[n] = LDF(((b) * 2 + (h)) * BH + boff + n * 2048); } while (0)
#define MF(c, bb, aa) asm volatile("v_mfma_f32_16x16x128_f8f6f4 %0, %1, %2, %0 cbsz:1 blgp:1" : "+v"(c) : "v"(aa), "v"(bb))
#define MMA(ai, bj, Bt) do { __builtin_amdgcn_s_setprio(1); _Pragma("unroll") for (int m = 0; m < 4; ++m) _Pragma("unroll") for (int n = 0; n < 2; ++n) MF(acc[ai * 4 + m][bj * 2 + n], Bt[n], At[m]); \
        if ((ai) == 1 && extra) { MF(acc[8][bj * 2], Bt[0], Ax); MF(acc[8][bj * 2 + 1], Bt[1], Ax); } __builtin_amdgcn_s_setprio(0); } while (0)

    f32x4 acc[9][4];
    v8i At[4], Ax, B0[2], B1[2];
#pragma unroll
    for (int m = 0; m < 9; ++m)
#pragma unroll
        for (int n = 0; n < 4; ++n) acc[m][n] = (f32x4){0.f, 0.f, 0.f, 0.f};
    Ax = (v8i){0, 0, 0, 0, 0, 0, 0, 0};

    auto tile_of = [&](int sg) { return sg < PARK ? sg + 1 : (sg - PARK) & 3; };
    auto used_of = [&](int sg) { return sg < PARK ? ~lmask : ((sg - PARK >= 1 && sg - PARK <= PARK) ? lmask : 0u); };
    unsigned xm = 0u, wm = 0u, used = used_of(0), bad = 0u;
    int ts = tile_of(0);
    unsigned a_pan = (unsigned)pan_of(ts) * 17u * 2048u, b_pan = (unsigned)(ts * 16 + g) * 32768u;
    bool moreS = wid == 4 || (wid == 5 && pan_of(ts) == NPAN - 1);
    auto slow_wait = [&]() {
        if (wid == 0) {
            const unsigned* fp = (const unsigned*)(wsl + (lane < 32 ? FW_OFF + (unsigned)g * 512u + (unsigned)ts * 128u + (unsigned)lane * 4u
                                                                    : FX_OFF + (unsigned)pan_of(ts) * 128u + (unsigned)(lane - 32) * 4u));
            unsigned* tmo = (unsigned*)(wsl + TMO_OFF);
            const u64 t0 = __builtin_amdgcn_s_memrealtime();
            u64 mk; unsigned dead = 0u;
            for (;;) {
                const unsigned v = __hip_atomic_load(fp, __ATOMIC_RELAXED, __HIP_MEMORY_SCOPE_SYSTEM);
                mk = __builtin_amdgcn_ballot_w64((int)(v - thr) >= 0);
                if (__builtin_popcount((unsigned)mk & (unsigned)(mk >> 32) & ~used) >= 2) break;
                if (rfl(__hip_atomic_load(tmo, __ATOMIC_RELAXED, __HIP_MEMORY_SCOPE_AGENT)) == call || __builtin_amdgcn_s_memrealtime() - t0 > DEADLINE_TICKS) {
                    if (lane == 0) __hip_atomic_store(tmo, call, __ATOMIC_RELAXED, __HIP_MEMORY_SCOPE_AGENT);
                    mk = ~0ull; dead = 1u; break;
                }
                __builtin_amdgcn_s_sleep(4);
            }
            if (lane == 0) *(LAS u32x4*)(lds + LDS_MBOX) = (u32x4){(unsigned)mk, (unsigned)(mk >> 32), dead, 0u};
            asm volatile("s_waitcnt vmcnt(0) lgkmcnt(0)" ::: "memory");
        }
        asm volatile("" ::: "memory"); BAR; asm volatile("" ::: "memory");
        const u32x4 mb = *(const LAS u32x4*)(lds + LDS_MBOX);
        wm |= rfl(mb.x); xm |= rfl(mb.y); bad |= rfl(mb.z);
    };
    unsigned sA, sB;
    auto pick = [&]() {
        unsigned av = xm & wm & ~used;
        if (__builtin_popcount(av) < 2) { slow_wait(); av = xm & wm & ~used; }
        sA = (unsigned)__builtin_ctz(av); av &= av - 1u; sB = (unsigned)__builtin_ctz(av);
        used |= (1u << sA) | (1u << sB);
    };

    const float sl = __builtin_bit_cast(float, rfl(__builtin_bit_cast(unsigned, (P.sx[0] * P.sw[0]) * -1.44269504089f)));
    pick();
    STAMP(1);
    unsigned a1 = sB * XS_SLAB + a_pan;
    bool more1 = moreS;
    {
        const unsigned a0 = sA * XS_SLAB + a_pan, b0 = sA * WS_SLAB + b_pan, b1 = sB * WS_SLAB + b_pan;
        STAGE_A1(0, a0, moreS); STAGE_B(0, 0, b0); STAGE_A0(0, a0); STAGE_B(0, 1, b0);
        if (wr == 1) BAR;
        WAIT_V(2); BAR;
        STAGE_B(1, 0, b1); STAGE_A0(1, a1); STAGE_B(1, 1, b1);
        WAIT_V(6); BAR;
        LDB(B0, 0, 0);
    }
    unsigned snapv = 0u;
    int snap_ts = -1;
    unsigned flip = 0u;
    for (int seg = 0; seg < PARK + 4; ++seg) {
        const int r = tile_of(seg), nit = seg < PARK ? 2 : ((r >= 1 && r <= PARK) ? 14 : 16), rn = tile_of(seg + 1);
        const int pan = pan_of(r);
        const bool extra = (wr == 0) || pan == NPAN - 1;
        for (int it = 0; it < nit; ++it) {
            unsigned a2, b2, a3, b3;
            if (it == nit - 1) {
                ts = rn; wm = 0u; xm = 0u; used = used_of(seg + 1);
                const int pn = pan_of(ts);
                a_pan = (unsigned)pn * 17u * 2048u; b_pan = (unsigned)(ts * 16 + g) * 32768u;
                moreS = wid == 4 || (wid == 5 && pn == NPAN - 1);
            }
            if (seg < PARK + 3 || it < nit - 1) {
                if (snap_ts == ts) { const u64 mk = __builtin_amdgcn_ballot_w64((int)(snapv - thr) >= 0); wm |= (unsigned)mk; xm |= (unsigned)(mk >> 32); }
                pick();
                a2 = sA * XS_SLAB + a_pan; b2 = sA * WS_SLAB + b_pan; a3 = sB * XS_SLAB + a_pan; b3 = sB * WS_SLAB + b_pan;
            } else {
                a2 = a3 = a_pan; b2 = b3 = b_pan;
            }
            flip ^= 1024u;
            const unsigned snap = LDS_SNAP + flip;
            LDA0(0); STAGE_A1(1, a1, more1);
            if (wid == 0) DMA(rsF, snap, snap_src, 0, 0);
            BAR; WAIT_L(0); SCHED; MMA(0, 0, B0); BAR; SCHED;
            LDB(B1, 0, 1); STAGE_B(0, 0, b2);
            BAR; WAIT_L(0); SCHED; MMA(0, 1, B1); BAR; SCHED;
            LDA1(0); STAGE_A0(0, a2);
            if (it == 0) WAIT_V(6); else WAIT_V(10);
            BAR; WAIT_L(0); SCHED; MMA(1, 0, B0); BAR; SCHED;
            LDB(B0, 1, 0); STAGE_B(0, 1, b2);
            WAIT_V(6); BAR; SCHED; MMA(1, 1, B1); BAR; SCHED;
            LDA0(1); STAGE_A1(0, a2, moreS);
            BAR; WAIT_L(0); SCHED; MMA(0, 0, B0); BAR; SCHED;
            LDB(B1, 1, 1); STAGE_B(1, 0, b3);
            BAR; WAIT_L(0); SCHED; MMA(0, 1, B1); BAR; SCHED;
            LDA1(1); STAGE_A0(1, a3);
            WAIT_V(10); BAR; WAIT_L(0); SCHED; MMA(1, 0, B0); BAR; SCHED;
            LDB(B0, 0, 0); STAGE_B(1, 1, b3);
            snap_ts = (it == nit - 2) ? rn : ts;
            snapv = *(const LAS unsigned*)(lds + snap + (unsigned)snap_ts * 128u + snap_rd);
            WAIT_V(6); BAR; SCHED; MMA(1, 1, B1); BAR; SCHED;
            a1 = a3; more1 = moreS;
        }
        if (seg >= PARK) STAMP(2 * (seg - PARK) + 2);
        SCHED; asm volatile("s_nop 15\n\ts_nop 15" ::: "memory"); SCHED;
        {
            int tz = threadIdx.x; asm volatile("" : "+v"(tz));
            const unsigned pv = (unsigned)(tz & 63) * 16u;
            const unsigned ov = (unsigned)((tz >> 4) & 3) * (unsigned)(4 * N * 4) + (unsigned)(tz & 15) * 16u;
            const char* pp = (const char*)rfl_ptr(wsl + PART_OFF + (size_t)(((blockIdx.x - NPUSH) * 3u + (unsigned)(seg < PARK ? seg : (seg < 2 * PARK ? seg - PARK : 0))) * 8u + (unsigned)wid) * 36864u);
            const char* ob = (const char*)rfl_ptr((unsigned char*)P.out + (size_t)(pan * 17 * 16) * (N * 4) + (size_t)((r * 16 + g) * 256 + wc * 64) * 4);
            u32x4 rsP = {(unsigned)(u64)pp, (unsigned)((u64)pp >> 32), (seg >= PARK && seg < 2 * PARK) ? 36864u : 0u, 0x00020000u};
            asm volatile("s_nop 4" : "+s"(ob), "+s"(pp), "+s"(rsP));
            const float cc = bad ? __builtin_nanf("") : -1.44269504089f;
#pragma unroll
            for (int m = 0; m < 9; ++m) {
                if (m < 8 || extra) {
                    if (seg < PARK) {
#define PST(n) asm volatile("global_store_dwordx4 %0, %1, %2 offset:%3" :: "v"(pv), "v"(acc[m][n]), "s"(pp + m * 4096), "n"(n * 1024) : "memory")
                        PST(0); PST(1); PST(2); PST(3);
                    } else {
                        const char* rowp = ob + (size_t)(m < 4 ? wr * 4 + m : 4 + wr * 5 + m) * (16 * N * 4);
#pragma unroll
                        for (int j = 0; j < 4; ++j) {
                            const f32x4 o = silu4(acc[m][0][j], acc[m][1][j], acc[m][2][j], acc[m][3][j], sl, cc);
                            asm volatile("global_store_dwordx4 %0, %1, %2" OUT_POL "\n\ts_nop 1" :: "v"(ov), "v"(o), "s"(rowp + (size_t)j * (N * 4)) : "memory");
                        }
                    }
                }
                asm volatile("s_nop 1\n\tbuffer_load_dwordx4 %0, %4, %5, %6 offen\n\tbuffer_load_dwordx4 %1, %4, %5, %6 offen offset:1024\n\t"
                             "buffer_load_dwordx4 %2, %4, %5, %6 offen offset:2048\n\tbuffer_load_dwordx4 %3, %4, %5, %6 offen offset:3072"
                             : "+v"(acc[m][0]), "+v"(acc[m][1]), "+v"(acc[m][2]), "+v"(acc[m][3]) : "v"(pv), "s"(rsP), "s"(m * 4096) : "memory");
                if (m == 0 && wr == 0) BAR;
            }
            WAIT_V(36);
        }
        if (seg >= PARK) STAMP(2 * (seg - PARK) + 3); else STAMP(12 + seg);
        if (wr == 1) BAR;
    }
    WAIT_V(0);
    if (wr == 0) BAR;
    STAMP(10);
}

void build_order(int s, unsigned* out) {
#define O_DELTA 0.0
#define O_TAU0 2.0
#define O_LEAD 0.0
#define O_XS 8.0
#define O_TR 45.0
    struct It { double key; unsigned short code; };
    It it[NCHUNK]; int n = 0;
    const double delta = O_DELTA, tau0 = O_TAU0, lead = O_LEAD, xs = O_XS, Tr = O_TR, T1 = 32 * tau0;
    for (int j = 0; j < 4; ++j)
        for (int p = 0; p < NPAN; ++p)
            it[n++] = It{(8 * j - lead) * tau0 + xs * tau0 * (((p + 2 * s) % NPAN) / (double)NPAN) - 0.01, (unsigned short)((j << 8) | p)};
    for (int r = 0; r < 4; ++r)
        for (int g = 0; g < 16; ++g)
            for (int j = 0; j < 4; ++j) {
                const int pos = 8 * j + ((s + g * 3 + (g >> 3) * 4) & 7);
                const double key = (r == 0 ? (g & 7) * delta + pos * tau0 : T1 + (r - 1) * Tr + (g & 7) * delta + pos * (Tr / 34.0)) + 1e-4 * g;
                it[n++] = It{key, (unsigned short)(0x8000 | (j << 8) | (r * 16 + g))};
            }
    std::stable_sort(it, it + NCHUNK, [](const It& a, const It& b) { return a.key < b.key; });
    for (int i = 0; i < NCHUNK; ++i) out[i] = it[i].code;
}
}

extern "C" void kernel_launch(int rank, int nranks, ncclComm_t comm,
                              hipStream_t stream,
                              void* const* d_in, const int* in_sizes, int n_in,
                              void* d_out, int out_size,
                              const int* in_dtypes, int out_dtype,
                              void* const* d_out_all,
                              void* const* d_ws_all, size_t ws_bytes) {
    (void)comm; (void)in_dtypes; (void)out_dtype; (void)d_out_all;
    if (nranks != NR || (unsigned)rank >= (unsigned)NR || n_in != 4 || ws_bytes < WS_NEED || out_size != M * N ||
        in_sizes[0] != M * KL || in_sizes[1] != KL * N) {
        fprintf(stderr, "kernel_launch: unexpected problem shape\n");
        return;
    }
    Params P;
    P.x = (const unsigned char*)d_in[0]; P.w = (const unsigned char*)d_in[1];
    P.sx = (const float*)d_in[2]; P.sw = (const float*)d_in[3];
    P.out = (float*)d_out;
    for (int p = 0; p < NR; ++p) P.ws[p] = (unsigned char*)d_ws_all[p];
    P.rank = rank; P.pad = 0;
    build_order(rank, P.order);
    for (int i = NCHUNK; i < NCHUNK + 4; ++i) P.order[i] = 0;
    HIP_CHECK(hipFuncSetAttribute((const void*)fused_kernel, hipFuncAttributeMaxDynamicSharedMemorySize, LDS_BYTES));
    hipLaunchKernelGGL(fused_kernel, dim3(NWG), dim3(NTHR), LDS_BYTES, stream, P);
    HIP_CHECK(hipGetLastError());
}
```

```cpp
#include <hip/hip_runtime.h>
#include <rccl/rccl.h>
#include <cstdio>
#include <algorithm>

#define HIP_CHECK(expr) do { hipError_t _e = (expr); if (_e != hipSuccess) { \
    fprintf(stderr, "HIP error %s at %s:%d\n", hipGetErrorString(_e), __FILE__, __LINE__); \
    return; } } while (0)

namespace {
#define LAS __attribute__((address_space(3)))
#define GAS __attribute__((address_space(1)))
typedef int v8i __attribute__((ext_vector_type(8)));
typedef int v4i __attribute__((ext_vector_type(4)));
typedef int v2i __attribute__((ext_vector_type(2)));
typedef unsigned u32x4 __attribute__((ext_vector_type(4)));
typedef float f32x4 __attribute__((ext_vector_type(4)));
typedef float f32x2 __attribute__((ext_vector_type(2)));
typedef unsigned long long u64;

constexpr int M = 4096, N = 16384, KL = 512, NR = 8;
constexpr int NPUSH = 16, NWG = 256, NGEMM = NWG - NPUSH, NTHR = 512;
constexpr int NCHUNK = 4 * 16 + 4 * 64;
constexpr int NUIN = 2 * NCHUNK;
constexpr int DPAN = 59, NOWN = 10;
constexpr int NUNIT = NUIN + NOWN * 16;
constexpr int SPSLOT = 15 + 14;
constexpr unsigned XS_SLAB = 256u * 2048u;
constexpr unsigned WS_SLAB = 1024u * 2048u;
constexpr unsigned X_BYTES = 32u * XS_SLAB, W_BYTES = 32u * WS_SLAB, PAR_BYTES = X_BYTES + W_BYTES;
constexpr unsigned CTR_OFF = 0;
constexpr unsigned TMO_OFF = 32768;
constexpr unsigned ROLE_OFF = 36864;
constexpr unsigned TICK_OFF = 40960;
constexpr unsigned TDONE_OFF = 45056;
constexpr unsigned DONE_OFF = 49152;
constexpr unsigned FX_OFF = 65536;
constexpr unsigned FW_OFF = 73728;
constexpr unsigned DBG_OFF = 131072;
constexpr unsigned DATA_OFF = 1u << 20;
constexpr size_t WS_NEED = (size_t)DATA_OFF + 2ull * PAR_BYTES;
constexpr u64 DEADLINE_TICKS = 100000000ull;
constexpr int HT = 16384, LDS_B = 4 * HT, LDS_SNAP = 8 * HT, LDS_MBOX = LDS_SNAP + 2048, LDS_BYTES = LDS_MBOX + 64;
#define OUT_POL " sc0 sc1"
#define SNAP_EARLY 1
#define PUSH_WIN 7

struct Params {
    const unsigned char* x; const unsigned char* w; const float* sx; const float* sw;
    unsigned char* ws[NR]; unsigned char* out[NR];
    int rank; int pad;
    unsigned order[NCHUNK];
};

__device__ __forceinline__ unsigned rfl(unsigned v) { return (unsigned)__builtin_amdgcn_readfirstlane((int)v); }
__device__ __forceinline__ unsigned char* rfl_ptr(unsigned char* p) {
    return (unsigned char*)(((u64)rfl((unsigned)((u64)p >> 32)) << 32) | (u64)rfl((unsigned)(u64)p));
}
__device__ __forceinline__ float mul1(float a, float b) { float r; asm("v_mul_f32 %0, %1, %2" : "=v"(r) : "s"(b), "v"(a)); return r; }
__device__ __forceinline__ f32x4 silu4(float a0, float a1, float a2, float a3, float sl, float c) {
    const f32x2 t0 = {mul1(a0, sl), mul1(a1, sl)}, t1 = {mul1(a2, sl), mul1(a3, sl)};
    const float L = 86.56170245f;
    f32x2 e0, e1;
    e0.x = __builtin_amdgcn_exp2f(__builtin_amdgcn_fmed3f(t0.x, -L, L)); e0.y = __builtin_amdgcn_exp2f(__builtin_amdgcn_fmed3f(t0.y, -L, L));
    e1.x = __builtin_amdgcn_exp2f(__builtin_amdgcn_fmed3f(t1.x, -L, L)); e1.y = __builtin_amdgcn_exp2f(__builtin_amdgcn_fmed3f(t1.y, -L, L));
    const f32x2 cv = {c, c};
    const f32x2 d0 = __builtin_elementwise_fma(e0, cv, cv), d1 = __builtin_elementwise_fma(e1, cv, cv);
    const f32x2 r0 = {__builtin_amdgcn_rcpf(d0.x), __builtin_amdgcn_rcpf(d0.y)}, r1 = {__builtin_amdgcn_rcpf(d1.x), __builtin_amdgcn_rcpf(d1.y)};
    const f32x2 o0 = t0 * r0, o1 = t1 * r1;
    return (f32x4){o0.x, o0.y, o1.x, o1.y};
}

#define WAIT_V(n) asm volatile("s_waitcnt vmcnt(" #n ")" ::: "memory")
#define WAIT_L(n) asm volatile("s_waitcnt lgkmcnt(" #n ")" ::: "memory")
#define BAR __builtin_amdgcn_s_barrier()
#define SCHED __builtin_amdgcn_sched_barrier(0)
#define STAMP(i)

__device__ __forceinline__ bool wait_words(const unsigned* p, unsigned target, bool active, unsigned* tmo, unsigned call, int lane) {
    const u64 t0 = __builtin_amdgcn_s_memrealtime();
    for (;;) {
        const unsigned v = __hip_atomic_load(p, __ATOMIC_RELAXED, __HIP_MEMORY_SCOPE_SYSTEM);
        if (__builtin_amdgcn_ballot_w64(active && (int)(v - target) < 0) == 0ull) return false;
        if (rfl(__hip_atomic_load(tmo, __ATOMIC_RELAXED, __HIP_MEMORY_SCOPE_AGENT)) == call || __builtin_amdgcn_s_memrealtime() - t0 > DEADLINE_TICKS) {
            if (lane == 0) __hip_atomic_store(tmo, call, __ATOMIC_RELAXED, __HIP_MEMORY_SCOPE_AGENT);
            return true;
        }
        __builtin_amdgcn_s_sleep(4);
    }
}

struct Mover {
    const Params& P; LAS unsigned char* lds; int wid, lane;
    unsigned w_src, x_src, tr_off, sc_off, w_dst;
    __device__ __forceinline__ Mover(const Params& P_, LAS unsigned char* lds_, int wid_, int lane_) : P(P_), lds(lds_), wid(wid_), lane(lane_) {
        w_src = (unsigned)(((wid >> 2) * 32 + (lane >> 2)) * N + (wid & 3) * 64 + (lane & 3) * 16);
        x_src = (unsigned)((wid * 16 + (lane & 15)) * KL + (lane >> 4) * 16);
        tr_off = (unsigned)(((lane & 15) >> 1) * 64 + (lane >> 4) * 16 + (lane & 1) * 8);
        sc_off = (unsigned)(2048 + (lane & 3) * 512 + (lane >> 2) * 16);
        w_dst = (unsigned)(((wid & 3) * 2 + (lane >> 5)) * 2048 + (wid >> 2) * 512 + (lane & 31) * 16);
    }
    __device__ __forceinline__ unsigned entry(int u) const { return u < NUIN ? P.order[u >> 1] : 0xff4000u; }
    __device__ __forceinline__ unsigned fwd_off(int u) const {
        const unsigned f = (unsigned)(u - NUIN), t = (unsigned)P.rank * NOWN + (f >> 4);
        return ((t & 15u) * 256u + (f & 15u) * 16u + (unsigned)wid * 2u) * (unsigned)(N * 4) + (DPAN + (t >> 4)) * 1024u;
    }
    __device__ __forceinline__ void stage(int u, unsigned b) const {
        const unsigned e = entry(u), hh = (unsigned)u & 1u, j = (e >> 8) & 3u, pan = e & 127u;
        const unsigned char* s; unsigned step;
        if (e & 0x8000u) { s = P.w + (size_t)(j * 128u + hh * 64u) * N + pan * 256u + w_src; step = 16u * N; }
        else if (e & 0x4000u) { s = P.out[P.rank] + fwd_off(u) + (unsigned)lane * 16u; step = N * 4u; }
        else { s = P.x + (size_t)(pan * 256u) * KL + j * 128u + hh * 64u + x_src; step = 128u * KL; }
        __builtin_amdgcn_global_load_lds((const GAS void*)s, (LAS void*)(lds + b), 16, 0, 17);
        __builtin_amdgcn_global_load_lds((const GAS void*)(s + step), (LAS void*)(lds + b + 1024), 16, 0, 17);
    }
    __device__ __forceinline__ u32x4 fetch(int u, unsigned b, int k, unsigned& off) const {
        const unsigned e = entry(u), hh = (unsigned)u & 1u, j = (e >> 8) & 3u, pan = e & 127u, slot = j * 8u + (unsigned)P.rank;
        const unsigned lb = (unsigned)(size_t)lds + b;
        unsigned a = lb + (unsigned)k * 1024u + (unsigned)lane * 16u;
        if (e & 0x8000u) {
            if (k == 0) {
                v2i g0, g1, g2, g3;
                const unsigned ra = lb + tr_off, wa = lb + sc_off;
                asm volatile("ds_read_b64_tr_b8 %0, %4\n\tds_read_b64_tr_b8 %1, %4 offset:512\n\t"
                             "ds_read_b64_tr_b8 %2, %4 offset:1024\n\tds_read_b64_tr_b8 %3, %4 offset:1536\n\ts_waitcnt lgkmcnt(0)\n\t"
                             "ds_write_b64 %5, %0\n\tds_write_b64 %5, %1 offset:8\n\tds_write_b64 %5, %2 offset:256\n\tds_write_b64 %5, %3 offset:264\n\t"
                             "s_waitcnt lgkmcnt(0)"
                             : "=&v"(g0), "=&v"(g1), "=&v"(g2), "=&v"(g3) : "v"(ra), "v"(wa) : "memory");
            }
            a += 2048u;
            off = X_BYTES + slot * WS_SLAB + (pan * 16u + (unsigned)k * 8u) * 2048u + hh * 1024u + w_dst;
        } else if (e & 0x4000u) {
            off = fwd_off(u) + (unsigned)k * (unsigned)(N * 4) + (unsigned)lane * 16u;
        } else {
            off = slot * XS_SLAB + (pan * 16u + (unsigned)wid + (unsigned)k * 8u) * 2048u + hh * 1024u + (unsigned)lane * 16u;
        }
        u32x4 v;
        asm volatile("ds_read_b128 %0, %1\n\ts_waitcnt lgkmcnt(0)" : "=&v"(v) : "v"(a) : "memory");
        return v;
    }
    __device__ __forceinline__ unsigned flag_off(int u) const {
        const unsigned e = P.order[u >> 1], j = (e >> 8) & 3u, pan = e & 127u, slot = j * 8u + (unsigned)P.rank;
        return ((e & 0x8000u) ? FW_OFF : FX_OFF) + pan * 128u + slot * 4u;
    }
};
#define STORE_WT(off, v, base) asm volatile("s_nop 4\n\tglobal_store_dwordx4 %0, %1, %2 sc0 sc1\n\ts_nop 1" :: "v"(off), "v"(v), "s"(base) : "memory")

__device__ __forceinline__ void pusher(const Params& P, LAS unsigned char* lds, int wid, int lane, int pi, int role) {
    const Mover mv(P, lds, wid, lane);
    const unsigned wbuf = (unsigned)wid * 8192u;
    mv.stage(pi, wbuf);
    __builtin_amdgcn_s_setprio(3);
    unsigned char* const wsl = P.ws[P.rank];
    unsigned call;
    {
        unsigned t = 0;
        if (lane == 0) t = __hip_atomic_fetch_add((unsigned*)(wsl + CTR_OFF) + pi * 16 + wid, 1u, __ATOMIC_RELAXED, __HIP_MEMORY_SCOPE_AGENT);
        call = rfl(t) + 1u;
    }
    unsigned* const tmo = (unsigned*)(wsl + TMO_OFF);
    unsigned char* dst[NR - 1];
    unsigned pbit[NR - 1];
    auto bases = [&](bool fwd) {
#pragma unroll
        for (int q = 0; q < NR - 1; ++q) {
            const unsigned p = (unsigned)(P.rank + 1 + (q + wid + pi) % (NR - 1)) & 7u;
            dst[q] = rfl_ptr(fwd ? P.out[p] : P.ws[p] + DATA_OFF + (size_t)(call & 1u) * PAR_BYTES);
            pbit[q] = 0x10000u << p;
        }
    };
    bases(false);
    const unsigned peer = (unsigned)(P.rank + 1 + lane) & 7u;
    unsigned char* flag_base = P.ws[peer];
    asm volatile("" : "+v"(flag_base));
    unsigned* const tick = (unsigned*)(wsl + TICK_OFF + (call & 1u) * 128u);
    const unsigned tbox = (unsigned)(size_t)lds + 65536u;
    auto signal = [&](int u, int i) {
        if (u < NUIN && wid == (i & 7)) {
            if (lane < NR - 1 && ((P.order[u >> 1] >> 16) >> peer & 1u))
                __hip_atomic_fetch_add((unsigned*)(flag_base + mv.flag_off(u)), 1u, __ATOMIC_RELAXED, __HIP_MEMORY_SCOPE_SYSTEM);
        }
    };
    bool fwd = false, bad = false;
    int uprev = -1, u = pi, unext = pi + NPUSH;
    WAIT_V(0);
    for (int i = 0; u < NUNIT; ++i) {
        if (unext < NUNIT) {
            if (unext >= NUIN && !fwd) {
                fwd = true;
                bad |= wait_words((const unsigned*)(wsl + TDONE_OFF) + (lane < NOWN ? lane : 0), 8u * call, lane < NOWN, tmo, call, lane);
            }
            mv.stage(unext, wbuf + (unsigned)((i + 1) & 1) * 4096u);
        }
        if (wid == 0 && lane == 0) { const unsigned one = 1u; asm volatile("global_atomic_add v255, %0, %1, off sc0" :: "v"(tick), "v"(one) : "memory", "v255"); }
        if (u >= NUIN && uprev < NUIN) bases(true);
        const unsigned e = mv.entry(u);
        const bool full = ((e | (0x10000u << P.rank)) >> 16) == 0xffu;
        int un2 = 0;
        auto meet = [&]() {
            if (wid == 0 && lane == 0) asm volatile("ds_write_b32 %0, v255\n\ts_waitcnt lgkmcnt(0)" :: "v"(tbox + (unsigned)(i & 1) * 4u) : "memory");
            BAR;
            unsigned t;
            asm volatile("ds_read_b32 %0, %1\n\ts_waitcnt lgkmcnt(0)" : "=&v"(t) : "v"(tbox + (unsigned)(i & 1) * 4u) : "memory");
            un2 = (int)rfl(t) + 2 * NPUSH;
            if (uprev >= 0) signal(uprev, i);
        };
        for (int k = 0; k < 2; ++k) {
            unsigned off;
            const u32x4 v = mv.fetch(u, wbuf + (unsigned)(i & 1) * 4096u, k, off);
#pragma unroll
            for (int q = 0; q < NR - 1; ++q) {
                if (e & pbit[q]) {
                    STORE_WT(off, v, dst[q]);
                    asm volatile("s_waitcnt vmcnt(%0)" :: "n"(PUSH_WIN) : "memory");
                }
                if (q == (PUSH_WIN - 1) % (NR - 1) && k == (PUSH_WIN - 1) / (NR - 1) && full) meet();
            }
        }
        if (!full) {
            const int c = __builtin_popcount((e & ~(0x10000u << P.rank)) >> 16);
            if (c >= 3) WAIT_V(6); else if (c == 2) WAIT_V(4); else if (c == 1) WAIT_V(2); else WAIT_V(0);
            meet();
        }
        uprev = u; u = unext; unext = un2;
    }
    WAIT_V(0);
    BAR;
    signal(uprev, 0);
    if (wid == 0) {
        if (lane < NR - 1) __hip_atomic_fetch_add((unsigned*)(flag_base + DONE_OFF + (unsigned)P.rank * 128u), 1u, __ATOMIC_RELAXED, __HIP_MEMORY_SCOPE_SYSTEM);
        STAMP(1);
        if (pi == 0) {
            bad |= wait_words((const unsigned*)(wsl + DONE_OFF + peer * 128u), (unsigned)NPUSH * call, lane < NR - 1, tmo, call, lane);
            if (lane < 8) __hip_atomic_store((unsigned*)(wsl + ROLE_OFF + ((call & 1u) ^ 1u) * 1024u + (unsigned)lane * 128u), 0u, __ATOMIC_RELAXED, __HIP_MEMORY_SCOPE_AGENT);
            if (lane == 8) __hip_atomic_store((unsigned*)(wsl + TICK_OFF + ((call & 1u) ^ 1u) * 128u), 0u, __ATOMIC_RELAXED, __HIP_MEMORY_SCOPE_AGENT);
            STAMP(2);
        }
    }
    if (bad && lane == 0) __hip_atomic_store((unsigned*)P.out[P.rank], 0x7fc00000u, __ATOMIC_RELAXED, __HIP_MEMORY_SCOPE_SYSTEM);
}
static_assert(PUSH_WIN >= 1 && PUSH_WIN <= 2 * (NR - 1), "window");

__device__ __forceinline__ void local_copy(const Params& P, LAS unsigned char* lds, unsigned call, int wid, int lane, int li) {
    const Mover mv(P, lds, wid, lane);
    const int nun = li + 2 * NGEMM < NUIN ? 3 : 2;
    const unsigned wbuf = (unsigned)wid * 12288u;
    for (int i = 0; i < nun; ++i) mv.stage(li + NGEMM * i, wbuf + (unsigned)i * 4096u);
    unsigned char* const wsl = P.ws[P.rank];
    unsigned char* const dst = rfl_ptr(wsl + DATA_OFF + (size_t)(call & 1u) * PAR_BYTES);
    WAIT_V(0);
    for (int i = 0; i < nun; ++i)
        for (int k = 0; k < 2; ++k) {
            unsigned off;
            const u32x4 v = mv.fetch(li + NGEMM * i, wbuf + (unsigned)i * 4096u, k, off);
            STORE_WT(off, v, dst);
        }
    WAIT_V(0);
    __syncthreads();
    if (wid < nun && lane == 0) __hip_atomic_fetch_add((unsigned*)(wsl + mv.flag_off(li + NGEMM * wid)), 1u, __ATOMIC_RELAXED, __HIP_MEMORY_SCOPE_SYSTEM);
}

__global__ __launch_bounds__(NTHR, 2) void fused_kernel(const Params P) {
    extern __shared__ __attribute__((aligned(16))) unsigned char shm[];
    LAS unsigned char* lds = (LAS unsigned char*)shm;
    const int tid = threadIdx.x, wid = __builtin_amdgcn_readfirstlane(tid >> 6), lane = tid & 63;
    unsigned char* const wsl = P.ws[P.rank];
    if (blockIdx.x < NPUSH) {
        pusher(P, lds, wid, lane, blockIdx.x, NGEMM + blockIdx.x);
        return;
    }
    if (wid == 0) {
        const unsigned xcc = (unsigned)__builtin_amdgcn_s_getreg((3 << 11) | 20) & 7u;
        unsigned t = 0;
        if (lane < 3) t = __hip_atomic_fetch_add((unsigned*)(wsl + (lane == 0 ? CTR_OFF + blockIdx.x * 64u : ROLE_OFF + (unsigned)(lane - 1) * 1024u + xcc * 128u)),
                                                 1u, __ATOMIC_RELAXED, __HIP_MEMORY_SCOPE_AGENT);
        const unsigned c = (unsigned)__builtin_amdgcn_readlane((int)t, 0) + 1u;
        unsigned tk = (c & 1u) ? (unsigned)__builtin_amdgcn_readlane((int)t, 2) : (unsigned)__builtin_amdgcn_readlane((int)t, 1), x = xcc;
        for (int k = 1; k < 8 && tk >= 30u; ++k) {
            x = (xcc + (unsigned)k) & 7u;
            unsigned t2 = 0;
            if (lane == 0) t2 = __hip_atomic_fetch_add((unsigned*)(wsl + ROLE_OFF + (c & 1u) * 1024u + x * 128u), 1u, __ATOMIC_RELAXED, __HIP_MEMORY_SCOPE_AGENT);
            tk = rfl(t2);
        }
        if (lane == 0) *(LAS v2i*)(lds + LDS_MBOX) = (v2i){(int)c, (int)(tk < 30u ? x * 30u + tk : 256u)};
    }
    __syncthreads();
    const v2i cr = *(const LAS v2i*)(lds + LDS_MBOX);
    const unsigned call = rfl((unsigned)cr.x);
    const int role = (int)rfl((unsigned)cr.y);
    if (role >= NGEMM) return;
    const int band = role / 60, i60 = role - band * 60, g = i60 >> 2, prow = band * 4 + (i60 & 3);
#define LC_DELAY 0
    if (LC_DELAY) { const u64 t0 = __builtin_amdgcn_s_memrealtime(); while (__builtin_amdgcn_s_memrealtime() - t0 < (u64)((role & 7) * LC_DELAY)) __builtin_amdgcn_s_sleep(8); }
    local_copy(P, lds, call, wid, lane, role);
    STAMP(11);

    u64 tiles = 0; int nseg = 0;
    for (int r = 0; r < 4; ++r) {
        const int sidx = r * 15 + g;
        unsigned t;
        if (sidx == SPSLOT) {
            const int lo = prow & 7, a = (prow >> 3) * 5 + (lo < 3 ? lo : lo - 1);
            if (lo == 3 || lo > 5) continue;
            const unsigned o = (unsigned)(P.rank * NOWN + a);
            t = (o & 15u) | (DPAN + (o >> 4)) << 4 | (unsigned)(a + 1) << 10;
        } else t = (unsigned)prow | (unsigned)(sidx - (sidx > SPSLOT)) << 4;
        tiles |= (u64)t << (16 * nseg); ++nseg;
    }
    auto tile_lane = [&](int sg) { return (unsigned)(tiles >> (16 * (sg & 3))) & 0xffffu; };
    auto tile_of = [&](int sg) { return rfl(tile_lane(sg)); };
    const int wr = wid >> 2, wc = wid & 3;
    const unsigned thr = 2u * call;
    unsigned char* const dbase = wsl + DATA_OFF + (size_t)(call & 1u) * PAR_BYTES;
    const __amdgpu_buffer_rsrc_t rsX = __builtin_amdgcn_make_buffer_rsrc((void*)dbase, 0, X_BYTES, 0x00020000);
    const __amdgpu_buffer_rsrc_t rsW = __builtin_amdgcn_make_buffer_rsrc((void*)(dbase + X_BYTES), 0, W_BYTES, 0x00020000);
    const __amdgpu_buffer_rsrc_t rsF = __builtin_amdgcn_make_buffer_rsrc((void*)wsl, 0, DATA_OFF, 0x00020000);
    const unsigned voff = (unsigned)(wid * 2048 + lane * 16);
    const unsigned ldsw = (unsigned)wid * 2048u;
    int aoff = wr * 8192 + lane * 16, boff = LDS_B + wc * 4096 + lane * 16;
    asm volatile("" : "+v"(aoff), "+v"(boff));
    const unsigned snap_src = (lane < 32 ? FW_OFF + ((tile_lane(lane >> 3) >> 4) & 63u) * 128u : FX_OFF + (tile_lane(lane >> 3) & 15u) * 128u) + (unsigned)(lane & 7) * 16u;
    const unsigned snap_rd = lane < 32 ? (unsigned)lane * 4u : 384u + (unsigned)lane * 4u;

#define DMA(rs, ldsoff, vo, so, imm) __builtin_amdgcn_raw_ptr_buffer_load_lds(rs, (LAS void*)(lds + (ldsoff)), 16, vo, so, imm, 17)
#define STAGE_A(b, h, so) do { DMA(rsX, ((b) * 2 + (h)) * HT + ldsw, voff, (so) + (h) * HT, 0); DMA(rsX, ((b) * 2 + (h)) * HT + ldsw, voff, (so) + (h) * HT, 1024); } while (0)
#define STAGE_B(b, h, so) do { DMA(rsW, LDS_B + ((b) * 2 + (h)) * HT + ldsw, voff, (so) + (h) * HT, 0); DMA(rsW, LDS_B + ((b) * 2 + (h)) * HT + ldsw, voff, (so) + (h) * HT, 1024); } while (0)
#define LDF(off) __builtin_shufflevector(*(const LAS v4i*)(lds + (off)), *(const LAS v4i*)(lds + (off) + 1024), 0, 1, 2, 3, 4, 5, 6, 7)
#define LDA(b, h) do { _Pragma("unroll") for (int m = 0; m < 4; ++m) At[m] = LDF(((b) * 2 + (h)) * HT + aoff + m * 2048); } while (0)
#define LDB(dst, b, h) do { _Pragma("unroll") for (int n = 0; n < 2; ++n) dst[n] = LDF(((b) * 2 + (h)) * HT + boff + n * 2048); } while (0)
#define MF(c, bb, aa) asm volatile("v_mfma_f32_16x16x128_f8f6f4 %0, %1, %2, %0 cbsz:1 blgp:1" : "+v"(c) : "v"(aa), "v"(bb))
#define MMA(ai, bj, Bt) do { __builtin_amdgcn_s_setprio(1); _Pragma("unroll") for (int m = 0; m < 4; ++m) _Pragma("unroll") for (int n = 0; n < 2; ++n) \
        MF(acc[ai * 4 + m][bj * 2 + n], Bt[n], At[m]); __builtin_amdgcn_s_setprio(0); } while (0)

    f32x4 acc[8][4];
    v8i At[4], B0[2], B1[2];
#pragma unroll
    for (int m = 0; m < 8; ++m)
#pragma unroll
        for (int n = 0; n < 4; ++n) acc[m][n] = (f32x4){0.f, 0.f, 0.f, 0.f};

    unsigned xm = 0u, wm = 0u, used = 0u, bad = 0u;
    int ts = 0;
    unsigned a_pan = (tile_of(0) & 15u) * 32768u, b_pan = ((tile_of(0) >> 4) & 63u) * 32768u;
    auto slow_wait = [&]() {
        if (wid == 0) {
            const unsigned* fp = (const unsigned*)(wsl + (lane < 32 ? FW_OFF + (b_pan >> 8) + (unsigned)lane * 4u : FX_OFF + (a_pan >> 8) + (unsigned)(lane - 32) * 4u));
            unsigned* tmo = (unsigned*)(wsl + TMO_OFF);
            const u64 t0 = __builtin_amdgcn_s_memrealtime();
            u64 mk; unsigned dead = 0u;
            for (;;) {
                const unsigned v = __hip_atomic_load(fp, __ATOMIC_RELAXED, __HIP_MEMORY_SCOPE_SYSTEM);
                mk = __builtin_amdgcn_ballot_w64((int)(v - thr) >= 0);
                if (__builtin_popcount((unsigned)mk & (unsigned)(mk >> 32) & ~used) >= 2) break;
                if (rfl(__hip_atomic_load(tmo, __ATOMIC_RELAXED, __HIP_MEMORY_SCOPE_AGENT)) == call || __builtin_amdgcn_s_memrealtime() - t0 > DEADLINE_TICKS) {
                    if (lane == 0) __hip_atomic_store(tmo, call, __ATOMIC_RELAXED, __HIP_MEMORY_SCOPE_AGENT);
                    mk = ~0ull; dead = 1u; break;
                }
                __builtin_amdgcn_s_sleep(4);
            }
            if (lane == 0) *(LAS u32x4*)(lds + LDS_MBOX + 16) = (u32x4){(unsigned)mk, (unsigned)(mk >> 32), dead, 0u};
            asm volatile("s_waitcnt vmcnt(0) lgkmcnt(0)" ::: "memory");
        }
        asm volatile("" ::: "memory"); BAR; asm volatile("" ::: "memory");
        const u32x4 mb = *(const LAS u32x4*)(lds + LDS_MBOX + 16);
        wm |= rfl(mb.x); xm |= rfl(mb.y); bad |= rfl(mb.z);
    };
    unsigned a2, b2, a3, b3;
    auto take = [&](unsigned av) {
        const unsigned sA = (unsigned)__builtin_ctz(av), sB = (unsigned)__builtin_ctz(av & (av - 1u));
        used |= (1u << sA) | (1u << sB);
        a2 = sA * XS_SLAB + a_pan; b2 = sA * WS_SLAB + b_pan; a3 = sB * XS_SLAB + a_pan; b3 = sB * WS_SLAB + b_pan;
    };

    const float sl = __builtin_bit_cast(float, rfl(__builtin_bit_cast(unsigned, (P.sx[0] * P.sw[0]) * -1.44269504089f)));
    slow_wait(); take(xm & wm);
    STAMP(1);
    unsigned a1 = a3;
    STAGE_A(0, 1, a2); STAGE_B(0, 0, b2); STAGE_A(0, 0, a2); STAGE_B(0, 1, b2);
    if (wr == 1) BAR;
    WAIT_V(2); BAR;
    STAGE_B(1, 0, b3); STAGE_A(1, 0, a3); STAGE_B(1, 1, b3);
    WAIT_V(6); BAR;
    LDB(B0, 0, 0);
    bool ready;
    { const unsigned av = xm & wm & ~used; ready = (av & (av - 1u)) != 0u; if (ready) take(av); }
    for (int seg = 0; seg < nseg; ++seg) {
        for (int it = 0; it < 16; ++it) {
            if (!ready) { slow_wait(); take(xm & wm & ~used); }
            const unsigned snap = LDS_SNAP + (unsigned)(it & 1) * 1024u;
            LDA(0, 0); STAGE_A(1, 1, a1);
            if (wid == 0) DMA(rsF, snap, snap_src, 0, 0);
            BAR; WAIT_L(0); SCHED; MMA(0, 0, B0); BAR; SCHED;
            LDB(B1, 0, 1); STAGE_B(0, 0, b2);
            BAR; WAIT_L(0); SCHED; MMA(0, 1, B1); BAR; SCHED;
            LDA(0, 1); STAGE_A(0, 0, a2);
            WAIT_V(10); BAR; WAIT_L(0); SCHED; MMA(1, 0, B0); BAR; SCHED;
            LDB(B0, 1, 0); STAGE_B(0, 1, b2);
            WAIT_V(6); BAR; SCHED; MMA(1, 1, B1); BAR; SCHED;
            LDA(1, 0); STAGE_A(0, 1, a2);
            BAR; WAIT_L(0); SCHED; MMA(0, 0, B0); BAR; SCHED;
            LDB(B1, 1, 1); STAGE_B(1, 0, b3);
            const unsigned snapv = *(const LAS unsigned*)(lds + snap + (unsigned)((it == 14 ? ts + 1 : ts) & 3) * 128u + snap_rd);
            BAR; WAIT_L(0); SCHED; MMA(0, 1, B1); BAR; SCHED;
            LDA(1, 1); STAGE_A(1, 0, a3);
            WAIT_V(10); BAR; WAIT_L(0); SCHED; MMA(1, 0, B0); BAR; SCHED;
            const u64 mk = __builtin_amdgcn_ballot_w64((int)(snapv - thr) >= 0);
            SCHED;
            LDB(B0, 0, 0); STAGE_B(1, 1, b3);
            a1 = a3;
            if (it == 14) {
                ++ts; wm = 0u; xm = 0u; used = 0u;
                const unsigned t = tile_of(ts);
                a_pan = (t & 15u) * 32768u; b_pan = ((t >> 4) & 63u) * 32768u;
            }
            wm |= (unsigned)mk; xm |= (unsigned)(mk >> 32);
            if (ts >= nseg) { wm = xm = ~0u; used = 0u; }
            { const unsigned av = xm & wm & ~used; ready = (av & (av - 1u)) != 0u; if (ready) take(av); }
            WAIT_V(6); BAR; SCHED; MMA(1, 1, B1); BAR; SCHED;
        }
        STAMP(2 * seg + 2);
        SCHED; asm volatile("s_nop 15\n\ts_nop 15" ::: "memory"); SCHED;
        const unsigned tcur = tile_of(seg);
        {
            int tz = threadIdx.x; asm volatile("" : "+v"(tz));
            const unsigned ov = (unsigned)((tz >> 4) & 3) * (unsigned)(4 * N * 4) + (unsigned)(tz & 15) * 16u;
            const char* ob = (const char*)rfl_ptr(P.out[P.rank] + (size_t)((tcur & 15u) * 256u + (unsigned)wr * 64u) * (N * 4) + (((tcur >> 4) & 63u) * 256u + (unsigned)wc * 64u) * 4u);
            asm volatile("s_nop 4" : "+s"(ob));
            const float cc = bad ? __builtin_nanf("") : -1.44269504089f;
#pragma unroll
            for (int m = 0; m < 8; ++m) {
                const char* rowp = ob + (size_t)((m >> 2) * 128 + (m & 3) * 16) * (N * 4);
#pragma unroll
                for (int j = 0; j < 4; ++j) {
                    const f32x4 o = silu4(acc[m][0][j], acc[m][1][j], acc[m][2][j], acc[m][3][j], sl, cc);
                    asm volatile("global_store_dwordx4 %0, %1, %2" OUT_POL "\n\ts_nop 1" :: "v"(ov), "v"(o), "s"(rowp + (size_t)j * (N * 4)) : "memory");
                }
#pragma unroll
                for (int n = 0; n < 4; ++n) acc[m][n] = (f32x4){0.f, 0.f, 0.f, 0.f};
                if (m == 0 && wr == 0) BAR;
            }
        }
        if (tcur >> 10) {
            WAIT_V(0);
            if (lane == 0) __hip_atomic_fetch_add((unsigned*)(wsl + TDONE_OFF) + ((tcur >> 10) - 1u), 1u, __ATOMIC_RELAXED, __HIP_MEMORY_SCOPE_AGENT);
        }
        STAMP(2 * seg + 3);
        if (wr == 1) BAR;
    }
    WAIT_V(0);
    if (wr == 0) BAR;
    STAMP(10);
}

constexpr double g_tune[8] = {2.0, 2.0, 0.0, 8.0, 45.0, 0.0, 3.0, 2.0};
void build_order(int s, unsigned* out) {
    struct It { double key; unsigned code; };
    It it[NCHUNK]; int n = 0;
    const double delta = g_tune[0], tau0 = g_tune[1], lead = g_tune[2], xs = g_tune[3], Tr = g_tune[4], T1 = 32 * tau0 + g_tune[5];
    const int grot = (int)g_tune[6], xrot = (int)g_tune[7];
    for (int j = 0; j < 4; ++j)
        for (int p = 0; p < 16; ++p)
            it[n++] = It{(8 * j - lead) * tau0 + xs * tau0 * (((p + xrot * s) & 15) / 16.0) - 0.01, 0xff0000u | (unsigned)((j << 8) | p)};
    for (int c = 0; c < 64; ++c) {
        const int sidx = c >= DPAN ? SPSLOT : c + (c >= SPSLOT), r = sidx / 15, g = sidx % 15;
        unsigned mask = 0xffu;
        if (c >= DPAN) { mask = 0; for (int t = (c - DPAN) * 16; t < (c - DPAN) * 16 + 16; ++t) mask |= 1u << (t / NOWN); }
        for (int j = 0; j < 4; ++j) {
            const int pos = 8 * j + (grot ? ((s + g * grot + (g >> 3) * 4) & 7) : 0);
            const double key = (r == 0 ? (g & 7) * delta + pos * tau0 : T1 + (r - 1) * Tr + (g & 7) * delta + pos * (Tr / 34.0)) + 1e-4 * c;
            it[n++] = It{key, mask << 16 | 0x8000u | (unsigned)((j << 8) | c)};
        }
    }
    std::stable_sort(it, it + NCHUNK, [](const It& a, const It& b) { return a.key < b.key; });
    for (int i = 0; i < NCHUNK; ++i) out[i] = it[i].code;
}
}

extern "C" void kernel_launch(int rank, int nranks, ncclComm_t comm,
                              hipStream_t stream,
                              void* const* d_in, const int* in_sizes, int n_in,
                              void* d_out, int out_size,
                              const int* in_dtypes, int out_dtype,
                              void* const* d_out_all,
                              void* const* d_ws_all, size_t ws_bytes) {
    (void)comm; (void)in_dtypes; (void)out_dtype;
    if (nranks != NR || (unsigned)rank >= (unsigned)NR || n_in != 4 || ws_bytes < WS_NEED || out_size != M * N ||
        in_sizes[0] != M * KL || in_sizes[1] != KL * N) {
        fprintf(stderr, "kernel_launch: unexpected problem shape\n");
        return;
    }
    Params P;
    P.x = (const unsigned char*)d_in[0]; P.w = (const unsigned char*)d_in[1];
    P.sx = (const float*)d_in[2]; P.sw = (const float*)d_in[3];
    for (int p = 0; p < NR; ++p) { P.ws[p] = (unsigned char*)d_ws_all[p]; P.out[p] = (unsigned char*)d_out_all[p]; }
    P.out[rank] = (unsigned char*)d_out;
    P.rank = rank; P.pad = 0;
    build_order(rank, P.order);
    HIP_CHECK(hipFuncSetAttribute((const void*)fused_kernel, hipFuncAttributeMaxDynamicSharedMemorySize, LDS_BYTES));
    hipLaunchKernelGGL(fused_kernel, dim3(NWG), dim3(NTHR), LDS_BYTES, stream, P);
    HIP_CHECK(hipGetLastError());
}
```

```cpp
#include <hip/hip_runtime.h>
#include <rccl/rccl.h>
#include <cstdio>
#include <algorithm>

#define HIP_CHECK(expr) do { hipError_t _e = (expr); if (_e != hipSuccess) { \
    fprintf(stderr, "HIP error %s at %s:%d\n", hipGetErrorString(_e), __FILE__, __LINE__); \
    return; } } while (0)

namespace {
#define LAS __attribute__((address_space(3)))
#define GAS __attribute__((address_space(1)))
typedef int v8i __attribute__((ext_vector_type(8)));
typedef int v4i __attribute__((ext_vector_type(4)));
typedef int v2i __attribute__((ext_vector_type(2)));
typedef unsigned u32x4 __attribute__((ext_vector_type(4)));
typedef float f32x4 __attribute__((ext_vector_type(4)));
typedef float f32x2 __attribute__((ext_vector_type(2)));
typedef unsigned long long u64;

constexpr int M = 4096, N = 16384, KL = 512, NR = 8;
constexpr int NPUSH = 16, NWG = 256, NGEMM = NWG - NPUSH, NTHR = 512;
constexpr int NCHUNK = 4 * 16 + 4 * 64;
constexpr int NUIN = 2 * NCHUNK;
constexpr int DPAN = 59, NOWN = 10;
constexpr int NUNIT = NUIN + NOWN * 16;
constexpr int SPSLOT = 15 + 14;
constexpr unsigned XS_SLAB = 256u * 2048u;
constexpr unsigned WS_SLAB = 1024u * 2048u;
constexpr unsigned X_BYTES = 32u * XS_SLAB, W_BYTES = 32u * WS_SLAB, PAR_BYTES = X_BYTES + W_BYTES;
constexpr unsigned CTR_OFF = 0;
constexpr unsigned TMO_OFF = 32768;
constexpr unsigned ROLE_OFF = 36864;
constexpr unsigned TICK_OFF = 40960;
constexpr unsigned TDONE_OFF = 45056;
constexpr unsigned DONE_OFF = 49152;
constexpr unsigned FX_OFF = 65536;
constexpr unsigned FW_OFF = 73728;
constexpr unsigned DBG_OFF = 131072;
constexpr unsigned DATA_OFF = 1u << 20;
constexpr size_t WS_NEED = (size_t)DATA_OFF + 2ull * PAR_BYTES;
constexpr u64 DEADLINE_TICKS = 100000000ull;
constexpr int HT = 16384, LDS_B = 4 * HT, LDS_SNAP = 8 * HT, LDS_MBOX = LDS_SNAP + 2048, LDS_BYTES = LDS_MBOX + 64;
#define OUT_POL " sc0 sc1"
#define PUSH_WIN 7

struct Params {
    const unsigned char* x; const unsigned char* w; const float* sx; const float* sw;
    unsigned char* ws[NR]; unsigned char* out[NR];
    int rank; int pad;
    unsigned order[NCHUNK];
};

__device__ __forceinline__ unsigned rfl(unsigned v) { return (unsigned)__builtin_amdgcn_readfirstlane((int)v); }
__device__ __forceinline__ unsigned char* rfl_ptr(unsigned char* p) {
    return (unsigned char*)(((u64)rfl((unsigned)((u64)p >> 32)) << 32) | (u64)rfl((unsigned)(u64)p));
}
__device__ __forceinline__ float mul1(float a, float b) { float r; asm("v_mul_f32 %0, %1, %2" : "=v"(r) : "s"(b), "v"(a)); return r; }
__device__ __forceinline__ f32x4 silu4(float a0, float a1, float a2, float a3, float sl, float c) {
    const f32x2 t0 = {mul1(a0, sl), mul1(a1, sl)}, t1 = {mul1(a2, sl), mul1(a3, sl)};
    const float L = 86.56170245f;
    f32x2 e0, e1;
    e0.x = __builtin_amdgcn_exp2f(__builtin_amdgcn_fmed3f(t0.x, -L, L)); e0.y = __builtin_amdgcn_exp2f(__builtin_amdgcn_fmed3f(t0.y, -L, L));
    e1.x = __builtin_amdgcn_exp2f(__builtin_amdgcn_fmed3f(t1.x, -L, L)); e1.y = __builtin_amdgcn_exp2f(__builtin_amdgcn_fmed3f(t1.y, -L, L));
    const f32x2 cv = {c, c};
    const f32x2 d0 = __builtin_elementwise_fma(e0, cv, cv), d1 = __builtin_elementwise_fma(e1, cv, cv);
    const f32x2 r0 = {__builtin_amdgcn_rcpf(d0.x), __builtin_amdgcn_rcpf(d0.y)}, r1 = {__builtin_amdgcn_rcpf(d1.x), __builtin_amdgcn_rcpf(d1.y)};
    const f32x2 o0 = t0 * r0, o1 = t1 * r1;
    return (f32x4){o0.x, o0.y, o1.x, o1.y};
}

#define WAIT_V(n) asm volatile("s_waitcnt vmcnt(" #n ")" ::: "memory")
#define WAIT_L(n) asm volatile("s_waitcnt lgkmcnt(" #n ")" ::: "memory")
#define BAR __builtin_amdgcn_s_barrier()
#define SCHED __builtin_amdgcn_sched_barrier(0)
#define STAMP(i)

__device__ __forceinline__ bool wait_words(const unsigned* p, unsigned target, bool active, unsigned* tmo, unsigned call, int lane) {
    const u64 t0 = __builtin_amdgcn_s_memrealtime();
    for (;;) {
        const unsigned v = __hip_atomic_load(p, __ATOMIC_RELAXED, __HIP_MEMORY_SCOPE_SYSTEM);
        if (__builtin_amdgcn_ballot_w64(active && (int)(v - target) < 0) == 0ull) return false;
        if (rfl(__hip_atomic_load(tmo, __ATOMIC_RELAXED, __HIP_MEMORY_SCOPE_AGENT)) == call || __builtin_amdgcn_s_memrealtime() - t0 > DEADLINE_TICKS) {
            if (lane == 0) __hip_atomic_store(tmo, call, __ATOMIC_RELAXED, __HIP_MEMORY_SCOPE_AGENT);
            return true;
        }
        __builtin_amdgcn_s_sleep(4);
    }
}

struct Mover {
    const Params& P; LAS unsigned char* lds; int wid, lane;
    unsigned w_src, x_src, tr_off, sc_off, w_dst;
    __device__ __forceinline__ Mover(const Params& P_, LAS unsigned char* lds_, int wid_, int lane_) : P(P_), lds(lds_), wid(wid_), lane(lane_) {
        w_src = (unsigned)(((wid >> 2) * 32 + (lane >> 2)) * N + (wid & 3) * 64 + (lane & 3) * 16);
        x_src = (unsigned)((wid * 16 + (lane & 15)) * KL + (lane >> 4) * 16);
        tr_off = (unsigned)(((lane & 15) >> 1) * 64 + (lane >> 4) * 16 + (lane & 1) * 8);
        sc_off = (unsigned)((lane & 3) * 512 + (lane >> 2) * 16);
        w_dst = (unsigned)(((wid & 3) * 2 + (lane >> 5)) * 2048 + (wid >> 2) * 512 + (lane & 31) * 16);
    }
    __device__ __forceinline__ unsigned entry(int u) const { return u < NUIN ? P.order[u >> 1] : 0xff4000u; }
    __device__ __forceinline__ unsigned fwd_off(int u) const {
        const unsigned f = (unsigned)(u - NUIN), t = (unsigned)P.rank * NOWN + (f >> 4);
        return ((t & 15u) * 256u + (f & 15u) * 16u + (unsigned)wid * 2u) * (unsigned)(N * 4) + (DPAN + (t >> 4)) * 1024u;
    }
    __device__ __forceinline__ void stage(int u, unsigned b) const {
        const unsigned e = entry(u), hh = (unsigned)u & 1u, j = (e >> 8) & 3u, pan = e & 127u;
        const unsigned char* s; unsigned step;
        if (e & 0x8000u) { s = P.w + (size_t)(j * 128u + hh * 64u) * N + pan * 256u + w_src; step = 16u * N; }
        else if (e & 0x4000u) { s = P.out[P.rank] + fwd_off(u) + (unsigned)lane * 16u; step = N * 4u; }
        else { s = P.x + (size_t)(pan * 256u) * KL + j * 128u + hh * 64u + x_src; step = 128u * KL; }
        __builtin_amdgcn_global_load_lds((const GAS void*)s, (LAS void*)(lds + b), 16, 0, 17);
        __builtin_amdgcn_global_load_lds((const GAS void*)(s + step), (LAS void*)(lds + b + 1024), 16, 0, 17);
    }
    __device__ __forceinline__ u32x4 fetch(int u, unsigned b, int k, unsigned& off) const {
        const unsigned e = entry(u), hh = (unsigned)u & 1u, j = (e >> 8) & 3u, pan = e & 127u, slot = j * 8u + (unsigned)P.rank;
        const unsigned lb = (unsigned)(size_t)lds + b;
        const unsigned a = lb + (unsigned)k * 1024u + (unsigned)lane * 16u;
        if (e & 0x8000u) {
            if (k == 0) {
                v2i g0, g1, g2, g3;
                const unsigned ra = lb + tr_off, wa = lb + sc_off;
                asm volatile("ds_read_b64_tr_b8 %0, %4\n\tds_read_b64_tr_b8 %1, %4 offset:512\n\t"
                             "ds_read_b64_tr_b8 %2, %4 offset:1024\n\tds_read_b64_tr_b8 %3, %4 offset:1536\n\ts_waitcnt lgkmcnt(0)\n\t"
                             "ds_write_b64 %5, %0\n\tds_write_b64 %5, %1 offset:8\n\tds_write_b64 %5, %2 offset:256\n\tds_write_b64 %5, %3 offset:264\n\t"
                             "s_waitcnt lgkmcnt(0)"
                             : "=&v"(g0), "=&v"(g1), "=&v"(g2), "=&v"(g3) : "v"(ra), "v"(wa) : "memory");
            }
            off = X_BYTES + slot * WS_SLAB + (pan * 16u + (unsigned)k * 8u) * 2048u + hh * 1024u + w_dst;
        } else if (e & 0x4000u) {
            off = fwd_off(u) + (unsigned)k * (unsigned)(N * 4) + (unsigned)lane * 16u;
        } else {
            off = slot * XS_SLAB + (pan * 16u + (unsigned)wid + (unsigned)k * 8u) * 2048u + hh * 1024u + (unsigned)lane * 16u;
        }
        u32x4 v;
        asm volatile("ds_read_b128 %0, %1\n\ts_waitcnt lgkmcnt(0)" : "=&v"(v) : "v"(a) : "memory");
        return v;
    }
    __device__ __forceinline__ unsigned flag_off(int u) const {
        const unsigned e = P.order[u >> 1], j = (e >> 8) & 3u, pan = e & 127u, slot = j * 8u + (unsigned)P.rank;
        return ((e & 0x8000u) ? FW_OFF : FX_OFF) + pan * 128u + slot * 4u;
    }
};
#define STORE_WT(off, v, base) asm volatile("s_nop 4\n\tglobal_store_dwordx4 %0, %1, %2 sc0 sc1\n\ts_nop 1" :: "v"(off), "v"(v), "s"(base) : "memory")

__device__ __forceinline__ void pusher(const Params& P, LAS unsigned char* lds, int wid, int lane, int pi) {
    const Mover mv(P, lds, wid, lane);
    const unsigned wbuf = (unsigned)wid * 4096u;
    mv.stage(pi, wbuf);
    __builtin_amdgcn_s_setprio(3);
    unsigned char* const wsl = P.ws[P.rank];
    unsigned call;
    {
        unsigned t = 0;
        if (lane == 0) t = __hip_atomic_fetch_add((unsigned*)(wsl + CTR_OFF) + pi * 16 + wid, 1u, __ATOMIC_RELAXED, __HIP_MEMORY_SCOPE_AGENT);
        call = rfl(t) + 1u;
    }
    unsigned* const tmo = (unsigned*)(wsl + TMO_OFF);
    unsigned char* dst[NR - 1];
    unsigned pbit[NR - 1];
    auto bases = [&](bool to_out) {
#pragma unroll
        for (int q = 0; q < NR - 1; ++q) {
            const unsigned p = (unsigned)(P.rank + 1 + (q + wid + pi) % (NR - 1)) & 7u;
            dst[q] = rfl_ptr(to_out ? P.out[p] : P.ws[p] + DATA_OFF + (size_t)(call & 1u) * PAR_BYTES);
            pbit[q] = 0x10000u << p;
        }
    };
    bases(false);
    const unsigned peer = (unsigned)(P.rank + 1 + lane) & 7u;
    unsigned char* flag_base = P.ws[peer];
    asm volatile("" : "+v"(flag_base));
    unsigned* const tick = (unsigned*)(wsl + TICK_OFF + (call & 1u) * 128u);
    const unsigned tbox = (unsigned)(size_t)lds + 65536u;
    auto signal = [&](int uu, int i) {
        if (uu < NUIN && wid == (i & 7)) {
            if (lane < NR - 1 && ((P.order[uu >> 1] >> 16) >> peer & 1u))
                __hip_atomic_fetch_add((unsigned*)(flag_base + mv.flag_off(uu)), 1u, __ATOMIC_RELAXED, __HIP_MEMORY_SCOPE_SYSTEM);
        }
    };
    bool fwd = false, bad = false;
    int uprev = -1, u = pi, unext = pi + NPUSH;
    unsigned nfwd = 0u;
    WAIT_V(0);
    for (int i = 0; u < NUNIT; ++i) {
        if (unext < NUNIT) {
            if (unext >= NUIN && !fwd) {
                fwd = true;
                bad |= wait_words((const unsigned*)(wsl + TDONE_OFF) + (lane < NOWN ? lane : 0), 8u * call, lane < NOWN, tmo, call, lane);
            }
            mv.stage(unext, wbuf + (unsigned)((i + 1) & 1) * 2048u);
        }
        if (wid == 0 && lane == 0) { const unsigned one = 1u; asm volatile("global_atomic_add v255, %0, %1, off sc0" :: "v"(tick), "v"(one) : "memory", "v255"); }
        if (u >= NUIN) { if (uprev < NUIN) bases(true); ++nfwd; }
        const unsigned e = mv.entry(u);
        const bool full = ((e | (0x10000u << P.rank)) >> 16) == 0xffu;
        int un2 = 0;
        auto meet = [&]() {
            if (wid == 0 && lane == 0) asm volatile("ds_write_b32 %0, v255\n\ts_waitcnt lgkmcnt(0)" :: "v"(tbox + (unsigned)(i & 1) * 4u) : "memory");
            BAR;
            unsigned t;
            asm volatile("ds_read_b32 %0, %1\n\ts_waitcnt lgkmcnt(0)" : "=&v"(t) : "v"(tbox + (unsigned)(i & 1) * 4u) : "memory");
            un2 = (int)rfl(t) + 2 * NPUSH;
            if (uprev >= 0) signal(uprev, i);
        };
        for (int k = 0; k < 2; ++k) {
            unsigned off;
            const u32x4 v = mv.fetch(u, wbuf + (unsigned)(i & 1) * 2048u, k, off);
#pragma unroll
            for (int q = 0; q < NR - 1; ++q) {
                if (e & pbit[q]) {
                    STORE_WT(off, v, dst[q]);
                    asm volatile("s_waitcnt vmcnt(%0)" :: "n"(PUSH_WIN) : "memory");
                }
                if (q == (PUSH_WIN - 1) % (NR - 1) && k == (PUSH_WIN - 1) / (NR - 1) && full) meet();
            }
        }
        if (!full) {
            const int c = __builtin_popcount((e & ~(0x10000u << P.rank)) >> 16);
            if (c >= 3) WAIT_V(6); else if (c == 2) WAIT_V(4); else if (c == 1) WAIT_V(2); else WAIT_V(0);
            meet();
        }
        uprev = u; u = unext; unext = un2;
    }
    WAIT_V(0);
    BAR;
    signal(uprev, 0);
    if (wid == 0) {
        if (lane < NR - 1 && nfwd) __hip_atomic_fetch_add((unsigned*)(flag_base + DONE_OFF + (unsigned)P.rank * 128u), nfwd, __ATOMIC_RELAXED, __HIP_MEMORY_SCOPE_SYSTEM);
        STAMP(1);
        if (pi == 0) {
            bad |= wait_words((const unsigned*)(wsl + DONE_OFF + peer * 128u), (unsigned)(NUNIT - NUIN) * call, lane < NR - 1, tmo, call, lane);
            if (lane < 8) __hip_atomic_store((unsigned*)(wsl + ROLE_OFF + ((call & 1u) ^ 1u) * 1024u + (unsigned)lane * 128u), 0u, __ATOMIC_RELAXED, __HIP_MEMORY_SCOPE_AGENT);
            if (lane == 8) __hip_atomic_store((unsigned*)(wsl + TICK_OFF + ((call & 1u) ^ 1u) * 128u), 0u, __ATOMIC_RELAXED, __HIP_MEMORY_SCOPE_AGENT);
            STAMP(2);
        }
    }
    if (bad && lane == 0) __hip_atomic_store((unsigned*)P.out[P.rank], 0x7fc00000u, __ATOMIC_RELAXED, __HIP_MEMORY_SCOPE_SYSTEM);
}
static_assert(PUSH_WIN >= 1 && PUSH_WIN <= 2 * (NR - 1), "window");

__device__ __forceinline__ void local_copy(const Params& P, const Mover& mv, unsigned call, int wid, int lane, int li) {
    const int nun = li + 2 * NGEMM < NUIN ? 3 : 2;
    const unsigned wbuf = (unsigned)wid * 8192u;
    for (int i = 0; i < nun; ++i) mv.stage(li + NGEMM * i, wbuf + (unsigned)i * 2048u);
    unsigned char* const wsl = P.ws[P.rank];
    unsigned char* const dst = rfl_ptr(wsl + DATA_OFF + (size_t)(call & 1u) * PAR_BYTES);
    WAIT_V(0);
    for (int i = 0; i < nun; ++i)
        for (int k = 0; k < 2; ++k) {
            unsigned off;
            const u32x4 v = mv.fetch(li + NGEMM * i, wbuf + (unsigned)i * 2048u, k, off);
            STORE_WT(off, v, dst);
        }
    WAIT_V(0);
    __syncthreads();
    if (wid < nun && lane == 0) __hip_atomic_fetch_add((unsigned*)(wsl + mv.flag_off(li + NGEMM * wid)), 1u, __ATOMIC_RELAXED, __HIP_MEMORY_SCOPE_SYSTEM);
}

__device__ __forceinline__ void gemm_role(const Params& P, LAS unsigned char* lds, const int wid, const int lane) {
    unsigned char* const wsl = P.ws[P.rank];
    if (wid == 0) {
        const unsigned xcc = (unsigned)__builtin_amdgcn_s_getreg((3 << 11) | 20) & 7u;
        unsigned t = 0;
        if (lane < 3) t = __hip_atomic_fetch_add((unsigned*)(wsl + (lane == 0 ? CTR_OFF + blockIdx.x * 64u : ROLE_OFF + (unsigned)(lane - 1) * 1024u + xcc * 128u)),
                                                 1u, __ATOMIC_RELAXED, __HIP_MEMORY_SCOPE_AGENT);
        const unsigned c = (unsigned)__builtin_amdgcn_readlane((int)t, 0) + 1u;
        unsigned tk = (c & 1u) ? (unsigned)__builtin_amdgcn_readlane((int)t, 2) : (unsigned)__builtin_amdgcn_readlane((int)t, 1), x = xcc;
        for (int k = 1; k < 8 && tk >= 30u; ++k) {
            x = (xcc + (unsigned)k) & 7u;
            unsigned t2 = 0;
            if (lane == 0) t2 = __hip_atomic_fetch_add((unsigned*)(wsl + ROLE_OFF + (c & 1u) * 1024u + x * 128u), 1u, __ATOMIC_RELAXED, __HIP_MEMORY_SCOPE_AGENT);
            tk = rfl(t2);
        }
        if (lane == 0) *(LAS v2i*)(lds + LDS_MBOX) = (v2i){(int)c, (int)(tk < 30u ? x * 30u + tk : 256u)};
    }
    __syncthreads();
    const v2i cr = *(const LAS v2i*)(lds + LDS_MBOX);
    const unsigned call = rfl((unsigned)cr.x);
    const int role = (int)rfl((unsigned)cr.y);
    if (role >= NGEMM) return;
    const int band = role / 60, i60 = role - band * 60, g = i60 >> 2, prow = band * 4 + (i60 & 3);
    { const Mover mv(P, lds, wid, lane); local_copy(P, mv, call, wid, lane, role); }
    STAMP(11);

    u64 tiles = 0; int nseg = 0;
    for (int r = 0; r < 4; ++r) {
        const int sidx = r * 15 + g;
        unsigned t;
        if (sidx == SPSLOT) {
            const int lo = prow & 7, a = (prow >> 3) * 5 + (lo < 3 ? lo : lo - 1);
            if (lo == 3 || lo > 5) continue;
            const unsigned o = (unsigned)(P.rank * NOWN + a);
            t = (o & 15u) | (DPAN + (o >> 4)) << 4 | (unsigned)(a + 1) << 10;
        } else t = (unsigned)prow | (unsigned)(sidx - (sidx > SPSLOT)) << 4;
        tiles |= (u64)t << (16 * nseg); ++nseg;
    }
    auto tile_lane = [&](int sg) { return (unsigned)(tiles >> (16 * (sg & 3))) & 0xffffu; };
    auto tile_of = [&](int sg) { return rfl(tile_lane(sg)); };
    const int wr = wid >> 2, wc = wid & 3;
    const unsigned thr = 2u * call;
    unsigned char* const dbase = rfl_ptr(wsl + DATA_OFF + (size_t)(call & 1u) * PAR_BYTES);
    const __amdgpu_buffer_rsrc_t rsX = __builtin_amdgcn_make_buffer_rsrc((void*)dbase, 0, X_BYTES, 0x00020000);
    const __amdgpu_buffer_rsrc_t rsW = __builtin_amdgcn_make_buffer_rsrc((void*)(dbase + X_BYTES), 0, W_BYTES, 0x00020000);
    const __amdgpu_buffer_rsrc_t rsF = __builtin_amdgcn_make_buffer_rsrc((void*)wsl, 0, DATA_OFF, 0x00020000);
    const unsigned voff = (unsigned)(wid * 2048 + lane * 16);
    const unsigned ldsw = (unsigned)wid * 2048u;
    int aoff = wr * 8192 + lane * 16, boff = LDS_B + wc * 4096 + lane * 16;
    asm volatile("" : "+v"(aoff), "+v"(boff));
    const unsigned snap_src = (lane < 32 ? FW_OFF + ((tile_lane(lane >> 3) >> 4) & 63u) * 128u : FX_OFF + (tile_lane(lane >> 3) & 15u) * 128u) + (unsigned)(lane & 7) * 16u;
    const unsigned snap_rd = lane < 32 ? (unsigned)lane * 4u : 384u + (unsigned)lane * 4u;

#define DMA(rs, ldsoff, vo, so, imm) __builtin_amdgcn_raw_ptr_buffer_load_lds(rs, (LAS void*)(lds + (ldsoff)), 16, vo, so, imm, 17)
#define STAGE_A(b, h, so) do { DMA(rsX, ((b) * 2 + (h)) * HT + ldsw, voff, (so) + (h) * HT, 0); DMA(rsX, ((b) * 2 + (h)) * HT + ldsw, voff, (so) + (h) * HT, 1024); } while (0)
#define STAGE_B(b, h, so) do { DMA(rsW, LDS_B + ((b) * 2 + (h)) * HT + ldsw, voff, (so) + (h) * HT, 0); DMA(rsW, LDS_B + ((b) * 2 + (h)) * HT + ldsw, voff, (so) + (h) * HT, 1024); } while (0)
#define LDF(off) __builtin_shufflevector(*(const LAS v4i*)(lds + (off)), *(const LAS v4i*)(lds + (off) + 1024), 0, 1, 2, 3, 4, 5, 6, 7)
#define LDA(b, h) do { _Pragma("unroll") for (int m = 0; m < 4; ++m) At[m] = LDF(((b) * 2 + (h)) * HT + aoff + m * 2048); } while (0)
#define LDB(dst, b, h) do { _Pragma("unroll") for (int n = 0; n < 2; ++n) dst[n] = LDF(((b) * 2 + (h)) * HT + boff + n * 2048); } while (0)
#define MF(c, bb, aa) asm volatile("v_mfma_f32_16x16x128_f8f6f4 %0, %1, %2, %0 cbsz:1 blgp:1" : "+v"(c) : "v"(aa), "v"(bb))
#define MMA(ai, bj, Bt) do { __builtin_amdgcn_s_setprio(1); _Pragma("unroll") for (int m = 0; m < 4; ++m) _Pragma("unroll") for (int n = 0; n < 2; ++n) \
        MF(acc[ai * 4 + m][bj * 2 + n], Bt[n], At[m]); __builtin_amdgcn_s_setprio(0); } while (0)

    f32x4 acc[8][4];
    v8i At[4], B0[2], B1[2];
#pragma unroll
    for (int m = 0; m < 8; ++m)
#pragma unroll
        for (int n = 0; n < 4; ++n) acc[m][n] = (f32x4){0.f, 0.f, 0.f, 0.f};

    unsigned xm = 0u, wm = 0u, used = 0u, bad = 0u;
    int ts = 0;
    unsigned a_pan = (tile_of(0) & 15u) * 32768u, b_pan = ((tile_of(0) >> 4) & 63u) * 32768u;
    auto slow_wait = [&]() {
        if (wid == 0) {
            const unsigned* fp = (const unsigned*)(wsl + (lane < 32 ? FW_OFF + (b_pan >> 8) + (unsigned)lane * 4u : FX_OFF + (a_pan >> 8) + (unsigned)(lane - 32) * 4u));
            unsigned* tmo = (unsigned*)(wsl + TMO_OFF);
            const u64 t0 = __builtin_amdgcn_s_memrealtime();
            u64 mk; unsigned dead = 0u;
            for (;;) {
                const unsigned v = __hip_atomic_load(fp, __ATOMIC_RELAXED, __HIP_MEMORY_SCOPE_SYSTEM);
                mk = __builtin_amdgcn_ballot_w64((int)(v - thr) >= 0);
                if (__builtin_popcount((unsigned)mk & (unsigned)(mk >> 32) & ~used) >= 2) break;
                if (rfl(__hip_atomic_load(tmo, __ATOMIC_RELAXED, __HIP_MEMORY_SCOPE_AGENT)) == call || __builtin_amdgcn_s_memrealtime() - t0 > DEADLINE_TICKS) {
                    if (lane == 0) __hip_atomic_store(tmo, call, __ATOMIC_RELAXED, __HIP_MEMORY_SCOPE_AGENT);
                    mk = ~0ull; dead = 1u; break;
                }
                __builtin_amdgcn_s_sleep(4);
            }
            if (lane == 0) *(LAS u32x4*)(lds + LDS_MBOX + 16) = (u32x4){(unsigned)mk, (unsigned)(mk >> 32), dead, 0u};
            asm volatile("s_waitcnt vmcnt(0) lgkmcnt(0)" ::: "memory");
        }
        asm volatile("" ::: "memory"); BAR; asm volatile("" ::: "memory");
        const u32x4 mb = *(const LAS u32x4*)(lds + LDS_MBOX + 16);
        wm |= rfl(mb.x); xm |= rfl(mb.y); bad |= rfl(mb.z);
    };
    unsigned a2, b2, a3, b3;
    auto take = [&](unsigned av) {
        const unsigned sA = (unsigned)__builtin_ctz(av), sB = (unsigned)__builtin_ctz(av & (av - 1u));
        used |= (1u << sA) | (1u << sB);
        a2 = sA * XS_SLAB + a_pan; b2 = sA * WS_SLAB + b_pan; a3 = sB * XS_SLAB + a_pan; b3 = sB * WS_SLAB + b_pan;
    };

    const float sl = __builtin_bit_cast(float, rfl(__builtin_bit_cast(unsigned, (P.sx[0] * P.sw[0]) * -1.44269504089f)));
    slow_wait(); take(xm & wm);
    STAMP(1);
    unsigned a1 = a3;
    STAGE_A(0, 1, a2); STAGE_B(0, 0, b2); STAGE_A(0, 0, a2); STAGE_B(0, 1, b2);
    if (wr == 1) BAR;
    WAIT_V(2); BAR;
    STAGE_B(1, 0, b3); STAGE_A(1, 0, a3); STAGE_B(1, 1, b3);
    WAIT_V(6); BAR;
    LDB(B0, 0, 0);
    bool ready;
    { const unsigned av = xm & wm & ~used; ready = (av & (av - 1u)) != 0u; if (ready) take(av); }
    for (int seg = 0; seg < nseg; ++seg) {
        for (int it = 0; it < 16; ++it) {
            if (!ready) { slow_wait(); take(xm & wm & ~used); }
            const unsigned snap = LDS_SNAP + (unsigned)(it & 1) * 1024u;
            LDA(0, 0); STAGE_A(1, 1, a1);
            if (wid == 0) DMA(rsF, snap, snap_src, 0, 0);
            BAR; WAIT_L(0); SCHED; MMA(0, 0, B0); BAR; SCHED;
            LDB(B1, 0, 1); STAGE_B(0, 0, b2);
            BAR; WAIT_L(0); SCHED; MMA(0, 1, B1); BAR; SCHED;
            LDA(0, 1); STAGE_A(0, 0, a2);
            WAIT_V(10); BAR; WAIT_L(0); SCHED; MMA(1, 0, B0); BAR; SCHED;
            LDB(B0, 1, 0); STAGE_B(0, 1, b2);
            WAIT_V(6); BAR; SCHED; MMA(1, 1, B1); BAR; SCHED;
            LDA(1, 0); STAGE_A(0, 1, a2);
            BAR; WAIT_L(0); SCHED; MMA(0, 0, B0); BAR; SCHED;
            LDB(B1, 1, 1); STAGE_B(1, 0, b3);
            const unsigned snapv = *(const LAS unsigned*)(lds + snap + (unsigned)((it == 14 ? ts + 1 : ts) & 3) * 128u + snap_rd);
            BAR; WAIT_L(0); SCHED; MMA(0, 1, B1); BAR; SCHED;
            LDA(1, 1); STAGE_A(1, 0, a3);
            WAIT_V(10); BAR; WAIT_L(0); SCHED; MMA(1, 0, B0); BAR; SCHED;
            const u64 mk = __builtin_amdgcn_ballot_w64((int)(snapv - thr) >= 0);
            SCHED;
            LDB(B0, 0, 0); STAGE_B(1, 1, b3);
            a1 = a3;
            if (it == 14) {
                ++ts; wm = 0u; xm = 0u; used = 0u;
                const unsigned t = tile_of(ts);
                a_pan = (t & 15u) * 32768u; b_pan = ((t >> 4) & 63u) * 32768u;
            }
            wm |= (unsigned)mk; xm |= (unsigned)(mk >> 32);
            if (ts >= nseg) { wm = xm = ~0u; used = 0u; }
            { const unsigned av = xm & wm & ~used; ready = (av & (av - 1u)) != 0u; if (ready) take(av); }
            WAIT_V(6); BAR; SCHED; MMA(1, 1, B1); BAR; SCHED;
        }
        STAMP(2 * seg + 2);
        SCHED; asm volatile("s_nop 15\n\ts_nop 15" ::: "memory"); SCHED;
        const unsigned tcur = tile_of(seg);
        {
            int tz = threadIdx.x; asm volatile("" : "+v"(tz));
            const unsigned ov = (unsigned)((tz >> 4) & 3) * (unsigned)(4 * N * 4) + (unsigned)(tz & 15) * 16u;
            const char* ob = (const char*)rfl_ptr(P.out[P.rank] + (size_t)((tcur & 15u) * 256u + (unsigned)wr * 64u) * (N * 4) + (((tcur >> 4) & 63u) * 256u + (unsigned)wc * 64u) * 4u);
            asm volatile("s_nop 4" : "+s"(ob));
            const float cc = bad ? __builtin_nanf("") : -1.44269504089f;
#pragma unroll
            for (int m = 0; m < 8; ++m) {
                const char* rowp = ob + (size_t)((m >> 2) * 128 + (m & 3) * 16) * (N * 4);
#pragma unroll
                for (int j = 0; j < 4; ++j) {
                    const f32x4 o = silu4(acc[m][0][j], acc[m][1][j], acc[m][2][j], acc[m][3][j], sl, cc);
                    asm volatile("global_store_dwordx4 %0, %1, %2" OUT_POL "\n\ts_nop 1" :: "v"(ov), "v"(o), "s"(rowp + (size_t)j * (N * 4)) : "memory");
                }
#pragma unroll
                for (int n = 0; n < 4; ++n) acc[m][n] = (f32x4){0.f, 0.f, 0.f, 0.f};
                if (m == 0 && wr == 0) BAR;
            }
        }
        if (tcur >> 10) {
            WAIT_V(0);
            if (lane == 0) __hip_atomic_fetch_add((unsigned*)(wsl + TDONE_OFF) + ((tcur >> 10) - 1u), 1u, __ATOMIC_RELAXED, __HIP_MEMORY_SCOPE_AGENT);
        }
        STAMP(2 * seg + 3);
        if (wr == 1) BAR;
    }
    WAIT_V(0);
    if (wr == 0) BAR;
    STAMP(10);
}

__global__ __launch_bounds__(NTHR, 2) void fused_kernel(const Params P) {
    extern __shared__ __attribute__((aligned(16))) unsigned char shm[];
    LAS unsigned char* lds = (LAS unsigned char*)shm;
    const int wid = __builtin_amdgcn_readfirstlane(threadIdx.x >> 6), lane = threadIdx.x & 63;
    if (blockIdx.x < NPUSH) pusher(P, lds, wid, lane, blockIdx.x);
    else gemm_role(P, lds, wid, lane);
}

constexpr double g_tune[8] = {1.5, 2.0, 10.0, 8.0, 45.0, 0.0, 0.0, 2.0};
void build_order(int s, unsigned* out) {
    struct It { double key; unsigned code; };
    It it[NCHUNK]; int n = 0;
    const double delta = g_tune[0], tau0 = g_tune[1], lead = g_tune[2], xs = g_tune[3], Tr = g_tune[4], T1 = 32 * tau0 + g_tune[5];
    const int grot = 3, xrot = (int)g_tune[7];
    const double early = g_tune[6];
    for (int j = 0; j < 4; ++j)
        for (int p = 0; p < 16; ++p)
            it[n++] = It{8 * j * tau0 + xs * tau0 * (((p + xrot * s) & 15) / 16.0) - 0.01, 0xff0000u | (unsigned)((j << 8) | p)};
    for (int c = 0; c < 64; ++c) {
        const int sidx = c >= DPAN ? SPSLOT : c + (c >= SPSLOT), r = sidx / 15, g = sidx % 15;
        unsigned mask = 0xffu;
        if (c >= DPAN) { mask = 0; for (int t = (c - DPAN) * 16; t < (c - DPAN) * 16 + 16; ++t) mask |= 1u << (t / NOWN); }
        for (int j = 0; j < 4; ++j) {
            const int pos = 8 * j + (grot ? ((s + g * grot + (g >> 3) * 4) & 7) : 0);
            const double st = g >= 7 ? (g - 7) * delta : g * delta + lead;
            const double key = (r == 0 ? st + pos * tau0 : T1 + (r - 1 - (sidx > SPSLOT && g == SPSLOT % 15 ? early : 0.0)) * Tr + st + pos * (Tr / 34.0)) + 1e-4 * c;
            it[n++] = It{key, mask << 16 | 0x8000u | (unsigned)((j << 8) | c)};
        }
    }
    std::stable_sort(it, it + NCHUNK, [](const It& a, const It& b) { return a.key < b.key; });
    for (int i = 0; i < NCHUNK; ++i) out[i] = it[i].code;
}
}

extern "C" void kernel_launch(int rank, int nranks, ncclComm_t comm,
                              hipStream_t stream,
                              void* const* d_in, const int* in_sizes, int n_in,
                              void* d_out, int out_size,
                              const int* in_dtypes, int out_dtype,
                              void* const* d_out_all,
                              void* const* d_ws_all, size_t ws_bytes) {
    (void)comm; (void)in_dtypes; (void)out_dtype;
    if (nranks != NR || (unsigned)rank >= (unsigned)NR || n_in != 4 || ws_bytes < WS_NEED || out_size != M * N ||
        in_sizes[0] != M * KL || in_sizes[1] != KL * N) {
        fprintf(stderr, "kernel_launch: unexpected problem shape\n");
        return;
    }
    Params P;
    P.x = (const unsigned char*)d_in[0]; P.w = (const unsigned char*)d_in[1];
    P.sx = (const float*)d_in[2]; P.sw = (const float*)d_in[3];
    for (int p = 0; p < NR; ++p) { P.ws[p] = (unsigned char*)d_ws_all[p]; P.out[p] = (unsigned char*)d_out_all[p]; }
    P.out[rank] = (unsigned char*)d_out;
    P.rank = rank; P.pad = 0;
    build_order(rank, P.order);
    HIP_CHECK(hipFuncSetAttribute((const void*)fused_kernel, hipFuncAttributeMaxDynamicSharedMemorySize, LDS_BYTES));
    hipLaunchKernelGGL(fused_kernel, dim3(NWG), dim3(NTHR), LDS_BYTES, stream, P);
    HIP_CHECK(hipGetLastError());
}
```

```cpp
#include <hip/hip_runtime.h>
#include <rccl/rccl.h>
#include <cstdio>
#include <algorithm>

#define HIP_CHECK(expr) do { hipError_t _e = (expr); if (_e != hipSuccess) { \
    fprintf(stderr, "HIP error %s at %s:%d\n", hipGetErrorString(_e), __FILE__, __LINE__); \
    return; } } while (0)

namespace {
#define LAS __attribute__((address_space(3)))
#define GAS __attribute__((address_space(1)))
typedef int v8i __attribute__((ext_vector_type(8)));
typedef int v4i __attribute__((ext_vector_type(4)));
typedef int v2i __attribute__((ext_vector_type(2)));
typedef unsigned u32x4 __attribute__((ext_vector_type(4)));
typedef float f32x4 __attribute__((ext_vector_type(4)));
typedef float f32x2 __attribute__((ext_vector_type(2)));
typedef unsigned long long u64;

constexpr int M = 4096, N = 16384, KL = 512, NR = 8;
constexpr int NPUSH = 16, NWG = 256, NGEMM = NWG - NPUSH, NTHR = 512;
constexpr int NCHUNK = 4 * 16 + 4 * 64;
constexpr int NUIN = 2 * NCHUNK;
constexpr int DPAN = 59, NOWN = 10;
constexpr int NUNIT = NUIN + NOWN * 16;
constexpr int SPSLOT = 15 + 14;
constexpr unsigned XS_SLAB = 256u * 2048u;
constexpr unsigned WS_SLAB = 1024u * 2048u;
constexpr unsigned X_BYTES = 32u * XS_SLAB, W_BYTES = 32u * WS_SLAB, PAR_BYTES = X_BYTES + W_BYTES;
constexpr unsigned CTR_OFF = 0;
constexpr unsigned TMO_OFF = 32768;
constexpr unsigned ROLE_OFF = 36864;
constexpr unsigned TICK_OFF = 40960;
constexpr unsigned TDONE_OFF = 45056;
constexpr unsigned DONE_OFF = 49152;
constexpr unsigned FX_OFF = 65536;
constexpr unsigned FW_OFF = 73728;
constexpr unsigned DBG_OFF = 131072;
constexpr unsigned DATA_OFF = 1u << 20;
constexpr size_t WS_NEED = (size_t)DATA_OFF + 2ull * PAR_BYTES;
constexpr u64 DEADLINE_TICKS = 100000000ull;
constexpr int HT = 16384, LDS_B = 4 * HT, LDS_SNAP = 8 * HT, LDS_MBOX = LDS_SNAP + 2048, LDS_BYTES = LDS_MBOX + 64;
#define OUT_POL " sc0 sc1"
#define PUSH_WIN 7

struct Params {
    const unsigned char* x; const unsigned char* w; const float* sx; const float* sw;
    unsigned char* ws[NR]; unsigned char* out[NR];
    int rank; int pad;
    unsigned order[NCHUNK];
};

__device__ __forceinline__ unsigned rfl(unsigned v) { return (unsigned)__builtin_amdgcn_readfirstlane((int)v); }
__device__ __forceinline__ unsigned char* rfl_ptr(unsigned char* p) {
    return (unsigned char*)(((u64)rfl((unsigned)((u64)p >> 32)) << 32) | (u64)rfl((unsigned)(u64)p));
}
__device__ __forceinline__ float mul1(float a, float b) { float r; asm("v_mul_f32 %0, %1, %2" : "=v"(r) : "s"(b), "v"(a)); return r; }
__device__ __forceinline__ f32x4 silu4(float a0, float a1, float a2, float a3, float sl, float c) {
    const f32x2 t0 = {mul1(a0, sl), mul1(a1, sl)}, t1 = {mul1(a2, sl), mul1(a3, sl)};
    const float L = 86.56170245f;
    f32x2 e0, e1;
    e0.x = __builtin_amdgcn_exp2f(__builtin_amdgcn_fmed3f(t0.x, -L, L)); e0.y = __builtin_amdgcn_exp2f(__builtin_amdgcn_fmed3f(t0.y, -L, L));
    e1.x = __builtin_amdgcn_exp2f(__builtin_amdgcn_fmed3f(t1.x, -L, L)); e1.y = __builtin_amdgcn_exp2f(__builtin_amdgcn_fmed3f(t1.y, -L, L));
    const f32x2 cv = {c, c};
    const f32x2 d0 = __builtin_elementwise_fma(e0, cv, cv), d1 = __builtin_elementwise_fma(e1, cv, cv);
    const f32x2 r0 = {__builtin_amdgcn_rcpf(d0.x), __builtin_amdgcn_rcpf(d0.y)}, r1 = {__builtin_amdgcn_rcpf(d1.x), __builtin_amdgcn_rcpf(d1.y)};
    const f32x2 o0 = t0 * r0, o1 = t1 * r1;
    return (f32x4){o0.x, o0.y, o1.x, o1.y};
}

#define WAIT_V(n) asm volatile("s_waitcnt vmcnt(" #n ")" ::: "memory")
#define WAIT_L(n) asm volatile("s_waitcnt lgkmcnt(" #n ")" ::: "memory")
#define BAR __builtin_amdgcn_s_barrier()
#define SCHED __builtin_amdgcn_sched_barrier(0)
#define STAMP(i)

__device__ __forceinline__ bool wait_words(const unsigned* p, unsigned target, bool active, unsigned* tmo, unsigned call, int lane) {
    const u64 t0 = __builtin_amdgcn_s_memrealtime();
    for (;;) {
        const unsigned v = __hip_atomic_load(p, __ATOMIC_RELAXED, __HIP_MEMORY_SCOPE_SYSTEM);
        if (__builtin_amdgcn_ballot_w64(active && (int)(v - target) < 0) == 0ull) return false;
        if (rfl(__hip_atomic_load(tmo, __ATOMIC_RELAXED, __HIP_MEMORY_SCOPE_AGENT)) == call || __builtin_amdgcn_s_memrealtime() - t0 > DEADLINE_TICKS) {
            if (lane == 0) __hip_atomic_store(tmo, call, __ATOMIC_RELAXED, __HIP_MEMORY_SCOPE_AGENT);
            return true;
        }
        __builtin_amdgcn_s_sleep(4);
    }
}

struct Mover {
    const Params& P; LAS unsigned char* lds; int wid, lane;
    unsigned w_src, x_src, tr_off, sc_off, w_dst;
    __device__ __forceinline__ Mover(const Params& P_, LAS unsigned char* lds_, int wid_, int lane_) : P(P_), lds(lds_), wid(wid_), lane(lane_) {
        w_src = (unsigned)(((wid >> 2) * 32 + (lane >> 2)) * N + (wid & 3) * 64 + (lane & 3) * 16);
        x_src = (unsigned)((wid * 16 + (lane & 15)) * KL + (lane >> 4) * 16);
        tr_off = (unsigned)(((lane & 15) >> 1) * 64 + (lane >> 4) * 16 + (lane & 1) * 8);
        sc_off = (unsigned)((lane & 3) * 512 + (lane >> 2) * 16);
        w_dst = (unsigned)(((wid & 3) * 2 + (lane >> 5)) * 2048 + (wid >> 2) * 512 + (lane & 31) * 16);
    }
    __device__ __forceinline__ unsigned entry(int u) const { return u < NUIN ? P.order[u >> 1] : 0xff4000u; }
    __device__ __forceinline__ unsigned fwd_off(int u) const {
        const unsigned f = (unsigned)(u - NUIN), t = (unsigned)P.rank * NOWN + (f >> 4);
        return ((t & 15u) * 256u + (f & 15u) * 16u + (unsigned)wid * 2u) * (unsigned)(N * 4) + (DPAN + (t >> 4)) * 1024u;
    }
    __device__ __forceinline__ void stage(int u, unsigned b) const {
        const unsigned e = entry(u), hh = (unsigned)u & 1u, j = (e >> 8) & 3u, pan = e & 127u;
        const unsigned char* s; unsigned step;
        if (e & 0x8000u) { s = P.w + (size_t)(j * 128u + hh * 64u) * N + pan * 256u + w_src; step = 16u * N; }
        else if (e & 0x4000u) { s = P.out[P.rank] + fwd_off(u) + (unsigned)lane * 16u; step = N * 4u; }
        else { s = P.x + (size_t)(pan * 256u) * KL + j * 128u + hh * 64u + x_src; step = 128u * KL; }
        __builtin_amdgcn_global_load_lds((const GAS void*)s, (LAS void*)(lds + b), 16, 0, 17);
        __builtin_amdgcn_global_load_lds((const GAS void*)(s + step), (LAS void*)(lds + b + 1024), 16, 0, 17);
    }
    __device__ __forceinline__ u32x4 fetch(int u, unsigned b, int k, unsigned& off) const {
        const unsigned e = entry(u), hh = (unsigned)u & 1u, j = (e >> 8) & 3u, pan = e & 127u, slot = j * 8u + (unsigned)P.rank;
        const unsigned lb = (unsigned)(size_t)lds + b;
        const unsigned a = lb + (unsigned)k * 1024u + (unsigned)lane * 16u;
        if (e & 0x8000u) {
            if (k == 0) {
                v2i g0, g1, g2, g3;
                const unsigned ra = lb + tr_off, wa = lb + sc_off;
                asm volatile("ds_read_b64_tr_b8 %0, %4\n\tds_read_b64_tr_b8 %1, %4 offset:512\n\t"
                             "ds_read_b64_tr_b8 %2, %4 offset:1024\n\tds_read_b64_tr_b8 %3, %4 offset:1536\n\ts_waitcnt lgkmcnt(0)\n\t"
                             "ds_write_b64 %5, %0\n\tds_write_b64 %5, %1 offset:8\n\tds_write_b64 %5, %2 offset:256\n\tds_write_b64 %5, %3 offset:264\n\t"
                             "s_waitcnt lgkmcnt(0)"
                             : "=&v"(g0), "=&v"(g1), "=&v"(g2), "=&v"(g3) : "v"(ra), "v"(wa) : "memory");
            }
            off = X_BYTES + slot * WS_SLAB + (pan * 16u + (unsigned)k * 8u) * 2048u + hh * 1024u + w_dst;
        } else if (e & 0x4000u) {
            off = fwd_off(u) + (unsigned)k * (unsigned)(N * 4) + (unsigned)lane * 16u;
        } else {
            off = slot * XS_SLAB + (pan * 16u + (unsigned)wid + (unsigned)k * 8u) * 2048u + hh * 1024u + (unsigned)lane * 16u;
        }
        u32x4 v;
        asm volatile("ds_read_b128 %0, %1\n\ts_waitcnt lgkmcnt(0)" : "=&v"(v) : "v"(a) : "memory");
        return v;
    }
    __device__ __forceinline__ unsigned flag_off(int u) const {
        const unsigned e = P.order[u >> 1], j = (e >> 8) & 3u, pan = e & 127u, slot = j * 8u + (unsigned)P.rank;
        return ((e & 0x8000u) ? FW_OFF : FX_OFF) + pan * 128u + slot * 4u;
    }
};
#define STORE_WT(off, v, base) asm volatile("s_nop 4\n\tglobal_store_dwordx4 %0, %1, %2 sc0 sc1\n\ts_nop 1" :: "v"(off), "v"(v), "s"(base) : "memory")

__device__ __forceinline__ void pusher(const Params& P, LAS unsigned char* lds, int wid, int lane, int pi) {
    const Mover mv(P, lds, wid, lane);
    const unsigned wbuf = (unsigned)wid * 4096u;
    mv.stage(pi, wbuf);
    __builtin_amdgcn_s_setprio(3);
    unsigned char* const wsl = P.ws[P.rank];
    unsigned call;
    {
        unsigned t = 0;
        if (lane == 0) t = __hip_atomic_fetch_add((unsigned*)(wsl + CTR_OFF) + pi * 16 + wid, 1u, __ATOMIC_RELAXED, __HIP_MEMORY_SCOPE_AGENT);
        call = rfl(t) + 1u;
    }
    unsigned* const tmo = (unsigned*)(wsl + TMO_OFF);
    unsigned char* dst[NR - 1];
    unsigned pbit[NR - 1];
    auto bases = [&](bool to_out) {
#pragma unroll
        for (int q = 0; q < NR - 1; ++q) {
            const unsigned p = (unsigned)(P.rank + 1 + (q + wid + pi) % (NR - 1)) & 7u;
            dst[q] = rfl_ptr(to_out ? P.out[p] : P.ws[p] + DATA_OFF + (size_t)(call & 1u) * PAR_BYTES);
            pbit[q] = 0x10000u << p;
        }
    };
    bases(false);
    const unsigned peer = (unsigned)(P.rank + 1 + lane) & 7u;
    unsigned char* flag_base = P.ws[peer];
    asm volatile("" : "+v"(flag_base));
    unsigned* const tick = (unsigned*)(wsl + TICK_OFF + (call & 1u) * 128u);
    const unsigned tbox = (unsigned)(size_t)lds + 65536u;
    auto signal = [&](int uu, int i) {
        if (uu < NUIN && wid == (i & 7)) {
            if (lane < NR - 1 && ((P.order[uu >> 1] >> 16) >> peer & 1u))
                __hip_atomic_fetch_add((unsigned*)(flag_base + mv.flag_off(uu)), 1u, __ATOMIC_RELAXED, __HIP_MEMORY_SCOPE_SYSTEM);
        }
    };
    bool fwd = false, bad = false;
    int uprev = -1, u = pi, unext = pi + NPUSH;
    unsigned nfwd = 0u;
    WAIT_V(0);
    for (int i = 0; u < NUNIT; ++i) {
        if (unext < NUNIT) {
            if (unext >= NUIN && !fwd) {
                fwd = true;
                bad |= wait_words((const unsigned*)(wsl + TDONE_OFF) + (lane < NOWN ? lane : 0), 8u * call, lane < NOWN, tmo, call, lane);
            }
            mv.stage(unext, wbuf + (unsigned)((i + 1) & 1) * 2048u);
        }
        if (wid == 0 && lane == 0) { const unsigned one = 1u; asm volatile("global_atomic_add v255, %0, %1, off sc0" :: "v"(tick), "v"(one) : "memory", "v255"); }
        if (u >= NUIN) { if (uprev < NUIN) bases(true); ++nfwd; }
        const unsigned e = mv.entry(u);
        const bool full = ((e | (0x10000u << P.rank)) >> 16) == 0xffu;
        int un2 = 0;
        auto meet = [&]() {
            if (wid == 0 && lane == 0) asm volatile("ds_write_b32 %0, v255\n\ts_waitcnt lgkmcnt(0)" :: "v"(tbox + (unsigned)(i & 1) * 4u) : "memory");
            BAR;
            unsigned t;
            asm volatile("ds_read_b32 %0, %1\n\ts_waitcnt lgkmcnt(0)" : "=&v"(t) : "v"(tbox + (unsigned)(i & 1) * 4u) : "memory");
            un2 = (int)rfl(t) + 2 * NPUSH;
            if (uprev >= 0) signal(uprev, i);
        };
        for (int k = 0; k < 2; ++k) {
            unsigned off;
            const u32x4 v = mv.fetch(u, wbuf + (unsigned)(i & 1) * 2048u, k, off);
#pragma unroll
            for (int q = 0; q < NR - 1; ++q) {
                if (e & pbit[q]) {
                    STORE_WT(off, v, dst[q]);
                    asm volatile("s_waitcnt vmcnt(%0)" :: "n"(PUSH_WIN) : "memory");
                }
                if (q == (PUSH_WIN - 1) % (NR - 1) && k == (PUSH_WIN - 1) / (NR - 1) && full) meet();
            }
        }
        if (!full) {
            const int c = __builtin_popcount((e & ~(0x10000u << P.rank)) >> 16);
            if (c >= 3) WAIT_V(6); else if (c == 2) WAIT_V(4); else if (c == 1) WAIT_V(2); else WAIT_V(0);
            meet();
        }
        uprev = u; u = unext; unext = un2;
    }
    WAIT_V(0);
    BAR;
    signal(uprev, 0);
    if (wid == 0) {
        if (lane < NR - 1 && nfwd) __hip_atomic_fetch_add((unsigned*)(flag_base + DONE_OFF + (unsigned)P.rank * 128u), nfwd, __ATOMIC_RELAXED, __HIP_MEMORY_SCOPE_SYSTEM);
        STAMP(1);
        if (pi == 0) {
            bad |= wait_words((const unsigned*)(wsl + DONE_OFF + peer * 128u), (unsigned)(NUNIT - NUIN) * call, lane < NR - 1, tmo, call, lane);
            if (lane < 8) __hip_atomic_store((unsigned*)(wsl + ROLE_OFF + ((call & 1u) ^ 1u) * 1024u + (unsigned)lane * 128u), 0u, __ATOMIC_RELAXED, __HIP_MEMORY_SCOPE_AGENT);
            if (lane == 8) __hip_atomic_store((unsigned*)(wsl + TICK_OFF + ((call & 1u) ^ 1u) * 128u), 0u, __ATOMIC_RELAXED, __HIP_MEMORY_SCOPE_AGENT);
            STAMP(2);
        }
    }
    if (bad && lane == 0) __hip_atomic_store((unsigned*)P.out[P.rank], 0x7fc00000u, __ATOMIC_RELAXED, __HIP_MEMORY_SCOPE_SYSTEM);
}
static_assert(PUSH_WIN >= 1 && PUSH_WIN <= 2 * (NR - 1), "window");

__device__ __forceinline__ void local_copy(const Params& P, const Mover& mv, unsigned call, int wid, int lane, int li) {
    const int nun = li + 2 * NGEMM < NUIN ? 3 : 2;
    const unsigned wbuf = (unsigned)wid * 8192u;
    for (int i = 0; i < nun; ++i) mv.stage(li + NGEMM * i, wbuf + (unsigned)i * 2048u);
    unsigned char* const wsl = P.ws[P.rank];
    unsigned char* const dst = rfl_ptr(wsl + DATA_OFF + (size_t)(call & 1u) * PAR_BYTES);
    WAIT_V(0);
    for (int i = 0; i < nun; ++i)
        for (int k = 0; k < 2; ++k) {
            unsigned off;
            const u32x4 v = mv.fetch(li + NGEMM * i, wbuf + (unsigned)i * 2048u, k, off);
            STORE_WT(off, v, dst);
        }
    WAIT_V(0);
    __syncthreads();
    if (wid < nun && lane == 0) __hip_atomic_fetch_add((unsigned*)(wsl + mv.flag_off(li + NGEMM * wid)), 1u, __ATOMIC_RELAXED, __HIP_MEMORY_SCOPE_SYSTEM);
}

__device__ __forceinline__ void gemm_role(const Params& P, LAS unsigned char* lds, const int wid, const int lane) {
    unsigned char* const wsl = P.ws[P.rank];
    if (wid == 0) {
        const unsigned xcc = (unsigned)__builtin_amdgcn_s_getreg((3 << 11) | 20) & 7u;
        unsigned t = 0;
        if (lane < 3) t = __hip_atomic_fetch_add((unsigned*)(wsl + (lane == 0 ? CTR_OFF + blockIdx.x * 64u : ROLE_OFF + (unsigned)(lane - 1) * 1024u + xcc * 128u)),
                                                 1u, __ATOMIC_RELAXED, __HIP_MEMORY_SCOPE_AGENT);
        const unsigned c = (unsigned)__builtin_amdgcn_readlane((int)t, 0) + 1u;
        unsigned tk = (c & 1u) ? (unsigned)__builtin_amdgcn_readlane((int)t, 2) : (unsigned)__builtin_amdgcn_readlane((int)t, 1), x = xcc;
        for (int k = 1; k < 8 && tk >= 30u; ++k) {
            x = (xcc + (unsigned)k) & 7u;
            unsigned t2 = 0;
            if (lane == 0) t2 = __hip_atomic_fetch_add((unsigned*)(wsl + ROLE_OFF + (c & 1u) * 1024u + x * 128u), 1u, __ATOMIC_RELAXED, __HIP_MEMORY_SCOPE_AGENT);
            tk = rfl(t2);
        }
        if (lane == 0) *(LAS v2i*)(lds + LDS_MBOX) = (v2i){(int)c, (int)(tk < 30u ? x * 30u + tk : 256u)};
    }
    __syncthreads();
    const v2i cr = *(const LAS v2i*)(lds + LDS_MBOX);
    const unsigned call = rfl((unsigned)cr.x);
    const int role = (int)rfl((unsigned)cr.y);
    if (role >= NGEMM) return;
    const int band = role / 60, i60 = role - band * 60, g = i60 >> 2, prow = band * 4 + (i60 & 3);
    { const Mover mv(P, lds, wid, lane); local_copy(P, mv, call, wid, lane, role); }
    STAMP(11);

    u64 tiles = 0; int nseg = 0;
    for (int r = 0; r < 4; ++r) {
        const int sidx = r * 15 + g;
        unsigned t;
        if (sidx == SPSLOT) {
            const int lo = prow & 7, a = (prow >> 3) * 5 + (lo < 3 ? lo : lo - 1);
            if (lo == 3 || lo > 5) continue;
            const unsigned o = (unsigned)(P.rank * NOWN + a);
            t = (o & 15u) | (DPAN + (o >> 4)) << 4 | (unsigned)(a + 1) << 10;
        } else t = (unsigned)prow | (unsigned)(sidx - (sidx > SPSLOT)) << 4;
        tiles |= (u64)t << (16 * nseg); ++nseg;
    }
    auto tile_lane = [&](int sg) { return (unsigned)(tiles >> (16 * (sg & 3))) & 0xffffu; };
    auto tile_of = [&](int sg) { return rfl(tile_lane(sg)); };
    const int wr = wid >> 2, wc = wid & 3;
    const unsigned thr = 2u * call;
    unsigned char* const dbase = rfl_ptr(wsl + DATA_OFF + (size_t)(call & 1u) * PAR_BYTES);
    const __amdgpu_buffer_rsrc_t rsX = __builtin_amdgcn_make_buffer_rsrc((void*)dbase, 0, X_BYTES, 0x00020000);
    const __amdgpu_buffer_rsrc_t rsW = __builtin_amdgcn_make_buffer_rsrc((void*)(dbase + X_BYTES), 0, W_BYTES, 0x00020000);
    const __amdgpu_buffer_rsrc_t rsF = __builtin_amdgcn_make_buffer_rsrc((void*)wsl, 0, DATA_OFF, 0x00020000);
    const unsigned voff = (unsigned)(wid * 2048 + lane * 16);
    const unsigned ldsw = (unsigned)wid * 2048u;
    int aoff = wr * 8192 + lane * 16, boff = LDS_B + wc * 4096 + lane * 16;
    asm volatile("" : "+v"(aoff), "+v"(boff));
    const unsigned snap_src = (lane < 32 ? FW_OFF + ((tile_lane(lane >> 3) >> 4) & 63u) * 128u : FX_OFF + (tile_lane(lane >> 3) & 15u) * 128u) + (unsigned)(lane & 7) * 16u;
    const unsigned snap_rd = lane < 32 ? (unsigned)lane * 4u : 384u + (unsigned)lane * 4u;

#define DMA(rs, ldsoff, vo, so, imm) __builtin_amdgcn_raw_ptr_buffer_load_lds(rs, (LAS void*)(lds + (ldsoff)), 16, vo, so, imm, 17)
#define STAGE_A(b, h, so) do { DMA(rsX, ((b) * 2 + (h)) * HT + ldsw, voff, (so) + (h) * HT, 0); DMA(rsX, ((b) * 2 + (h)) * HT + ldsw, voff, (so) + (h) * HT, 1024); } while (0)
#define STAGE_B(b, h, so) do { DMA(rsW, LDS_B + ((b) * 2 + (h)) * HT + ldsw, voff, (so) + (h) * HT, 0); DMA(rsW, LDS_B + ((b) * 2 + (h)) * HT + ldsw, voff, (so) + (h) * HT, 1024); } while (0)
#define LDF(off) __builtin_shufflevector(*(const LAS v4i*)(lds + (off)), *(const LAS v4i*)(lds + (off) + 1024), 0, 1, 2, 3, 4, 5, 6, 7)
#define LDA(b, h) do { _Pragma("unroll") for (int m = 0; m < 4; ++m) At[m] = LDF(((b) * 2 + (h)) * HT + aoff + m * 2048); } while (0)
#define LDB(dst, b, h) do { _Pragma("unroll") for (int n = 0; n < 2; ++n) dst[n] = LDF(((b) * 2 + (h)) * HT + boff + n * 2048); } while (0)
#define MF(c, bb, aa) asm volatile("v_mfma_f32_16x16x128_f8f6f4 %0, %1, %2, %0 cbsz:1 blgp:1" : "+v"(c) : "v"(aa), "v"(bb))

    f32x4 acc[8][4];
    v8i At[4], B0[2], B1[2];
#pragma unroll
    for (int m = 0; m < 8; ++m)
#pragma unroll
        for (int n = 0; n < 4; ++n) acc[m][n] = (f32x4){0.f, 0.f, 0.f, 0.f};

    unsigned xm = 0u, wm = 0u, used = 0u, bad = 0u;
    int ts = 0;
    unsigned a_pan = (tile_of(0) & 15u) * 32768u, b_pan = ((tile_of(0) >> 4) & 63u) * 32768u;
    auto slow_wait = [&]() {
        if (wid == 0) {
            const unsigned* fp = (const unsigned*)(wsl + (lane < 32 ? FW_OFF + (b_pan >> 8) + (unsigned)lane * 4u : FX_OFF + (a_pan >> 8) + (unsigned)(lane - 32) * 4u));
            unsigned* tmo = (unsigned*)(wsl + TMO_OFF);
            const u64 t0 = __builtin_amdgcn_s_memrealtime();
            u64 mk; unsigned dead = 0u;
            for (;;) {
                const unsigned v = __hip_atomic_load(fp, __ATOMIC_RELAXED, __HIP_MEMORY_SCOPE_SYSTEM);
                mk = __builtin_amdgcn_ballot_w64((int)(v - thr) >= 0);
                if (__builtin_popcount((unsigned)mk & (unsigned)(mk >> 32) & ~used) >= 2) break;
                if (rfl(__hip_atomic_load(tmo, __ATOMIC_RELAXED, __HIP_MEMORY_SCOPE_AGENT)) == call || __builtin_amdgcn_s_memrealtime() - t0 > DEADLINE_TICKS) {
                    if (lane == 0) __hip_atomic_store(tmo, call, __ATOMIC_RELAXED, __HIP_MEMORY_SCOPE_AGENT);
                    mk = ~0ull; dead = 1u; break;
                }
                __builtin_amdgcn_s_sleep(4);
            }
            if (lane == 0) *(LAS u32x4*)(lds + LDS_MBOX + 16) = (u32x4){(unsigned)mk, (unsigned)(mk >> 32), dead, 0u};
            asm volatile("s_waitcnt vmcnt(0) lgkmcnt(0)" ::: "memory");
        }
        asm volatile("" ::: "memory"); BAR; asm volatile("" ::: "memory");
        const u32x4 mb = *(const LAS u32x4*)(lds + LDS_MBOX + 16);
        wm |= rfl(mb.x); xm |= rfl(mb.y); bad |= rfl(mb.z);
    };
    unsigned a2, b2, a3, b3;
    auto take = [&](unsigned av) {
        const unsigned sA = (unsigned)__builtin_ctz(av), sB = (unsigned)__builtin_ctz(av & (av - 1u));
        used |= (1u << sA) | (1u << sB);
        a2 = sA * XS_SLAB + a_pan; b2 = sA * WS_SLAB + b_pan; a3 = sB * XS_SLAB + a_pan; b3 = sB * WS_SLAB + b_pan;
    };

    const float sl = __builtin_bit_cast(float, rfl(__builtin_bit_cast(unsigned, (P.sx[0] * P.sw[0]) * -1.44269504089f)));
    slow_wait(); take(xm & wm);
    STAMP(1);
    unsigned a1 = a3, b1 = b3;
    STAGE_A(0, 0, a2); STAGE_B(0, 0, b2); STAGE_B(0, 1, b2); STAGE_A(0, 1, a2);
    STAGE_A(1, 0, a3); STAGE_B(1, 0, b3);
    WAIT_V(4); BAR;
    if (wr == 1) BAR;
    bool ready;
    { const unsigned av = xm & wm & ~used; ready = (av & (av - 1u)) != 0u; if (ready) take(av); }
#define MMA2(ai) do { __builtin_amdgcn_s_setprio(1); _Pragma("unroll") for (int m = 0; m < 4; ++m) { \
        MF(acc[ai * 4 + m][0], B0[0], At[m]); MF(acc[ai * 4 + m][1], B0[1], At[m]); MF(acc[ai * 4 + m][2], B1[0], At[m]); MF(acc[ai * 4 + m][3], B1[1], At[m]); } \
        __builtin_amdgcn_s_setprio(0); } while (0)
    for (int seg = 0; seg < nseg; ++seg) {
        for (int it = 0; it < 16; ++it) {
            if (!ready) { slow_wait(); take(xm & wm & ~used); }
            const unsigned snap = LDS_SNAP + (unsigned)(it & 1) * 1024u;
            LDB(B0, 0, 0); LDA(0, 0); LDB(B1, 0, 1); STAGE_B(1, 1, b1); STAGE_A(1, 1, a1);
            if (wid == 0) DMA(rsF, snap, snap_src, 0, 0);
            WAIT_L(0); BAR; SCHED; MMA2(0); BAR; SCHED;
            LDA(0, 1); STAGE_A(0, 0, a2); STAGE_B(0, 0, b2);
            WAIT_V(4); WAIT_L(0); BAR; SCHED; MMA2(1); BAR; SCHED;
            LDB(B0, 1, 0); LDA(1, 0); LDB(B1, 1, 1); STAGE_B(0, 1, b2); STAGE_A(0, 1, a2);
            const unsigned snapv = *(const LAS unsigned*)(lds + snap + (unsigned)((it == 14 ? ts + 1 : ts) & 3) * 128u + snap_rd);
            WAIT_L(0); BAR; SCHED; MMA2(0); BAR; SCHED;
            const u64 mk = __builtin_amdgcn_ballot_w64((int)(snapv - thr) >= 0);
            SCHED;
            LDA(1, 1); STAGE_A(1, 0, a3); STAGE_B(1, 0, b3);
            a1 = a3; b1 = b3;
            if (it == 14) {
                ++ts; wm = 0u; xm = 0u; used = 0u;
                const unsigned t = tile_of(ts);
                a_pan = (t & 15u) * 32768u; b_pan = ((t >> 4) & 63u) * 32768u;
            }
            wm |= (unsigned)mk; xm |= (unsigned)(mk >> 32);
            if (ts >= nseg) { wm = xm = ~0u; used = 0u; }
            { const unsigned av = xm & wm & ~used; ready = (av & (av - 1u)) != 0u; if (ready) take(av); }
            WAIT_V(4); WAIT_L(0); BAR; SCHED; MMA2(1); BAR; SCHED;
        }
        STAMP(2 * seg + 2);
        SCHED; asm volatile("s_nop 15\n\ts_nop 15" ::: "memory"); SCHED;
        const unsigned tcur = tile_of(seg);
        {
            int tz = threadIdx.x; asm volatile("" : "+v"(tz));
            const unsigned ov = (unsigned)((tz >> 4) & 3) * (unsigned)(4 * N * 4) + (unsigned)(tz & 15) * 16u;
            const char* ob = (const char*)rfl_ptr(P.out[P.rank] + (size_t)((tcur & 15u) * 256u + (unsigned)wr * 64u) * (N * 4) + (((tcur >> 4) & 63u) * 256u + (unsigned)wc * 64u) * 4u);
            asm volatile("s_nop 4" : "+s"(ob));
            const float cc = bad ? __builtin_nanf("") : -1.44269504089f;
#pragma unroll
            for (int m = 0; m < 8; ++m) {
                const char* rowp = ob + (size_t)((m >> 2) * 128 + (m & 3) * 16) * (N * 4);
#pragma unroll
                for (int j = 0; j < 4; ++j) {
                    const f32x4 o = silu4(acc[m][0][j], acc[m][1][j], acc[m][2][j], acc[m][3][j], sl, cc);
                    asm volatile("global_store_dwordx4 %0, %1, %2" OUT_POL "\n\ts_nop 1" :: "v"(ov), "v"(o), "s"(rowp + (size_t)j * (N * 4)) : "memory");
                }
#pragma unroll
                for (int n = 0; n < 4; ++n) acc[m][n] = (f32x4){0.f, 0.f, 0.f, 0.f};
                if (m == 0 && wr == 0) BAR;
            }
        }
        if (tcur >> 10) {
            WAIT_V(0);
            if (lane == 0) __hip_atomic_fetch_add((unsigned*)(wsl + TDONE_OFF) + ((tcur >> 10) - 1u), 1u, __ATOMIC_RELAXED, __HIP_MEMORY_SCOPE_AGENT);
        }
        STAMP(2 * seg + 3);
        if (wr == 1) BAR;
    }
    WAIT_V(0);
    if (wr == 0) BAR;
    STAMP(10);
}

__global__ __launch_bounds__(NTHR, 2) void fused_kernel(const Params P) {
    extern __shared__ __attribute__((aligned(16))) unsigned char shm[];
    LAS unsigned char* lds = (LAS unsigned char*)shm;
    const int wid = __builtin_amdgcn_readfirstlane(threadIdx.x >> 6), lane = threadIdx.x & 63;
    if (blockIdx.x < NPUSH) pusher(P, lds, wid, lane, blockIdx.x);
    else gemm_role(P, lds, wid, lane);
}

constexpr double g_tune[8] = {1.5, 2.0, 10.0, 8.0, 45.0, 0.0, 0.0, 2.0};
void build_order(int s, unsigned* out) {
    struct It { double key; unsigned code; };
    It it[NCHUNK]; int n = 0;
    const double delta = g_tune[0], tau0 = g_tune[1], lead = g_tune[2], xs = g_tune[3], Tr = g_tune[4], T1 = 32 * tau0 + g_tune[5];
    const int grot = 3, xrot = (int)g_tune[7];
    const double early = g_tune[6];
    for (int j = 0; j < 4; ++j)
        for (int p = 0; p < 16; ++p)
            it[n++] = It{8 * j * tau0 + xs * tau0 * (((p + xrot * s) & 15) / 16.0) - 0.01, 0xff0000u | (unsigned)((j << 8) | p)};
    for (int c = 0; c < 64; ++c) {
        const int sidx = c >= DPAN ? SPSLOT : c + (c >= SPSLOT), r = sidx / 15, g = sidx % 15;
        unsigned mask = 0xffu;
        if (c >= DPAN) { mask = 0; for (int t = (c - DPAN) * 16; t < (c - DPAN) * 16 + 16; ++t) mask |= 1u << (t / NOWN); }
        for (int j = 0; j < 4; ++j) {
            const int pos = 8 * j + (grot ? ((s + g * grot + (g >> 3) * 4) & 7) : 0);
            const double st = g >= 7 ? (g - 7) * delta : g * delta + lead;
            const double key = (r == 0 ? st + pos * tau0 : T1 + (r - 1 - (sidx > SPSLOT && g == SPSLOT % 15 ? early : 0.0)) * Tr + st + pos * (Tr / 34.0)) + 1e-4 * c;
            it[n++] = It{key, mask << 16 | 0x8000u | (unsigned)((j << 8) | c)};
        }
    }
    std::stable_sort(it, it + NCHUNK, [](const It& a, const It& b) { return a.key < b.key; });
    for (int i = 0; i < NCHUNK; ++i) out[i] = it[i].code;
}
}

extern "C" void kernel_launch(int rank, int nranks, ncclComm_t comm,
                              hipStream_t stream,
                              void* const* d_in, const int* in_sizes, int n_in,
                              void* d_out, int out_size,
                              const int* in_dtypes, int out_dtype,
                              void* const* d_out_all,
                              void* const* d_ws_all, size_t ws_bytes) {
    (void)comm; (void)in_dtypes; (void)out_dtype;
    if (nranks != NR || (unsigned)rank >= (unsigned)NR || n_in != 4 || ws_bytes < WS_NEED || out_size != M * N ||
        in_sizes[0] != M * KL || in_sizes[1] != KL * N) {
        fprintf(stderr, "kernel_launch: unexpected problem shape\n");
        return;
    }
    Params P;
    P.x = (const unsigned char*)d_in[0]; P.w = (const unsigned char*)d_in[1];
    P.sx = (const float*)d_in[2]; P.sw = (const float*)d_in[3];
    for (int p = 0; p < NR; ++p) { P.ws[p] = (unsigned char*)d_ws_all[p]; P.out[p] = (unsigned char*)d_out_all[p]; }
    P.out[rank] = (unsigned char*)d_out;
    P.rank = rank; P.pad = 0;
    build_order(rank, P.order);
    HIP_CHECK(hipFuncSetAttribute((const void*)fused_kernel, hipFuncAttributeMaxDynamicSharedMemorySize, LDS_BYTES));
    hipLaunchKernelGGL(fused_kernel, dim3(NWG), dim3(NTHR), LDS_BYTES, stream, P);
    HIP_CHECK(hipGetLastError());
}
```

```cpp
#include <hip/hip_runtime.h>
#include <rccl/rccl.h>
#include <cstdio>
#include <algorithm>

#define HIP_CHECK(expr) do { hipError_t _e = (expr); if (_e != hipSuccess) { \
    fprintf(stderr, "HIP error %s at %s:%d\n", hipGetErrorString(_e), __FILE__, __LINE__); \
    return; } } while (0)

namespace {
#define LAS __attribute__((address_space(3)))
#define GAS __attribute__((address_space(1)))
typedef int v8i __attribute__((ext_vector_type(8)));
typedef int v4i __attribute__((ext_vector_type(4)));
typedef int v2i __attribute__((ext_vector_type(2)));
typedef unsigned u32x4 __attribute__((ext_vector_type(4)));
typedef float f32x4 __attribute__((ext_vector_type(4)));
typedef float f32x2 __attribute__((ext_vector_type(2)));
typedef unsigned long long u64;

constexpr int M = 4096, N = 16384, KL = 512, NR = 8;
constexpr int NPUSH = 16, NWG = 256, NGEMM = NWG - NPUSH, NTHR = 512;
constexpr int NCHUNK = 4 * 16 + 4 * 64;
constexpr int NUIN = 2 * NCHUNK;
constexpr int DPAN = 59, NOWN = 10;
constexpr int NUNIT = NUIN + 9 * 16 + 4;
#define SPG 0
constexpr int SPSLOT = 15 + SPG;
__host__ __device__ constexpr unsigned slot_tile(unsigned q, unsigned i) {
    return i < 8u ? ((q & 1u) * 8u + i) | (60u + (q >> 1)) << 4 : (i == 8u ? q : (i == 9u ? 8u + (q >> 2) : i)) | (unsigned)DPAN << 4;
}
constexpr unsigned XS_SLAB = 256u * 2048u;
constexpr unsigned WS_SLAB = 1024u * 2048u;
constexpr unsigned X_BYTES = 32u * XS_SLAB, W_BYTES = 32u * WS_SLAB, PAR_BYTES = X_BYTES + W_BYTES;
constexpr unsigned CTR_OFF = 0;
constexpr unsigned TMO_OFF = 32768;
constexpr unsigned ROLE_OFF = 36864;
constexpr unsigned TICK_OFF = 40960;
constexpr unsigned TDONE_OFF = 45056;
constexpr unsigned DONE_OFF = 49152;
constexpr unsigned FX_OFF = 65536;
constexpr unsigned FW_OFF = 73728;
constexpr unsigned DBG_OFF = 131072;
constexpr unsigned DATA_OFF = 1u << 20;
constexpr size_t WS_NEED = (size_t)DATA_OFF + 2ull * PAR_BYTES;
constexpr u64 DEADLINE_TICKS = 100000000ull;
constexpr int HT = 16384, LDS_B = 4 * HT, LDS_SNAP = 8 * HT, LDS_MBOX = LDS_SNAP + 2048, LDS_BYTES = LDS_MBOX + 64;
#define OUT_POL " sc0 sc1"
#define PUSH_WIN 7

struct Params {
    const unsigned char* x; const unsigned char* w; const float* sx; const float* sw;
    unsigned char* ws[NR]; unsigned char* out[NR];
    int rank; int pad;
    unsigned order[NCHUNK];
};

__device__ __forceinline__ unsigned rfl(unsigned v) { return (unsigned)__builtin_amdgcn_readfirstlane((int)v); }
__device__ __forceinline__ unsigned char* rfl_ptr(unsigned char* p) {
    return (unsigned char*)(((u64)rfl((unsigned)((u64)p >> 32)) << 32) | (u64)rfl((unsigned)(u64)p));
}
__device__ __forceinline__ float mul1(float a, float b) { float r; asm("v_mul_f32 %0, %1, %2" : "=v"(r) : "s"(b), "v"(a)); return r; }
__device__ __forceinline__ f32x4 silu4(float a0, float a1, float a2, float a3, float sl, float c) {
    const f32x2 t0 = {mul1(a0, sl), mul1(a1, sl)}, t1 = {mul1(a2, sl), mul1(a3, sl)};
    const float L = 86.56170245f;
    f32x2 e0, e1;
    e0.x = __builtin_amdgcn_exp2f(__builtin_amdgcn_fmed3f(t0.x, -L, L)); e0.y = __builtin_amdgcn_exp2f(__builtin_amdgcn_fmed3f(t0.y, -L, L));
    e1.x = __builtin_amdgcn_exp2f(__builtin_amdgcn_fmed3f(t1.x, -L, L)); e1.y = __builtin_amdgcn_exp2f(__builtin_amdgcn_fmed3f(t1.y, -L, L));
    const f32x2 cv = {c, c};
    const f32x2 d0 = __builtin_elementwise_fma(e0, cv, cv), d1 = __builtin_elementwise_fma(e1, cv, cv);
    const f32x2 r0 = {__builtin_amdgcn_rcpf(d0.x), __builtin_amdgcn_rcpf(d0.y)}, r1 = {__builtin_amdgcn_rcpf(d1.x), __builtin_amdgcn_rcpf(d1.y)};
    const f32x2 o0 = t0 * r0, o1 = t1 * r1;
    return (f32x4){o0.x, o0.y, o1.x, o1.y};
}

#define WAIT_V(n) asm volatile("s_waitcnt vmcnt(" #n ")" ::: "memory")
#define WAIT_L(n) asm volatile("s_waitcnt lgkmcnt(" #n ")" ::: "memory")
#define BAR __builtin_amdgcn_s_barrier()
#define SCHED __builtin_amdgcn_sched_barrier(0)
#define STAMP(i)

__device__ __forceinline__ bool wait_words(const unsigned* p, unsigned target, bool active, unsigned* tmo, unsigned call, int lane) {
    const u64 t0 = __builtin_amdgcn_s_memrealtime();
    for (;;) {
        const unsigned v = __hip_atomic_load(p, __ATOMIC_RELAXED, __HIP_MEMORY_SCOPE_SYSTEM);
        if (__builtin_amdgcn_ballot_w64(active && (int)(v - target) < 0) == 0ull) return false;
        if (rfl(__hip_atomic_load(tmo, __ATOMIC_RELAXED, __HIP_MEMORY_SCOPE_AGENT)) == call || __builtin_amdgcn_s_memrealtime() - t0 > DEADLINE_TICKS) {
            if (lane == 0) __hip_atomic_store(tmo, call, __ATOMIC_RELAXED, __HIP_MEMORY_SCOPE_AGENT);
            return true;
        }
        __builtin_amdgcn_s_sleep(4);
    }
}

struct Mover {
    const Params& P; LAS unsigned char* lds; int wid, lane;
    unsigned w_src, x_src, tr_off, sc_off, w_dst;
    __device__ __forceinline__ Mover(const Params& P_, LAS unsigned char* lds_, int wid_, int lane_) : P(P_), lds(lds_), wid(wid_), lane(lane_) {
        w_src = (unsigned)(((wid >> 2) * 32 + (lane >> 2)) * N + (wid & 3) * 64 + (lane & 3) * 16);
        x_src = (unsigned)((wid * 16 + (lane & 15)) * KL + (lane >> 4) * 16);
        tr_off = (unsigned)(((lane & 15) >> 1) * 64 + (lane >> 4) * 16 + (lane & 1) * 8);
        sc_off = (unsigned)((lane & 3) * 512 + (lane >> 2) * 16);
        w_dst = (unsigned)(((wid & 3) * 2 + (lane >> 5)) * 2048 + (wid >> 2) * 512 + (lane & 31) * 16);
    }
    __device__ __forceinline__ unsigned entry(int u) const {
        return u < NUIN ? P.order[u >> 1] : (u < NUIN + 4 ? (P.rank < 4 ? 0xf04000u : 0x0f4000u) : 0xff4000u);
    }
    __device__ __forceinline__ unsigned fwd_off(int u) const {
        const unsigned f = (unsigned)(u - NUIN), f4 = f - 4u;
        const unsigned t = slot_tile((unsigned)P.rank, f < 4u ? 9u : f4 >> 4), blk = f < 4u ? ((unsigned)P.rank & 3u) * 4u + f : f4 & 15u;
        return ((t & 15u) * 256u + blk * 16u + (unsigned)wid * 2u) * (unsigned)(N * 4) + (t >> 4) * 1024u;
    }
    __device__ __forceinline__ void stage(int u, unsigned b) const {
        const unsigned e = entry(u), hh = (unsigned)u & 1u, j = (e >> 8) & 3u, pan = e & 127u;
        const unsigned char* s; unsigned step;
        if (e & 0x8000u) { s = P.w + (size_t)(j * 128u + hh * 64u) * N + pan * 256u + w_src; step = 16u * N; }
        else if (e & 0x4000u) { s = P.out[P.rank] + fwd_off(u) + (unsigned)lane * 16u; step = N * 4u; }
        else { s = P.x + (size_t)(pan * 256u) * KL + j * 128u + hh * 64u + x_src; step = 128u * KL; }
        __builtin_amdgcn_global_load_lds((const GAS void*)s, (LAS void*)(lds + b), 16, 0, 17);
        __builtin_amdgcn_global_load_lds((const GAS void*)(s + step), (LAS void*)(lds + b + 1024), 16, 0, 17);
    }
    __device__ __forceinline__ u32x4 fetch(int u, unsigned b, int k, unsigned& off) const {
        const unsigned e = entry(u), hh = (unsigned)u & 1u, j = (e >> 8) & 3u, pan = e & 127u, slot = j * 8u + (unsigned)P.rank;
        const unsigned lb = (unsigned)(size_t)lds + b;
        const unsigned a = lb + (unsigned)k * 1024u + (unsigned)lane * 16u;
        if (e & 0x8000u) {
            if (k == 0) {
                v2i g0, g1, g2, g3;
                const unsigned ra = lb + tr_off, wa = lb + sc_off;
                asm volatile("ds_read_b64_tr_b8 %0, %4\n\tds_read_b64_tr_b8 %1, %4 offset:512\n\t"
                             "ds_read_b64_tr_b8 %2, %4 offset:1024\n\tds_read_b64_tr_b8 %3, %4 offset:1536\n\ts_waitcnt lgkmcnt(0)\n\t"
                             "ds_write_b64 %5, %0\n\tds_write_b64 %5, %1 offset:8\n\tds_write_b64 %5, %2 offset:256\n\tds_write_b64 %5, %3 offset:264\n\t"
                             "s_waitcnt lgkmcnt(0)"
                             : "=&v"(g0), "=&v"(g1), "=&v"(g2), "=&v"(g3) : "v"(ra), "v"(wa) : "memory");
            }
            off = X_BYTES + slot * WS_SLAB + (pan * 16u + (unsigned)k * 8u) * 2048u + hh * 1024u + w_dst;
        } else if (e & 0x4000u) {
            off = fwd_off(u) + (unsigned)k * (unsigned)(N * 4) + (unsigned)lane * 16u;
        } else {
            off = slot * XS_SLAB + (pan * 16u + (unsigned)wid + (unsigned)k * 8u) * 2048u + hh * 1024u + (unsigned)lane * 16u;
        }
        u32x4 v;
        asm volatile("ds_read_b128 %0, %1\n\ts_waitcnt lgkmcnt(0)" : "=&v"(v) : "v"(a) : "memory");
        return v;
    }
    __device__ __forceinline__ unsigned flag_off(int u) const {
        const unsigned e = P.order[u >> 1], j = (e >> 8) & 3u, pan = e & 127u, slot = j * 8u + (unsigned)P.rank;
        return ((e & 0x8000u) ? FW_OFF : FX_OFF) + pan * 128u + slot * 4u;
    }
};
#define STORE_WT(off, v, base) asm volatile("s_nop 4\n\tglobal_store_dwordx4 %0, %1, %2 sc0 sc1\n\ts_nop 1" :: "v"(off), "v"(v), "s"(base) : "memory")

__device__ __forceinline__ void pusher(const Params& P, LAS unsigned char* lds, int wid, int lane, int pi) {
    const Mover mv(P, lds, wid, lane);
    const unsigned wbuf = (unsigned)wid * 4096u;
    mv.stage(pi, wbuf);
    __builtin_amdgcn_s_setprio(3);
    unsigned char* const wsl = P.ws[P.rank];
    unsigned call;
    {
        unsigned t = 0;
        if (lane == 0) t = __hip_atomic_fetch_add((unsigned*)(wsl + CTR_OFF) + pi * 16 + wid, 1u, __ATOMIC_RELAXED, __HIP_MEMORY_SCOPE_AGENT);
        call = rfl(t) + 1u;
    }
    unsigned* const tmo = (unsigned*)(wsl + TMO_OFF);
    unsigned char* dst[NR - 1];
    unsigned pbit[NR - 1];
    auto bases = [&](bool to_out) {
#pragma unroll
        for (int q = 0; q < NR - 1; ++q) {
            const unsigned p = (unsigned)(P.rank + 1 + (q + wid + pi) % (NR - 1)) & 7u;
            dst[q] = rfl_ptr(to_out ? P.out[p] : P.ws[p] + DATA_OFF + (size_t)(call & 1u) * PAR_BYTES);
            pbit[q] = 0x10000u << p;
        }
    };
    bases(false);
    const unsigned peer = (unsigned)(P.rank + 1 + lane) & 7u;
    unsigned char* flag_base = P.ws[peer];
    asm volatile("" : "+v"(flag_base));
    unsigned* const tick = (unsigned*)(wsl + TICK_OFF + (call & 1u) * 128u);
    const unsigned tbox = (unsigned)(size_t)lds + 65536u;
    auto signal = [&](int uu, int i) {
        if (uu < NUIN && wid == (i & 7)) {
            if (lane < NR - 1 && ((P.order[uu >> 1] >> 16) >> peer & 1u))
                __hip_atomic_fetch_add((unsigned*)(flag_base + mv.flag_off(uu)), 1u, __ATOMIC_RELAXED, __HIP_MEMORY_SCOPE_SYSTEM);
        }
    };
    bool fwd = false, bad = false;
    int uprev = -1, u = pi, unext = pi + NPUSH;
    unsigned nfwd = 0u;
    WAIT_V(0);
    for (int i = 0; u < NUNIT; ++i) {
        if (unext < NUNIT) {
            if (unext >= NUIN && !fwd) {
                fwd = true;
                bad |= wait_words((const unsigned*)(wsl + TDONE_OFF) + (lane < NOWN ? lane : 0), 8u * call, lane < NOWN, tmo, call, lane);
            }
            mv.stage(unext, wbuf + (unsigned)((i + 1) & 1) * 2048u);
        }
        if (wid == 0 && lane == 0) { const unsigned one = 1u; asm volatile("global_atomic_add v255, %0, %1, off sc0" :: "v"(tick), "v"(one) : "memory", "v255"); }
        if (u >= NUIN) { if (uprev < NUIN) bases(true); ++nfwd; }
        const unsigned e = mv.entry(u);
        const bool full = ((e | (0x10000u << P.rank)) >> 16) == 0xffu;
        int un2 = 0;
        auto meet = [&]() {
            if (wid == 0 && lane == 0) asm volatile("ds_write_b32 %0, v255\n\ts_waitcnt lgkmcnt(0)" :: "v"(tbox + (unsigned)(i & 1) * 4u) : "memory");
            BAR;
            unsigned t;
            asm volatile("ds_read_b32 %0, %1\n\ts_waitcnt lgkmcnt(0)" : "=&v"(t) : "v"(tbox + (unsigned)(i & 1) * 4u) : "memory");
            un2 = (int)rfl(t) + 2 * NPUSH;
            if (uprev >= 0) signal(uprev, i);
        };
        for (int k = 0; k < 2; ++k) {
            unsigned off;
            const u32x4 v = mv.fetch(u, wbuf + (unsigned)(i & 1) * 2048u, k, off);
#pragma unroll
            for (int q = 0; q < NR - 1; ++q) {
                if (e & pbit[q]) {
                    STORE_WT(off, v, dst[q]);
                    asm volatile("s_waitcnt vmcnt(%0)" :: "n"(PUSH_WIN) : "memory");
                }
                if (q == (PUSH_WIN - 1) % (NR - 1) && k == (PUSH_WIN - 1) / (NR - 1) && full) meet();
            }
        }
        if (!full) {
            const int c = __builtin_popcount((e & ~(0x10000u << P.rank)) >> 16);
            if (c >= 4) WAIT_V(8); else if (c >= 2) WAIT_V(4); else if (c == 1) WAIT_V(2); else WAIT_V(0);
            meet();
        }
        uprev = u; u = unext; unext = un2;
    }
    WAIT_V(0);
    BAR;
    signal(uprev, 0);
    if (wid == 0) {
        if (lane < NR - 1 && nfwd) __hip_atomic_fetch_add((unsigned*)(flag_base + DONE_OFF + (unsigned)P.rank * 128u), nfwd, __ATOMIC_RELAXED, __HIP_MEMORY_SCOPE_SYSTEM);
        STAMP(1);
        if (pi == 0) {
            bad |= wait_words((const unsigned*)(wsl + DONE_OFF + peer * 128u), (unsigned)(NUNIT - NUIN) * call, lane < NR - 1, tmo, call, lane);
            if (lane < 8) __hip_atomic_store((unsigned*)(wsl + ROLE_OFF + ((call & 1u) ^ 1u) * 1024u + (unsigned)lane * 128u), 0u, __ATOMIC_RELAXED, __HIP_MEMORY_SCOPE_AGENT);
            if (lane == 8) __hip_atomic_store((unsigned*)(wsl + TICK_OFF + ((call & 1u) ^ 1u) * 128u), 0u, __ATOMIC_RELAXED, __HIP_MEMORY_SCOPE_AGENT);
            STAMP(2);
        }
    }
    if (bad && lane == 0) __hip_atomic_store((unsigned*)P.out[P.rank], 0x7fc00000u, __ATOMIC_RELAXED, __HIP_MEMORY_SCOPE_SYSTEM);
}
static_assert(PUSH_WIN >= 1 && PUSH_WIN <= 2 * (NR - 1), "window");

__device__ __forceinline__ void local_copy(const Params& P, const Mover& mv, unsigned call, int wid, int lane, int li) {
    const int nun = li + 2 * NGEMM < NUIN ? 3 : 2;
    const unsigned wbuf = (unsigned)wid * 8192u;
    for (int i = 0; i < nun; ++i) mv.stage(li + NGEMM * i, wbuf + (unsigned)i * 2048u);
    unsigned char* const wsl = P.ws[P.rank];
    unsigned char* const dst = rfl_ptr(wsl + DATA_OFF + (size_t)(call & 1u) * PAR_BYTES);
    WAIT_V(0);
    for (int i = 0; i < nun; ++i)
        for (int k = 0; k < 2; ++k) {
            unsigned off;
            const u32x4 v = mv.fetch(li + NGEMM * i, wbuf + (unsigned)i * 2048u, k, off);
            STORE_WT(off, v, dst);
        }
    WAIT_V(0);
    __syncthreads();
    if (wid < nun && lane == 0) __hip_atomic_fetch_add((unsigned*)(wsl + mv.flag_off(li + NGEMM * wid)), 1u, __ATOMIC_RELAXED, __HIP_MEMORY_SCOPE_SYSTEM);
}

__device__ __forceinline__ void gemm_role(const Params& P, LAS unsigned char* lds, const int wid, const int lane) {
    unsigned char* const wsl = P.ws[P.rank];
    if (wid == 0) {
        const unsigned xcc = (unsigned)__builtin_amdgcn_s_getreg((3 << 11) | 20) & 7u;
        unsigned t = 0;
        if (lane < 3) t = __hip_atomic_fetch_add((unsigned*)(wsl + (lane == 0 ? CTR_OFF + blockIdx.x * 64u : ROLE_OFF + (unsigned)(lane - 1) * 1024u + xcc * 128u)),
                                                 1u, __ATOMIC_RELAXED, __HIP_MEMORY_SCOPE_AGENT);
        const unsigned c = (unsigned)__builtin_amdgcn_readlane((int)t, 0) + 1u;
        unsigned tk = (c & 1u) ? (unsigned)__builtin_amdgcn_readlane((int)t, 2) : (unsigned)__builtin_amdgcn_readlane((int)t, 1), x = xcc;
        for (int k = 1; k < 8 && tk >= 30u; ++k) {
            x = (xcc + (unsigned)k) & 7u;
            unsigned t2 = 0;
            if (lane == 0) t2 = __hip_atomic_fetch_add((unsigned*)(wsl + ROLE_OFF + (c & 1u) * 1024u + x * 128u), 1u, __ATOMIC_RELAXED, __HIP_MEMORY_SCOPE_AGENT);
            tk = rfl(t2);
        }
        if (lane == 0) *(LAS v2i*)(lds + LDS_MBOX) = (v2i){(int)c, (int)(tk < 30u ? x * 30u + tk : 256u)};
    }
    __syncthreads();
    const v2i cr = *(const LAS v2i*)(lds + LDS_MBOX);
    const unsigned call = rfl((unsigned)cr.x);
    const int role = (int)rfl((unsigned)cr.y);
    if (role >= NGEMM) return;
    const int band = role / 60, i60 = role - band * 60, g = i60 >> 2, prow = band * 4 + (i60 & 3);
    { const Mover mv(P, lds, wid, lane); local_copy(P, mv, call, wid, lane, role); }
    STAMP(11);

    u64 tiles = 0;
    constexpr int nseg = 4;
    for (int r = 0; r < 4; ++r) {
        const int sidx = r * 15 + g;
        const unsigned t = sidx == SPSLOT ? slot_tile((unsigned)P.rank, (unsigned)prow) | (prow < NOWN ? (unsigned)(prow + 1) << 10 : 0u)
                                          : (unsigned)prow | (unsigned)(sidx - (sidx > SPSLOT)) << 4;
        tiles |= (u64)t << (16 * r);
    }
    auto tile_lane = [&](int sg) { return (unsigned)(tiles >> (16 * (sg & 3))) & 0xffffu; };
    auto tile_of = [&](int sg) { return rfl(tile_lane(sg)); };
    const int wr = wid >> 2, wc = wid & 3;
    const unsigned thr = 2u * call;
    unsigned char* const dbase = rfl_ptr(wsl + DATA_OFF + (size_t)(call & 1u) * PAR_BYTES);
    const __amdgpu_buffer_rsrc_t rsX = __builtin_amdgcn_make_buffer_rsrc((void*)dbase, 0, X_BYTES, 0x00020000);
    const __amdgpu_buffer_rsrc_t rsW = __builtin_amdgcn_make_buffer_rsrc((void*)(dbase + X_BYTES), 0, W_BYTES, 0x00020000);
    const __amdgpu_buffer_rsrc_t rsF = __builtin_amdgcn_make_buffer_rsrc((void*)wsl, 0, DATA_OFF, 0x00020000);
    const unsigned voff = (unsigned)(wid * 2048 + lane * 16);
    const unsigned ldsw = (unsigned)wid * 2048u;
    int aoff = wr * 8192 + lane * 16, boff = LDS_B + wc * 4096 + lane * 16;
    asm volatile("" : "+v"(aoff), "+v"(boff));
    const unsigned snap_src = (lane < 32 ? FW_OFF + ((tile_lane(lane >> 3) >> 4) & 63u) * 128u : FX_OFF + (tile_lane(lane >> 3) & 15u) * 128u) + (unsigned)(lane & 7) * 16u;
    const unsigned snap_rd = lane < 32 ? (unsigned)lane * 4u : 384u + (unsigned)lane * 4u;

#define DMA(rs, ldsoff, vo, so, imm) __builtin_amdgcn_raw_ptr_buffer_load_lds(rs, (LAS void*)(lds + (ldsoff)), 16, vo, so, imm, 17)
#define STAGE_A(b, h, so) do { DMA(rsX, ((b) * 2 + (h)) * HT + ldsw, voff, (so) + (h) * HT, 0); DMA(rsX, ((b) * 2 + (h)) * HT + ldsw, voff, (so) + (h) * HT, 1024); } while (0)
#define STAGE_B(b, h, so) do { DMA(rsW, LDS_B + ((b) * 2 + (h)) * HT + ldsw, voff, (so) + (h) * HT, 0); DMA(rsW, LDS_B + ((b) * 2 + (h)) * HT + ldsw, voff, (so) + (h) * HT, 1024); } while (0)
#define LDF(off) __builtin_shufflevector(*(const LAS v4i*)(lds + (off)), *(const LAS v4i*)(lds + (off) + 1024), 0, 1, 2, 3, 4, 5, 6, 7)
#define LDA(b, h) do { _Pragma("unroll") for (int m = 0; m < 4; ++m) At[m] = LDF(((b) * 2 + (h)) * HT + aoff + m * 2048); } while (0)
#define LDB(dst, b, h) do { _Pragma("unroll") for (int n = 0; n < 2; ++n) dst[n] = LDF(((b) * 2 + (h)) * HT + boff + n * 2048); } while (0)
#define MF(c, bb, aa) asm volatile("v_mfma_f32_16x16x128_f8f6f4 %0, %1, %2, %0 cbsz:1 blgp:1" : "+v"(c) : "v"(aa), "v"(bb))

    f32x4 acc[8][4];
    v8i At[4], B0[2], B1[2];
#pragma unroll
    for (int m = 0; m < 8; ++m)
#pragma unroll
        for (int n = 0; n < 4; ++n) acc[m][n] = (f32x4){0.f, 0.f, 0.f, 0.f};

    unsigned xm = 0u, wm = 0u, used = 0u, bad = 0u;
    int ts = 0;
    unsigned a_pan = (tile_of(0) & 15u) * 32768u, b_pan = ((tile_of(0) >> 4) & 63u) * 32768u;
    auto slow_wait = [&]() {
        if (wid == 0) {
            const unsigned* fp = (const unsigned*)(wsl + (lane < 32 ? FW_OFF + (b_pan >> 8) + (unsigned)lane * 4u : FX_OFF + (a_pan >> 8) + (unsigned)(lane - 32) * 4u));
            unsigned* tmo = (unsigned*)(wsl + TMO_OFF);
            const u64 t0 = __builtin_amdgcn_s_memrealtime();
            u64 mk; unsigned dead = 0u;
            for (;;) {
                const unsigned v = __hip_atomic_load(fp, __ATOMIC_RELAXED, __HIP_MEMORY_SCOPE_SYSTEM);
                mk = __builtin_amdgcn_ballot_w64((int)(v - thr) >= 0);
                if (__builtin_popcount((unsigned)mk & (unsigned)(mk >> 32) & ~used) >= 2) break;
                if (rfl(__hip_atomic_load(tmo, __ATOMIC_RELAXED, __HIP_MEMORY_SCOPE_AGENT)) == call || __builtin_amdgcn_s_memrealtime() - t0 > DEADLINE_TICKS) {
                    if (lane == 0) __hip_atomic_store(tmo, call, __ATOMIC_RELAXED, __HIP_MEMORY_SCOPE_AGENT);
                    mk = ~0ull; dead = 1u; break;
                }
                __builtin_amdgcn_s_sleep(4);
            }
            if (lane == 0) *(LAS u32x4*)(lds + LDS_MBOX + 16) = (u32x4){(unsigned)mk, (unsigned)(mk >> 32), dead, 0u};
            asm volatile("s_waitcnt vmcnt(0) lgkmcnt(0)" ::: "memory");
        }
        asm volatile("" ::: "memory"); BAR; asm volatile("" ::: "memory");
        const u32x4 mb = *(const LAS u32x4*)(lds + LDS_MBOX + 16);
        wm |= rfl(mb.x); xm |= rfl(mb.y); bad |= rfl(mb.z);
    };
    unsigned a2, b2, a3, b3;
    auto take = [&](unsigned av) {
        const unsigned sA = (unsigned)__builtin_ctz(av), sB = (unsigned)__builtin_ctz(av & (av - 1u));
        used |= (1u << sA) | (1u << sB);
        a2 = sA * XS_SLAB + a_pan; b2 = sA * WS_SLAB + b_pan; a3 = sB * XS_SLAB + a_pan; b3 = sB * WS_SLAB + b_pan;
    };

    const float sl = __builtin_bit_cast(float, rfl(__builtin_bit_cast(unsigned, (P.sx[0] * P.sw[0]) * -1.44269504089f)));
    slow_wait(); take(xm & wm);
    STAMP(1);
    unsigned a1 = a3, b1 = b3;
    STAGE_A(0, 0, a2); STAGE_B(0, 0, b2); STAGE_B(0, 1, b2); STAGE_A(0, 1, a2);
    STAGE_A(1, 0, a3); STAGE_B(1, 0, b3);
    WAIT_V(4); BAR;
    if (wr == 1) BAR;
    bool ready;
    { const unsigned av = xm & wm & ~used; ready = (av & (av - 1u)) != 0u; if (ready) take(av); }
#define MMA2(ai) do { __builtin_amdgcn_s_setprio(1); _Pragma("unroll") for (int m = 0; m < 4; ++m) { \
        MF(acc[ai * 4 + m][0], B0[0], At[m]); MF(acc[ai * 4 + m][1], B0[1], At[m]); MF(acc[ai * 4 + m][2], B1[0], At[m]); MF(acc[ai * 4 + m][3], B1[1], At[m]); } \
        __builtin_amdgcn_s_setprio(0); } while (0)
    for (int seg = 0; seg < nseg; ++seg) {
        for (int it = 0; it < 16; ++it) {
            if (!ready) { slow_wait(); take(xm & wm & ~used); }
            const unsigned snap = LDS_SNAP + (unsigned)(it & 1) * 1024u;
            LDB(B0, 0, 0); LDA(0, 0); LDB(B1, 0, 1); STAGE_B(1, 1, b1); STAGE_A(1, 1, a1);
            if (wid == 0) DMA(rsF, snap, snap_src, 0, 0);
            WAIT_L(0); BAR; SCHED; MMA2(0); BAR; SCHED;
            LDA(0, 1); STAGE_A(0, 0, a2); STAGE_B(0, 0, b2);
            WAIT_V(4); WAIT_L(0); BAR; SCHED; MMA2(1); BAR; SCHED;
            LDB(B0, 1, 0); LDA(1, 0); LDB(B1, 1, 1); STAGE_B(0, 1, b2); STAGE_A(0, 1, a2);
            const unsigned snapv = *(const LAS unsigned*)(lds + snap + (unsigned)((it == 14 ? ts + 1 : ts) & 3) * 128u + snap_rd);
            WAIT_L(0); BAR; SCHED; MMA2(0); BAR; SCHED;
            const u64 mk = __builtin_amdgcn_ballot_w64((int)(snapv - thr) >= 0);
            SCHED;
            LDA(1, 1); STAGE_A(1, 0, a3); STAGE_B(1, 0, b3);
            a1 = a3; b1 = b3;
            if (it == 14) {
                ++ts; wm = 0u; xm = 0u; used = 0u;
                const unsigned t = tile_of(ts);
                a_pan = (t & 15u) * 32768u; b_pan = ((t >> 4) & 63u) * 32768u;
            }
            wm |= (unsigned)mk; xm |= (unsigned)(mk >> 32);
            if (ts >= nseg) { wm = xm = ~0u; used = 0u; }
            { const unsigned av = xm & wm & ~used; ready = (av & (av - 1u)) != 0u; if (ready) take(av); }
            WAIT_V(4); WAIT_L(0); BAR; SCHED; MMA2(1); BAR; SCHED;
        }
        STAMP(2 * seg + 2);
        SCHED; asm volatile("s_nop 15\n\ts_nop 15" ::: "memory"); SCHED;
        const unsigned tcur = tile_of(seg);
        {
            int tz = threadIdx.x; asm volatile("" : "+v"(tz));
            const unsigned ov = (unsigned)((tz >> 4) & 3) * (unsigned)(4 * N * 4) + (unsigned)(tz & 15) * 16u;
            const char* ob = (const char*)rfl_ptr(P.out[P.rank] + (size_t)((tcur & 15u) * 256u + (unsigned)wr * 64u) * (N * 4) + (((tcur >> 4) & 63u) * 256u + (unsigned)wc * 64u) * 4u);
            asm volatile("s_nop 4" : "+s"(ob));
            const float cc = bad ? __builtin_nanf("") : -1.44269504089f;
#pragma unroll
            for (int m = 0; m < 8; ++m) {
                const char* rowp = ob + (size_t)((m >> 2) * 128 + (m & 3) * 16) * (N * 4);
#pragma unroll
                for (int j = 0; j < 4; ++j) {
                    const f32x4 o = silu4(acc[m][0][j], acc[m][1][j], acc[m][2][j], acc[m][3][j], sl, cc);
                    asm volatile("global_store_dwordx4 %0, %1, %2" OUT_POL "\n\ts_nop 1" :: "v"(ov), "v"(o), "s"(rowp + (size_t)j * (N * 4)) : "memory");
                }
#pragma unroll
                for (int n = 0; n < 4; ++n) acc[m][n] = (f32x4){0.f, 0.f, 0.f, 0.f};
                if (m == 0 && wr == 0) BAR;
            }
        }
        if (tcur >> 10) {
            WAIT_V(0);
            if (lane == 0) __hip_atomic_fetch_add((unsigned*)(wsl + TDONE_OFF) + ((tcur >> 10) - 1u), 1u, __ATOMIC_RELAXED, __HIP_MEMORY_SCOPE_AGENT);
        }
        STAMP(2 * seg + 3);
        if (wr == 1) BAR;
    }
    WAIT_V(0);
    if (wr == 0) BAR;
    STAMP(10);
}

__global__ __launch_bounds__(NTHR, 2) void fused_kernel(const Params P) {
    extern __shared__ __attribute__((aligned(16))) unsigned char shm[];
    LAS unsigned char* lds = (LAS unsigned char*)shm;
    const int wid = __builtin_amdgcn_readfirstlane(threadIdx.x >> 6), lane = threadIdx.x & 63;
    if (blockIdx.x < NPUSH) pusher(P, lds, wid, lane, blockIdx.x);
    else gemm_role(P, lds, wid, lane);
}

constexpr double g_tune[8] = {0.0, 2.0, 6.0, 11.0, 45.0, 0.0, 0.0, -1.0};
void build_order(int s, unsigned* out) {
    struct It { double key; unsigned code; };
    It it[NCHUNK]; int n = 0;
    const double delta = g_tune[0], tau0 = g_tune[1], lead = g_tune[2], xs = g_tune[3], Tr = g_tune[4], T1 = 32 * tau0 + g_tune[5];
    const int grot = 3, xrot = (int)g_tune[7];
    for (int j = 0; j < 4; ++j)
        for (int p = 0; p < 16; ++p)
            it[n++] = It{8 * j * tau0 + xs * tau0 * ((xrot < 0 ? (p & 3) * 4 + (((p >> 2) + s) & 3) : (p + xrot * s) & 15) / 16.0) - 0.01, 0xff0000u | (unsigned)((j << 8) | p)};
    for (int c = 0; c < 64; ++c) {
        const int sidx = c >= DPAN ? SPSLOT : c + (c >= SPSLOT), r = sidx / 15, g = sidx % 15;
        unsigned mask = 0xffu;
        if (c > DPAN) mask = 3u << (2 * (c - DPAN - 1));
        for (int j = 0; j < 4; ++j) {
            const int pos = 8 * j + (grot ? ((s + g * grot + (g >> 3) * 4) & 7) : 0);
            const double st = g >= 7 ? (g - 7) * delta : g * delta + lead;
            const double key = (r == 0 ? st + pos * tau0 : T1 + (r - 1) * Tr + st + pos * (Tr / 34.0)) + 1e-4 * c;
            it[n++] = It{key, mask << 16 | 0x8000u | (unsigned)((j << 8) | c)};
        }
    }
    std::stable_sort(it, it + NCHUNK, [](const It& a, const It& b) { return a.key < b.key; });
    for (int i = 0; i < NCHUNK; ++i) out[i] = it[i].code;
}
}

extern "C" void kernel_launch(int rank, int nranks, ncclComm_t comm,
                              hipStream_t stream,
                              void* const* d_in, const int* in_sizes, int n_in,
                              void* d_out, int out_size,
                              const int* in_dtypes, int out_dtype,
                              void* const* d_out_all,
                              void* const* d_ws_all, size_t ws_bytes) {
    (void)comm; (void)in_dtypes; (void)out_dtype;
    if (nranks != NR || (unsigned)rank >= (unsigned)NR || n_in != 4 || ws_bytes < WS_NEED || out_size != M * N ||
        in_sizes[0] != M * KL || in_sizes[1] != KL * N) {
        fprintf(stderr, "kernel_launch: unexpected problem shape\n");
        return;
    }
    Params P;
    P.x = (const unsigned char*)d_in[0]; P.w = (const unsigned char*)d_in[1];
    P.sx = (const float*)d_in[2]; P.sw = (const float*)d_in[3];
    for (int p = 0; p < NR; ++p) { P.ws[p] = (unsigned char*)d_ws_all[p]; P.out[p] = (unsigned char*)d_out_all[p]; }
    P.out[rank] = (unsigned char*)d_out;
    P.rank = rank; P.pad = 0;
    build_order(rank, P.order);
    HIP_CHECK(hipFuncSetAttribute((const void*)fused_kernel, hipFuncAttributeMaxDynamicSharedMemorySize, LDS_BYTES));
    hipLaunchKernelGGL(fused_kernel, dim3(NWG), dim3(NTHR), LDS_BYTES, stream, P);
    HIP_CHECK(hipGetLastError());
}
```

```cpp
#include <hip/hip_runtime.h>
#include <rccl/rccl.h>
#include <cstdio>
#include <algorithm>

#define HIP_CHECK(expr) do { hipError_t _e = (expr); if (_e != hipSuccess) { \
    fprintf(stderr, "HIP error %s at %s:%d\n", hipGetErrorString(_e), __FILE__, __LINE__); \
    return; } } while (0)

namespace {
#define LAS __attribute__((address_space(3)))
#define GAS __attribute__((address_space(1)))
typedef int v8i __attribute__((ext_vector_type(8)));
typedef int v4i __attribute__((ext_vector_type(4)));
typedef int v2i __attribute__((ext_vector_type(2)));
typedef unsigned u32x4 __attribute__((ext_vector_type(4)));
typedef float f32x4 __attribute__((ext_vector_type(4)));
typedef float f32x2 __attribute__((ext_vector_type(2)));
typedef unsigned long long u64;

constexpr int M = 4096, N = 16384, KL = 512, NR = 8;
constexpr int NPUSH = 16, NWG = 256, NGEMM = NWG - NPUSH, NTHR = 512;
constexpr int NCHUNK = 4 * 16 + 4 * 64;
constexpr int NUIN = 2 * NCHUNK;
constexpr int DPAN = 59, NOWN = 10;
constexpr int NUNIT = NUIN + 9 * 16 + 4;
constexpr int SPSLOT = 15;
__host__ __device__ constexpr unsigned slot_tile(unsigned q, unsigned i) {
    return i < 8u ? ((q & 1u) * 8u + i) | (60u + (q >> 1)) << 4 : (i == 8u ? q : (i == 9u ? 8u + (q >> 2) : i)) | (unsigned)DPAN << 4;
}
constexpr unsigned XS_SLAB = 256u * 2048u;
constexpr unsigned WS_SLAB = 1024u * 2048u;
constexpr unsigned X_BYTES = 32u * XS_SLAB, W_BYTES = 32u * WS_SLAB, PAR_BYTES = X_BYTES + W_BYTES;
constexpr unsigned CTR_OFF = 0;
constexpr unsigned TMO_OFF = 32768;
constexpr unsigned ROLE_OFF = 36864;
constexpr unsigned TICK_OFF = 40960;
constexpr unsigned TDONE_OFF = 45056;
constexpr unsigned DONE_OFF = 49152;
constexpr unsigned FX_OFF = 65536;
constexpr unsigned FW_OFF = 73728;
constexpr unsigned DATA_OFF = 1u << 20;
constexpr size_t WS_NEED = (size_t)DATA_OFF + 2ull * PAR_BYTES;
constexpr u64 DEADLINE_TICKS = 100000000ull;
constexpr int HT = 16384, LDS_B = 4 * HT, LDS_SNAP = 8 * HT, LDS_MBOX = LDS_SNAP + 2048, LDS_BYTES = LDS_MBOX + 64;
#define OUT_POL " sc0 sc1"
#define PUSH_WIN 10

struct Params {
    const unsigned char* x; const unsigned char* w; const float* sx; const float* sw;
    unsigned char* ws[NR]; unsigned char* out[NR];
    int rank; int pad;
    unsigned order[NCHUNK];
};

__device__ __forceinline__ unsigned rfl(unsigned v) { return (unsigned)__builtin_amdgcn_readfirstlane((int)v); }
__device__ __forceinline__ unsigned char* rfl_ptr(unsigned char* p) {
    return (unsigned char*)(((u64)rfl((unsigned)((u64)p >> 32)) << 32) | (u64)rfl((unsigned)(u64)p));
}
__device__ __forceinline__ float mul1(float a, float b) { float r; asm("v_mul_f32 %0, %1, %2" : "=v"(r) : "s"(b), "v"(a)); return r; }
__device__ __forceinline__ f32x4 silu4(float a0, float a1, float a2, float a3, float sl, float c) {
    const f32x2 t0 = {mul1(a0, sl), mul1(a1, sl)}, t1 = {mul1(a2, sl), mul1(a3, sl)};
    const float L = 86.56170245f;
    f32x2 e0, e1;
    e0.x = __builtin_amdgcn_exp2f(__builtin_amdgcn_fmed3f(t0.x, -L, L)); e0.y = __builtin_amdgcn_exp2f(__builtin_amdgcn_fmed3f(t0.y, -L, L));
    e1.x = __builtin_amdgcn_exp2f(__builtin_amdgcn_fmed3f(t1.x, -L, L)); e1.y = __builtin_amdgcn_exp2f(__builtin_amdgcn_fmed3f(t1.y, -L, L));
    const f32x2 cv = {c, c};
    const f32x2 d0 = __builtin_elementwise_fma(e0, cv, cv), d1 = __builtin_elementwise_fma(e1, cv, cv);
    const f32x2 r0 = {__builtin_amdgcn_rcpf(d0.x), __builtin_amdgcn_rcpf(d0.y)}, r1 = {__builtin_amdgcn_rcpf(d1.x), __builtin_amdgcn_rcpf(d1.y)};
    const f32x2 o0 = t0 * r0, o1 = t1 * r1;
    return (f32x4){o0.x, o0.y, o1.x, o1.y};
}

#define WAIT_V(n) asm volatile("s_waitcnt vmcnt(" #n ")" ::: "memory")
#define WAIT_L(n) asm volatile("s_waitcnt lgkmcnt(" #n ")" ::: "memory")
#define BAR __builtin_amdgcn_s_barrier()
#define SCHED __builtin_amdgcn_sched_barrier(0)

__device__ __forceinline__ bool wait_words(const unsigned* p, unsigned target, bool active, unsigned* tmo, unsigned call, int lane) {
    const u64 t0 = __builtin_amdgcn_s_memrealtime();
    for (;;) {
        const unsigned v = __hip_atomic_load(p, __ATOMIC_RELAXED, __HIP_MEMORY_SCOPE_SYSTEM);
        if (__builtin_amdgcn_ballot_w64(active && (int)(v - target) < 0) == 0ull) return false;
        if (rfl(__hip_atomic_load(tmo, __ATOMIC_RELAXED, __HIP_MEMORY_SCOPE_AGENT)) == call || __builtin_amdgcn_s_memrealtime() - t0 > DEADLINE_TICKS) {
            if (lane == 0) __hip_atomic_store(tmo, call, __ATOMIC_RELAXED, __HIP_MEMORY_SCOPE_AGENT);
            return true;
        }
        __builtin_amdgcn_s_sleep(4);
    }
}

struct Mover {
    const Params& P; LAS unsigned char* lds; int wid, lane;
    unsigned w_src, x_src, tr_off, sc_off, w_dst;
    __device__ __forceinline__ Mover(const Params& P_, LAS unsigned char* lds_, int wid_, int lane_) : P(P_), lds(lds_), wid(wid_), lane(lane_) {
        w_src = (unsigned)(((wid >> 2) * 32 + (lane >> 2)) * N + (wid & 3) * 64 + (lane & 3) * 16);
        x_src = (unsigned)((wid * 16 + (lane & 15)) * KL + (lane >> 4) * 16);
        tr_off = (unsigned)(((lane & 15) >> 1) * 64 + (lane >> 4) * 16 + (lane & 1) * 8);
        sc_off = (unsigned)((lane & 3) * 512 + (lane >> 2) * 16);
        w_dst = (unsigned)(((wid & 3) * 2 + (lane >> 5)) * 2048 + (wid >> 2) * 512 + (lane & 31) * 16);
    }
    __device__ __forceinline__ unsigned entry(int u) const {
        return u < NUIN ? P.order[u >> 1] : (u < NUIN + 4 ? (P.rank < 4 ? 0xf04000u : 0x0f4000u) : 0xff4000u);
    }
    __device__ __forceinline__ unsigned fwd_off(int u) const {
        const unsigned f = (unsigned)(u - NUIN), f4 = f - 4u;
        const unsigned t = slot_tile((unsigned)P.rank, f < 4u ? 9u : f4 >> 4), blk = f < 4u ? ((unsigned)P.rank & 3u) * 4u + f : f4 & 15u;
        return ((t & 15u) * 256u + blk * 16u + (unsigned)wid * 2u) * (unsigned)(N * 4) + (t >> 4) * 1024u;
    }
    __device__ __forceinline__ void stage(int u, unsigned b) const {
        const unsigned e = entry(u), hh = (unsigned)u & 1u, j = (e >> 8) & 3u, pan = e & 127u;
        const unsigned char* s; unsigned step;
        if (e & 0x8000u) { s = P.w + (size_t)(j * 128u + hh * 64u) * N + pan * 256u + w_src; step = 16u * N; }
        else if (e & 0x4000u) { s = P.out[P.rank] + fwd_off(u) + (unsigned)lane * 16u; step = N * 4u; }
        else { s = P.x + (size_t)(pan * 256u) * KL + j * 128u + hh * 64u + x_src; step = 128u * KL; }
        __builtin_amdgcn_global_load_lds((const GAS void*)s, (LAS void*)(lds + b), 16, 0, 17);
        __builtin_amdgcn_global_load_lds((const GAS void*)(s + step), (LAS void*)(lds + b + 1024), 16, 0, 17);
    }
    __device__ __forceinline__ u32x4 fetch(int u, unsigned b, int k, unsigned& off) const {
        const unsigned e = entry(u), hh = (unsigned)u & 1u, j = (e >> 8) & 3u, pan = e & 127u, slot = j * 8u + (unsigned)P.rank;
        const unsigned lb = (unsigned)(size_t)lds + b;
        const unsigned a = lb + (unsigned)k * 1024u + (unsigned)lane * 16u;
        if (e & 0x8000u) {
            if (k == 0) {
                v2i g0, g1, g2, g3;
                const unsigned ra = lb + tr_off, wa = lb + sc_off;
                asm volatile("ds_read_b64_tr_b8 %0, %4\n\tds_read_b64_tr_b8 %1, %4 offset:512\n\t"
                             "ds_read_b64_tr_b8 %2, %4 offset:1024\n\tds_read_b64_tr_b8 %3, %4 offset:1536\n\ts_waitcnt lgkmcnt(0)\n\t"
                             "ds_write_b64 %5, %0\n\tds_write_b64 %5, %1 offset:8\n\tds_write_b64 %5, %2 offset:256\n\tds_write_b64 %5, %3 offset:264\n\t"
                             "s_waitcnt lgkmcnt(0)"
                             : "=&v"(g0), "=&v"(g1), "=&v"(g2), "=&v"(g3) : "v"(ra), "v"(wa) : "memory");
            }
            off = X_BYTES + slot * WS_SLAB + (pan * 16u + (unsigned)k * 8u) * 2048u + hh * 1024u + w_dst;
        } else if (e & 0x4000u) {
            off = fwd_off(u) + (unsigned)k * (unsigned)(N * 4) + (unsigned)lane * 16u;
        } else {
            off = slot * XS_SLAB + (pan * 16u + (unsigned)wid + (unsigned)k * 8u) * 2048u + hh * 1024u + (unsigned)lane * 16u;
        }
        u32x4 v;
        asm volatile("ds_read_b128 %0, %1\n\ts_waitcnt lgkmcnt(0)" : "=&v"(v) : "v"(a) : "memory");
        return v;
    }
    __device__ __forceinline__ unsigned flag_off(int u) const {
        const unsigned e = P.order[u >> 1], j = (e >> 8) & 3u, pan = e & 127u, slot = j * 8u + (unsigned)P.rank;
        return ((e & 0x8000u) ? FW_OFF : FX_OFF) + pan * 128u + slot * 4u;
    }
};
#define STORE_WT(off, v, base) asm volatile("s_nop 4\n\tglobal_store_dwordx4 %0, %1, %2 sc0 sc1\n\ts_nop 1" :: "v"(off), "v"(v), "s"(base) : "memory")

__device__ __forceinline__ void pusher(const Params& P, LAS unsigned char* lds, int wid, int lane, int pi) {
    const Mover mv(P, lds, wid, lane);
    const unsigned wbuf = (unsigned)wid * 4096u;
    mv.stage(pi, wbuf);
    __builtin_amdgcn_s_setprio(3);
    unsigned char* const wsl = P.ws[P.rank];
    unsigned call;
    {
        unsigned t = 0;
        if (lane == 0) t = __hip_atomic_fetch_add((unsigned*)(wsl + CTR_OFF) + pi * 16 + wid, 1u, __ATOMIC_RELAXED, __HIP_MEMORY_SCOPE_AGENT);
        call = rfl(t) + 1u;
    }
    unsigned* const tmo = (unsigned*)(wsl + TMO_OFF);
    unsigned char* dst[NR - 1];
    unsigned pbit[NR - 1];
    auto bases = [&](bool to_out) {
#pragma unroll
        for (int q = 0; q < NR - 1; ++q) {
            const unsigned p = (unsigned)(P.rank + 1 + (q + wid + pi) % (NR - 1)) & 7u;
            dst[q] = rfl_ptr(to_out ? P.out[p] : P.ws[p] + DATA_OFF + (size_t)(call & 1u) * PAR_BYTES);
            pbit[q] = 0x10000u << p;
        }
    };
    bases(false);
    const unsigned peer = (unsigned)(P.rank + 1 + lane) & 7u;
    unsigned char* flag_base = P.ws[peer];
    asm volatile("" : "+v"(flag_base));
    unsigned* const tick = (unsigned*)(wsl + TICK_OFF);
    const unsigned tick0 = (call - 1u) * (unsigned)NUNIT;
    const unsigned tbox = (unsigned)(size_t)lds + 65536u;
    auto signal = [&](int uu, int i) {
        if (uu < NUIN && wid == (i & 7)) {
            if (lane < NR - 1 && ((P.order[uu >> 1] >> 16) >> peer & 1u))
                __hip_atomic_fetch_add((unsigned*)(flag_base + mv.flag_off(uu)), 1u, __ATOMIC_RELAXED, __HIP_MEMORY_SCOPE_SYSTEM);
        }
    };
    bool fwd = false, bad = false;
    int uprev = -1, u = pi, unext = pi + NPUSH;
    unsigned nfwd = 0u;
    WAIT_V(0);
    for (int i = 0; u < NUNIT; ++i) {
        if (unext < NUNIT) {
            if (unext >= NUIN && !fwd) {
                fwd = true;
                bad |= wait_words((const unsigned*)(wsl + TDONE_OFF) + (lane < NOWN ? lane : 0), 8u * call, lane < NOWN, tmo, call, lane);
            }
            mv.stage(unext, wbuf + (unsigned)((i + 1) & 1) * 2048u);
        }
        if (wid == 0 && lane == 0) { const unsigned one = 1u; asm volatile("global_atomic_add v255, %0, %1, off sc0" :: "v"(tick), "v"(one) : "memory", "v255"); }
        if (u >= NUIN) { if (uprev < NUIN) bases(true); ++nfwd; }
        const unsigned e = mv.entry(u);
        const bool full = ((e | (0x10000u << P.rank)) >> 16) == 0xffu;
        int un2 = 0;
        auto meet = [&]() {
            if (wid == 0 && lane == 0) asm volatile("ds_write_b32 %0, v255\n\ts_waitcnt lgkmcnt(0)" :: "v"(tbox + (unsigned)(i & 1) * 4u) : "memory");
            BAR;
            unsigned t;
            asm volatile("ds_read_b32 %0, %1\n\ts_waitcnt lgkmcnt(0)" : "=&v"(t) : "v"(tbox + (unsigned)(i & 1) * 4u) : "memory");
            un2 = (int)(rfl(t) - tick0) + 2 * NPUSH;
            if (uprev >= 0) signal(uprev, i);
        };
        for (int k = 0; k < 2; ++k) {
            unsigned off;
            const u32x4 v = mv.fetch(u, wbuf + (unsigned)(i & 1) * 2048u, k, off);
#pragma unroll
            for (int q = 0; q < NR - 1; ++q) {
                if (e & pbit[q]) {
                    STORE_WT(off, v, dst[q]);
                    asm volatile("s_waitcnt vmcnt(%0)" :: "n"(PUSH_WIN) : "memory");
                }
                if (q == (PUSH_WIN - 1) % (NR - 1) && k == (PUSH_WIN - 1) / (NR - 1) && full) meet();
            }
        }
        if (!full) {
            const int c = __builtin_popcount((e & ~(0x10000u << P.rank)) >> 16);
            if (c >= 4) WAIT_V(8); else if (c >= 2) WAIT_V(4); else if (c == 1) WAIT_V(2); else WAIT_V(0);
            meet();
        }
        uprev = u; u = unext; unext = un2;
    }
    WAIT_V(0);
    BAR;
    signal(uprev, 0);
    if (wid == 0) {
        if (lane < NR - 1 && nfwd) __hip_atomic_fetch_add((unsigned*)(flag_base + DONE_OFF + (unsigned)P.rank * 128u), nfwd, __ATOMIC_RELAXED, __HIP_MEMORY_SCOPE_SYSTEM);
        if (pi == 0)
            bad |= wait_words((const unsigned*)(wsl + DONE_OFF + peer * 128u), (unsigned)(NUNIT - NUIN) * call, lane < NR - 1, tmo, call, lane);
    }
    if (bad && lane == 0) __hip_atomic_store((unsigned*)P.out[P.rank], 0x7fc00000u, __ATOMIC_RELAXED, __HIP_MEMORY_SCOPE_SYSTEM);
}
static_assert(PUSH_WIN >= 1 && PUSH_WIN <= 2 * (NR - 1), "window");

__device__ __forceinline__ void local_copy(const Params& P, const Mover& mv, unsigned call, int wid, int lane, int li) {
    const int nun = li + 2 * NGEMM < NUIN ? 3 : 2;
    const unsigned wbuf = (unsigned)wid * 8192u;
    for (int i = 0; i < nun; ++i) mv.stage(li + NGEMM * i, wbuf + (unsigned)i * 2048u);
    unsigned char* const wsl = P.ws[P.rank];
    unsigned char* const dst = rfl_ptr(wsl + DATA_OFF + (size_t)(call & 1u) * PAR_BYTES);
    WAIT_V(0);
    for (int i = 0; i < nun; ++i)
        for (int k = 0; k < 2; ++k) {
            unsigned off;
            const u32x4 v = mv.fetch(li + NGEMM * i, wbuf + (unsigned)i * 2048u, k, off);
            STORE_WT(off, v, dst);
        }
    WAIT_V(0);
    __syncthreads();
    if (wid < nun && lane == 0) __hip_atomic_fetch_add((unsigned*)(wsl + mv.flag_off(li + NGEMM * wid)), 1u, __ATOMIC_RELAXED, __HIP_MEMORY_SCOPE_SYSTEM);
}

__device__ __forceinline__ void gemm_role(const Params& P, LAS unsigned char* lds, const int wid, const int lane) {
    unsigned char* const wsl = P.ws[P.rank];
    if (wid == 0 && lane == 0) {
        const unsigned xcc = (unsigned)__builtin_amdgcn_s_getreg((3 << 11) | 20) & 7u;
        const unsigned c = __hip_atomic_fetch_add((unsigned*)(wsl + CTR_OFF + blockIdx.x * 64u), 1u, __ATOMIC_RELAXED, __HIP_MEMORY_SCOPE_AGENT) + 1u;
        unsigned rl = 256u;
        for (unsigned k = 0; k < 8u && rl == 256u; ++k) {
            const unsigned x = (xcc + k) & 7u;
            u64* const ctr = (u64*)(wsl + ROLE_OFF + x * 128u);
            const u64 seen = __hip_atomic_fetch_max(ctr, (u64)c << 32, __ATOMIC_RELAXED, __HIP_MEMORY_SCOPE_AGENT);
            const u64 t = __hip_atomic_fetch_add(ctr, 1ull + (seen >> 63), __ATOMIC_RELAXED, __HIP_MEMORY_SCOPE_AGENT);
            if ((unsigned)(t >> 32) == c && (unsigned)t < 30u) rl = x * 30u + (unsigned)t;
        }
        *(LAS v2i*)(lds + LDS_MBOX) = (v2i){(int)c, (int)rl};
    }
    __syncthreads();
    const v2i cr = *(const LAS v2i*)(lds + LDS_MBOX);
    const unsigned call = rfl((unsigned)cr.x);
    const int role = (int)rfl((unsigned)cr.y);
    if (role >= NGEMM) return;
    const int band = role / 60, i60 = role - band * 60, g = i60 >> 2, prow = band * 4 + (i60 & 3);
    { const Mover mv(P, lds, wid, lane); local_copy(P, mv, call, wid, lane, role); }

    u64 tiles = 0;
    constexpr int nseg = 4;
    for (int r = 0; r < 4; ++r) {
        const int sidx = r * 15 + g;
        const unsigned t = sidx == SPSLOT ? slot_tile((unsigned)P.rank, (unsigned)prow) | (prow < NOWN ? (unsigned)(prow + 1) << 10 : 0u)
                                          : (unsigned)prow | (unsigned)(sidx - (sidx > SPSLOT)) << 4;
        tiles |= (u64)t << (16 * r);
    }
    auto tile_lane = [&](int sg) { return (unsigned)(tiles >> (16 * (sg & 3))) & 0xffffu; };
    auto tile_of = [&](int sg) { return rfl(tile_lane(sg)); };
    const int wr = wid >> 2, wc = wid & 3;
    const unsigned thr = 2u * call;
    unsigned char* const dbase = rfl_ptr(wsl + DATA_OFF + (size_t)(call & 1u) * PAR_BYTES);
    const __amdgpu_buffer_rsrc_t rsX = __builtin_amdgcn_make_buffer_rsrc((void*)dbase, 0, X_BYTES, 0x00020000);
    const __amdgpu_buffer_rsrc_t rsW = __builtin_amdgcn_make_buffer_rsrc((void*)(dbase + X_BYTES), 0, W_BYTES, 0x00020000);
    const __amdgpu_buffer_rsrc_t rsF = __builtin_amdgcn_make_buffer_rsrc((void*)wsl, 0, DATA_OFF, 0x00020000);
    const unsigned voff = (unsigned)(wid * 2048 + lane * 16);
    const unsigned ldsw = (unsigned)wid * 2048u;
    int aoff = wr * 8192 + lane * 16, boff = LDS_B + wc * 4096 + lane * 16;
    asm volatile("" : "+v"(aoff), "+v"(boff));
    const unsigned snap_src = (lane < 32 ? FW_OFF + ((tile_lane(lane >> 3) >> 4) & 63u) * 128u : FX_OFF + (tile_lane(lane >> 3) & 15u) * 128u) + (unsigned)(lane & 7) * 16u;
    const unsigned snap_rd = lane < 32 ? (unsigned)lane * 4u : 384u + (unsigned)lane * 4u;

#define DMA(rs, ldsoff, vo, so, imm) __builtin_amdgcn_raw_ptr_buffer_load_lds(rs, (LAS void*)(lds + (ldsoff)), 16, vo, so, imm, 17)
#define STAGE_A(b, h, so) do { DMA(rsX, ((b) * 2 + (h)) * HT + ldsw, voff, (so) + (h) * HT, 0); DMA(rsX, ((b) * 2 + (h)) * HT + ldsw, voff, (so) + (h) * HT, 1024); } while (0)
#define STAGE_B(b, h, so) do { DMA(rsW, LDS_B + ((b) * 2 + (h)) * HT + ldsw, voff, (so) + (h) * HT, 0); DMA(rsW, LDS_B + ((b) * 2 + (h)) * HT + ldsw, voff, (so) + (h) * HT, 1024); } while (0)
#define LDF(off) __builtin_shufflevector(*(const LAS v4i*)(lds + (off)), *(const LAS v4i*)(lds + (off) + 1024), 0, 1, 2, 3, 4, 5, 6, 7)
#define LDA(b, h) do { _Pragma("unroll") for (int m = 0; m < 4; ++m) At[m] = LDF(((b) * 2 + (h)) * HT + aoff + m * 2048); } while (0)
#define LDB(dst, b, h) do { _Pragma("unroll") for (int n = 0; n < 2; ++n) dst[n] = LDF(((b) * 2 + (h)) * HT + boff + n * 2048); } while (0)
#define MF(c, bb, aa) asm volatile("v_mfma_f32_16x16x128_f8f6f4 %0, %1, %2, %0 cbsz:1 blgp:1" : "+v"(c) : "v"(aa), "v"(bb))

    f32x4 acc[8][4];
    v8i At[4], B0[2], B1[2];
#pragma unroll
    for (int m = 0; m < 8; ++m)
#pragma unroll
        for (int n = 0; n < 4; ++n) acc[m][n] = (f32x4){0.f, 0.f, 0.f, 0.f};

    unsigned xm = 0u, wm = 0u, used = 0u, bad = 0u;
    int ts = 0;
    unsigned a_pan = (tile_of(0) & 15u) * 32768u, b_pan = ((tile_of(0) >> 4) & 63u) * 32768u;
    auto slow_wait = [&]() {
        if (wid == 0) {
            const unsigned* fp = (const unsigned*)(wsl + (lane < 32 ? FW_OFF + (b_pan >> 8) + (unsigned)lane * 4u : FX_OFF + (a_pan >> 8) + (unsigned)(lane - 32) * 4u));
            unsigned* tmo = (unsigned*)(wsl + TMO_OFF);
            const u64 t0 = __builtin_amdgcn_s_memrealtime();
            u64 mk; unsigned dead = 0u;
            for (;;) {
                const unsigned v = __hip_atomic_load(fp, __ATOMIC_RELAXED, __HIP_MEMORY_SCOPE_SYSTEM);
                mk = __builtin_amdgcn_ballot_w64((int)(v - thr) >= 0);
                if (__builtin_popcount((unsigned)mk & (unsigned)(mk >> 32) & ~used) >= 2) break;
                if (rfl(__hip_atomic_load(tmo, __ATOMIC_RELAXED, __HIP_MEMORY_SCOPE_AGENT)) == call || __builtin_amdgcn_s_memrealtime() - t0 > DEADLINE_TICKS) {
                    if (lane == 0) __hip_atomic_store(tmo, call, __ATOMIC_RELAXED, __HIP_MEMORY_SCOPE_AGENT);
                    mk = ~0ull; dead = 1u; break;
                }
                __builtin_amdgcn_s_sleep(4);
            }
            if (lane == 0) *(LAS u32x4*)(lds + LDS_MBOX + 16) = (u32x4){(unsigned)mk, (unsigned)(mk >> 32), dead, 0u};
            asm volatile("s_waitcnt vmcnt(0) lgkmcnt(0)" ::: "memory");
        }
        asm volatile("" ::: "memory"); BAR; asm volatile("" ::: "memory");
        const u32x4 mb = *(const LAS u32x4*)(lds + LDS_MBOX + 16);
        wm |= rfl(mb.x); xm |= rfl(mb.y); bad |= rfl(mb.z);
    };
    unsigned a2, b2, a3, b3;
    auto take = [&](unsigned av) {
        const unsigned sA = (unsigned)__builtin_ctz(av), sB = (unsigned)__builtin_ctz(av & (av - 1u));
        used |= (1u << sA) | (1u << sB);
        a2 = sA * XS_SLAB + a_pan; b2 = sA * WS_SLAB + b_pan; a3 = sB * XS_SLAB + a_pan; b3 = sB * WS_SLAB + b_pan;
    };

    const float sl = __builtin_bit_cast(float, rfl(__builtin_bit_cast(unsigned, (P.sx[0] * P.sw[0]) * -1.44269504089f)));
    slow_wait(); take(xm & wm);
    unsigned a1 = a3, b1 = b3;
    STAGE_A(0, 0, a2); STAGE_B(0, 0, b2); STAGE_B(0, 1, b2); STAGE_A(0, 1, a2);
    STAGE_A(1, 0, a3); STAGE_B(1, 0, b3);
    WAIT_V(4); BAR;
    if (wr == 1) BAR;
    bool ready;
    { const unsigned av = xm & wm & ~used; ready = (av & (av - 1u)) != 0u; if (ready) take(av); }
#define MMA2(ai) do { __builtin_amdgcn_s_setprio(1); _Pragma("unroll") for (int m = 0; m < 4; ++m) { \
        MF(acc[ai * 4 + m][0], B0[0], At[m]); MF(acc[ai * 4 + m][1], B0[1], At[m]); MF(acc[ai * 4 + m][2], B1[0], At[m]); MF(acc[ai * 4 + m][3], B1[1], At[m]); } \
        __builtin_amdgcn_s_setprio(0); } while (0)
    for (int seg = 0; seg < nseg; ++seg) {
        for (int it = 0; it < 16; ++it) {
            if (!ready) { slow_wait(); take(xm & wm & ~used); }
            const unsigned snap = LDS_SNAP + (unsigned)(it & 1) * 1024u;
            LDB(B0, 0, 0); LDA(0, 0); LDB(B1, 0, 1); STAGE_B(1, 1, b1); STAGE_A(1, 1, a1);
            if (wid == 0) DMA(rsF, snap, snap_src, 0, 0);
            WAIT_L(0); BAR; SCHED; MMA2(0); BAR; SCHED;
            LDA(0, 1); STAGE_A(0, 0, a2); STAGE_B(0, 0, b2);
            WAIT_V(4); WAIT_L(0); BAR; SCHED; MMA2(1); BAR; SCHED;
            LDB(B0, 1, 0); LDA(1, 0); LDB(B1, 1, 1); STAGE_B(0, 1, b2); STAGE_A(0, 1, a2);
            const unsigned snapv = *(const LAS unsigned*)(lds + snap + (unsigned)((it == 14 ? ts + 1 : ts) & 3) * 128u + snap_rd);
            WAIT_L(0); BAR; SCHED; MMA2(0); BAR; SCHED;
            const u64 mk = __builtin_amdgcn_ballot_w64((int)(snapv - thr) >= 0);
            SCHED;
            LDA(1, 1); STAGE_A(1, 0, a3); STAGE_B(1, 0, b3);
            a1 = a3; b1 = b3;
            if (it == 14) {
                ++ts; wm = 0u; xm = 0u; used = 0u;
                const unsigned t = tile_of(ts);
                a_pan = (t & 15u) * 32768u; b_pan = ((t >> 4) & 63u) * 32768u;
            }
            wm |= (unsigned)mk; xm |= (unsigned)(mk >> 32);
            if (ts >= nseg) { wm = xm = ~0u; used = 0u; }
            { const unsigned av = xm & wm & ~used; ready = (av & (av - 1u)) != 0u; if (ready) take(av); }
            WAIT_V(4); WAIT_L(0); BAR; SCHED; MMA2(1); BAR; SCHED;
        }
        SCHED; asm volatile("s_nop 15\n\ts_nop 15" ::: "memory"); SCHED;
        const unsigned tcur = tile_of(seg);
        {
            int tz = threadIdx.x; asm volatile("" : "+v"(tz));
            const unsigned ov = (unsigned)((tz >> 4) & 3) * (unsigned)(4 * N * 4) + (unsigned)(tz & 15) * 16u;
            const char* ob = (const char*)rfl_ptr(P.out[P.rank] + (size_t)((tcur & 15u) * 256u + (unsigned)wr * 64u) * (N * 4) + (((tcur >> 4) & 63u) * 256u + (unsigned)wc * 64u) * 4u);
            asm volatile("s_nop 4" : "+s"(ob));
            const float cc = bad ? __builtin_nanf("") : -1.44269504089f;
#pragma unroll
            for (int m = 0; m < 8; ++m) {
                const char* rowp = ob + (size_t)((m >> 2) * 128 + (m & 3) * 16) * (N * 4);
#pragma unroll
                for (int j = 0; j < 4; ++j) {
                    const f32x4 o = silu4(acc[m][0][j], acc[m][1][j], acc[m][2][j], acc[m][3][j], sl, cc);
                    asm volatile("global_store_dwordx4 %0, %1, %2" OUT_POL "\n\ts_nop 1" :: "v"(ov), "v"(o), "s"(rowp + (size_t)j * (N * 4)) : "memory");
                }
#pragma unroll
                for (int n = 0; n < 4; ++n) acc[m][n] = (f32x4){0.f, 0.f, 0.f, 0.f};
                if (m == 0 && wr == 0) BAR;
            }
        }
        if (tcur >> 10) {
            WAIT_V(0);
            if (lane == 0) __hip_atomic_fetch_add((unsigned*)(wsl + TDONE_OFF) + ((tcur >> 10) - 1u), 1u, __ATOMIC_RELAXED, __HIP_MEMORY_SCOPE_AGENT);
        }
        if (wr == 1) BAR;
    }
    WAIT_V(0);
    if (wr == 0) BAR;
}

__global__ __launch_bounds__(NTHR, 2) void fused_kernel(const Params P) {
    extern __shared__ __attribute__((aligned(16))) unsigned char shm[];
    LAS unsigned char* lds = (LAS unsigned char*)shm;
    const int wid = __builtin_amdgcn_readfirstlane(threadIdx.x >> 6), lane = threadIdx.x & 63;
    if (blockIdx.x < NPUSH) pusher(P, lds, wid, lane, blockIdx.x);
    else gemm_role(P, lds, wid, lane);
}

void build_order(int s, unsigned* out) {
    struct It { double key; unsigned code; };
    It it[NCHUNK]; int n = 0;
    const double tau0 = 2.0;
    const double Tr = 45.0;
    const double xs = 11.0;
    const double bias = 6.0;
    for (int j = 0; j < 4; ++j)
        for (int p = 0; p < 16; ++p)
            it[n++] = It{8 * j * tau0 + xs * tau0 * (((p & 3) * 4 + (((p >> 2) + s) & 3)) / 16.0) - 0.01, 0xff0000u | (unsigned)((j << 8) | p)};
    for (int c = 0; c < 64; ++c) {
        const int sidx = c >= DPAN ? SPSLOT : c + (c >= SPSLOT), r = sidx / 15, g = sidx % 15;
        const unsigned mask = c > DPAN ? 3u << (2 * (c - DPAN - 1)) : 0xffu;
        for (int j = 0; j < 4; ++j) {
            const int pos = 8 * j + ((s + g * 3 + (g >> 3) * 4) & 7);
            const double key = (g >= 7 ? 0.0 : bias) + (r == 0 ? pos * tau0 : 32 * tau0 + (r - 1) * Tr + pos * (Tr / 34.0)) + 1e-4 * c;
            it[n++] = It{key, mask << 16 | 0x8000u | (unsigned)((j << 8) | c)};
        }
    }
    std::stable_sort(it, it + NCHUNK, [](const It& a, const It& b) { return a.key < b.key; });
    for (int i = 0; i < NCHUNK; ++i) out[i] = it[i].code;
}
}

extern "C" void kernel_launch(int rank, int nranks, ncclComm_t comm,
                              hipStream_t stream,
                              void* const* d_in, const int* in_sizes, int n_in,
                              void* d_out, int out_size,
                              const int* in_dtypes, int out_dtype,
                              void* const* d_out_all,
                              void* const* d_ws_all, size_t ws_bytes) {
    (void)comm; (void)in_dtypes; (void)out_dtype;
    if (nranks != NR || (unsigned)rank >= (unsigned)NR || n_in != 4 || ws_bytes < WS_NEED || out_size != M * N ||
        in_sizes[0] != M * KL || in_sizes[1] != KL * N) {
        fprintf(stderr, "kernel_launch: unexpected problem shape\n");
        return;
    }
    Params P;
    P.x = (const unsigned char*)d_in[0]; P.w = (const unsigned char*)d_in[1];
    P.sx = (const float*)d_in[2]; P.sw = (const float*)d_in[3];
    for (int p = 0; p < NR; ++p) { P.ws[p] = (unsigned char*)d_ws_all[p]; P.out[p] = (unsigned char*)d_out_all[p]; }
    P.out[rank] = (unsigned char*)d_out;
    P.rank = rank; P.pad = 0;
    build_order(rank, P.order);
    HIP_CHECK(hipFuncSetAttribute((const void*)fused_kernel, hipFuncAttributeMaxDynamicSharedMemorySize, LDS_BYTES));
    hipLaunchKernelGGL(fused_kernel, dim3(NWG), dim3(NTHR), LDS_BYTES, stream, P);
    HIP_CHECK(hipGetLastError());
}
```
